```python
import numpy as np

kernel_name = "gpu_gemm_rs_m4096_k4096_n4096_e4m3e5m2_dqrelu"


ABSMAX_RTOL = 0.05

COMPUTE_DTYPE = "float32"
EPILOGUE = "dequant_relu"
IN_QUANT = None
OUT_QUANT = None
IN_FP8 = "e4m3e5m2"
IN_DTYPES = {"x": "e4m3", "w_mat": "e5m2",
             "scale_x": "float32", "scale_w": "float32"}
WORLD_SIZE = 8

def _snap_fp8(x, mant_bits, max_abs, min_normal):
    a = np.abs(x.astype(np.float64))
    _m, _e = np.frexp(a)
    q = np.where(a >= min_normal, np.ldexp(1.0, _e - 1 - mant_bits),
                 min_normal * 2.0 ** -mant_bits)
    snapped = np.minimum(np.round(a / q) * q, max_abs)
    return (np.sign(x) * snapped).astype(np.float32)

def setup_inputs():

    x = _snap_fp8(np.random.randn(4096, 4096).astype(np.float32), 3, 448.0, 2.0 ** -6)
    w_mat = _snap_fp8(np.random.randn(4096, 4096).astype(np.float32), 2, 57344.0, 2.0 ** -14)
    scale_x = (0.02 * (1.0 + np.random.rand(1))).astype(np.float32)
    scale_w = (0.02 * (1.0 + np.random.rand(1))).astype(np.float32)
    return {"x": x, "w_mat": w_mat, "scale_x": scale_x, "scale_w": scale_w}

def reference(x, w_mat, scale_x, scale_w):
    acc = (x @ w_mat).astype(np.float32)
    y = (acc * (scale_x * scale_w)).astype(np.float32)
    return np.maximum(y, 0.0).astype(np.float32)
```

```cpp
#include <hip/hip_runtime.h>
#include <rccl/rccl.h>
#include <cstdio>

#define HIP_CHECK(expr) do { hipError_t _e = (expr); if (_e != hipSuccess) { \
    fprintf(stderr, "HIP error %s at %s:%d\n", hipGetErrorString(_e), __FILE__, __LINE__); \
    return; } } while (0)

namespace {

typedef unsigned u32;
typedef unsigned long long u64;
typedef unsigned u32x4 __attribute__((ext_vector_type(4)));
typedef int i32x2 __attribute__((ext_vector_type(2)));
typedef int i32x8 __attribute__((ext_vector_type(8)));
typedef float f32x4 __attribute__((ext_vector_type(4)));
typedef float f32x16 __attribute__((ext_vector_type(16)));
#define LAS __attribute__((address_space(3)))
#define GAS __attribute__((address_space(1)))

constexpr int R = 8;
constexpr int MS = 512;
constexpr int KS = 512;
constexpr int NN = 4096;
constexpr int NWG = 256;
constexpr int NLD = 3;
constexpr int NTHREADS = 512;
constexpr int RR = 4;
constexpr int PIECE = 4096;
constexpr int LDS_RING = RR * 8 * PIECE;
constexpr int LDS_READY = LDS_RING;
constexpr int LDS_DONE = LDS_RING + 128;
constexpr int LDS_BYTES = LDS_RING + 256;

constexpr size_t WS_CNT = 0;
constexpr size_t WS_TMO = 32768;
constexpr size_t WS_FX = 65536;
constexpr size_t WS_FW = 131072;
constexpr size_t WS_DBG = 262144;
constexpr size_t WS_XB = 1ull << 20;
constexpr size_t XB_PAR = 2ull << 20;
constexpr size_t WS_WB = 8ull << 20;
constexpr size_t WB_PAR = 16ull << 20;
constexpr size_t WS_END = WS_WB + 2 * WB_PAR;

constexpr u64 DEADLINE_TICKS = 40000000ull;

struct Params {
  const unsigned char* x;
  const unsigned char* w;
  const float* sx;
  const float* sw;
  float* out;
  unsigned char* peer[R];
  int rank;
  int pad;
};

__device__ __forceinline__ u64 rt() { return __builtin_amdgcn_s_memrealtime(); }

#define STAMP(slot) do {} while (0)

#define ST16(base, voff, val, imm) \
  asm volatile("global_store_dwordx4 %0, %1, %2 offset:" #imm " sc0 sc1\n\ts_nop 1" :: "v"(voff), "v"(val), "s"(base) : "memory")

__device__ __forceinline__ void flag_store(unsigned char* p, u32 v) {
  __hip_atomic_store((GAS u32*)p, v, __ATOMIC_RELAXED, __HIP_MEMORY_SCOPE_SYSTEM);
}
__device__ __forceinline__ u32 flag_load(const unsigned char* p) {
  return __hip_atomic_load((GAS u32*)p, __ATOMIC_RELAXED, __HIP_MEMORY_SCOPE_SYSTEM);
}
__device__ __forceinline__ u32 tmo_load(unsigned char* ws) {
  return __hip_atomic_load((GAS u32*)(ws + WS_TMO), __ATOMIC_RELAXED, __HIP_MEMORY_SCOPE_AGENT);
}
__device__ __forceinline__ void tmo_store(unsigned char* ws, u32 call) {
  __hip_atomic_store((GAS u32*)(ws + WS_TMO), call, __ATOMIC_RELAXED, __HIP_MEMORY_SCOPE_AGENT);
}

__device__ __forceinline__ i32x8 read_bfrag(u32 addr, int imm) {
  LAS unsigned char* b = (LAS unsigned char*)(size_t)addr;
  i32x2 r0 = __builtin_amdgcn_ds_read_tr8_b64_v2i32((LAS i32x2*)(b + imm));
  i32x2 r1 = __builtin_amdgcn_ds_read_tr8_b64_v2i32((LAS i32x2*)(b + imm + 256));
  i32x2 r2 = __builtin_amdgcn_ds_read_tr8_b64_v2i32((LAS i32x2*)(b + imm + 512));
  i32x2 r3 = __builtin_amdgcn_ds_read_tr8_b64_v2i32((LAS i32x2*)(b + imm + 768));
  i32x8 r;
  r[0] = r0.x; r[1] = r0.y; r[2] = r1.x; r[3] = r1.y; r[4] = r2.x; r[5] = r2.y; r[6] = r3.x; r[7] = r3.y;
  return r;
}

__global__ void __launch_bounds__(NTHREADS, 2) fused_kernel(Params P) {
  extern __shared__ __attribute__((aligned(16))) unsigned char lds_raw[];
  LAS unsigned char* L = (LAS unsigned char*)lds_raw;
  volatile LAS u32* READY = (volatile LAS u32*)(L + LDS_READY);
  volatile LAS u32* DONE = (volatile LAS u32*)(L + LDS_DONE);

  const int c = blockIdx.x;
  const int lane = threadIdx.x & 63;
  const int wave = __builtin_amdgcn_readfirstlane(threadIdx.x >> 6);
  const int u = (c & 7) * 8 + ((c >> 3) & 7);
  const int mt = c >> 6;
  unsigned char* const ws = P.peer[0];
  STAMP(0);

  GAS u32* cntp = (GAS u32*)(ws + WS_CNT + c * 128 + wave * 4);
  const u32 cnt_old = __hip_atomic_load(cntp, __ATOMIC_RELAXED, __HIP_MEMORY_SCOPE_AGENT);
  if (wave == 4) ((LAS u32*)(L + LDS_RING))[lane] = 0u;

  if (wave < 4) {
    const int f = c * 4 + wave, xd = f >> 7;
    u32x4 xv0, xv1;
    if (xd != 0) {
      const int t = (P.rank + xd) & 7, mb = (f >> 3) & 15, i = f & 7;
      const GAS u32x4* src = (const GAS u32x4*)(P.x + (size_t)(t * MS + mb * 32 + (lane & 31)) * KS + i * 64 + 32 * (lane >> 5));
      xv0 = src[0]; xv1 = src[1];
    }
    const u32 call = __builtin_amdgcn_readfirstlane(cnt_old) + 1;
    if (lane == 0) __hip_atomic_store(cntp, call, __ATOMIC_RELAXED, __HIP_MEMORY_SCOPE_AGENT);
    const u32 par = call & 1;
    if (xd != 0) {
      const int mb = (f >> 3) & 15, i = f & 7;
      unsigned char* pd = P.peer[xd];
      unsigned char* dst = pd + WS_XB + par * XB_PAR + (size_t)((xd * 16 + mb) * 8 + i) * 2048;
      const u32 vo = lane * 16;
      ST16(dst, vo, xv0, 0);
      ST16(dst, vo, xv1, 1024);
      __builtin_amdgcn_s_barrier();
      asm volatile("s_waitcnt vmcnt(0)" ::: "memory");
      if (lane == 0) flag_store(pd + WS_FX + (mb * 8 + xd) * 128 + i * 4, call);
    } else {
      __builtin_amdgcn_s_barrier();
    }
    STAMP(1);
    const int mymb = mt * 4 + wave;
    {
      const int dd = 1 + (lane < 56 ? lane : 55) / 8;
      const unsigned char* fp = ws + WS_FX + (mymb * 8 + dd) * 128 + (lane & 7) * 4;
      const u64 t0 = rt();
      while (true) {
        const u32 v = flag_load(fp);
        if (__all(v >= call)) break;
        if (tmo_load(ws) == call || rt() - t0 > DEADLINE_TICKS) { tmo_store(ws, call); break; }
        __builtin_amdgcn_s_sleep(2);
      }
      asm volatile("" ::: "memory");
    }
    STAMP(2);
    const auto rs_x = __builtin_amdgcn_make_buffer_rsrc((void*)P.x, 0, 4096 * KS, 0x00020000);
    const auto rs_xb = __builtin_amdgcn_make_buffer_rsrc((void*)(ws + WS_XB + par * XB_PAR), 0, (int)XB_PAR, 0x00020000);
    const u32 vo_own = (u32)(P.rank * MS + mymb * 32 + (lane & 31)) * KS + 32 * (lane >> 5);
    const u32 vo_rem = (u32)mymb * 16384 + lane * 16;
    const u32 b_lane = (u32)(size_t)L + ((lane >> 5) * 32 + ((lane & 15) >> 1)) * 32 + ((lane >> 4) & 1) * 16 + (lane & 1) * 8;

    f32x16 acc0 = {}, acc1 = {};
    i32x8 A0[8], A1[8];
    auto loadA = [&](i32x8 (&A)[8], int i) {
      {
        u32x4 a = __builtin_amdgcn_raw_buffer_load_b128(rs_x, vo_own, i * 64, 0);
        u32x4 b = __builtin_amdgcn_raw_buffer_load_b128(rs_x, vo_own + 16, i * 64, 0);
        A[0][0] = a.x; A[0][1] = a.y; A[0][2] = a.z; A[0][3] = a.w; A[0][4] = b.x; A[0][5] = b.y; A[0][6] = b.z; A[0][7] = b.w;
      }
#pragma unroll
      for (int d = 1; d < 8; ++d) {
        u32x4 a = __builtin_amdgcn_raw_buffer_load_b128(rs_xb, vo_rem, d * 262144 + i * 2048, 17);
        u32x4 b = __builtin_amdgcn_raw_buffer_load_b128(rs_xb, vo_rem + 1024, d * 262144 + i * 2048, 17);
        A[d][0] = a.x; A[d][1] = a.y; A[d][2] = a.z; A[d][3] = a.w; A[d][4] = b.x; A[d][5] = b.y; A[d][6] = b.z; A[d][7] = b.w;
      }
    };
    auto round = [&](i32x8 (&A)[8], int i) {
      const int sb = (i & (RR - 1)) * 8;
      const u32 baddr = b_lane + sb * PIECE;
      u32 rem = 0xFFu;
      while (rem) {
        const u32 v = READY[sb + (lane & 7)];
        u32 m = (u32)__ballot(v >= (u32)(i + 1)) & rem;
        if (!m) { __builtin_amdgcn_s_sleep(1); continue; }
        asm volatile("" ::: "memory");
        rem &= ~m;
#define STEP(p) if (m & (1u << p)) { \
          i32x8 b0 = read_bfrag(baddr, p * PIECE), b1 = read_bfrag(baddr, p * PIECE + 2048); \
          acc0 = __builtin_amdgcn_mfma_scale_f32_32x32x64_f8f6f4(b0, A[p], acc0, 1, 0, 0, 0, 0, 0); \
          acc1 = __builtin_amdgcn_mfma_scale_f32_32x32x64_f8f6f4(b1, A[p], acc1, 1, 0, 0, 0, 0, 0); }
        STEP(0) STEP(1) STEP(2) STEP(3) STEP(4) STEP(5) STEP(6) STEP(7)
#undef STEP
      }
      asm volatile("s_waitcnt lgkmcnt(0)" ::: "memory");
      if (lane == 0) DONE[wave] = (u32)(i + 1);
    };
    loadA(A0, 0);
#pragma unroll 1
    for (int i = 0; i < 8; i += 2) {
      loadA(A1, i + 1);
      round(A0, i);
      STAMP(8 + i);
      if (i + 2 < 8) loadA(A0, i + 2);
      round(A1, i + 1);
      STAMP(9 + i);
    }
    const float s = P.sx[0] * P.sw[0];
    const bool bad = tmo_load(ws) == call;
    const float poison = __builtin_nanf("");
    GAS f32x4* o = (GAS f32x4*)(P.out + (size_t)(mymb * 32 + (lane & 31)) * NN + u * 64 + 4 * (lane >> 5));
#pragma unroll
    for (int q = 0; q < 4; ++q) {
      f32x4 v0, v1;
#pragma unroll
      for (int e = 0; e < 4; ++e) {
        v0[e] = bad ? poison : fmaxf(acc0[4 * q + e] * s, 0.f);
        v1[e] = bad ? poison : fmaxf(acc1[4 * q + e] * s, 0.f);
      }
      o[2 * q] = v0;
      o[8 + 2 * q] = v1;
    }
    STAMP(3);
  } else if (wave < 4 + NLD) {
    const u32 call = __builtin_amdgcn_readfirstlane(cnt_old) + 1;
    if (lane == 0) __hip_atomic_store(cntp, call, __ATOMIC_RELAXED, __HIP_MEMORY_SCOPE_AGENT);
    const u32 par = call & 1;
    asm volatile("s_waitcnt lgkmcnt(0)" ::: "memory");
    __builtin_amdgcn_s_barrier();
    const int j = wave - 4;
    constexpr int NS = (8 + NLD - 1) / NLD;
    int fs = lane >> 2; if (fs >= NS) fs = 0;
    int fd = j + fs * NLD; if (fd >= 8) fd = j;
    const unsigned char* fp = ws + WS_FW + (u * 8 + fd) * 128 + (lane & 3) * 4;
    const unsigned char* wb = ws + WS_WB + par * WB_PAR + (size_t)u * 8 * PIECE + lane * 16;
    const unsigned char* wown = P.w + (size_t)(lane >> 1) * NN + u * 64 + (lane & 1) * 16;
    const u32 fbase = call * 16;
    int nxt[NS], pub[NS];
#pragma unroll
    for (int s = 0; s < NS; ++s) nxt[s] = pub[s] = (j + s * NLD < 8) ? 0 : 8;
    u64 t0 = rt();
    bool dead = false;
    while (true) {
      const u32 fv = flag_load(fp);
      const u32 dv = DONE[lane & 3];
      u32 dmin = min(min(__builtin_amdgcn_readlane(dv, 0), __builtin_amdgcn_readlane(dv, 1)),
                     min(__builtin_amdgcn_readlane(dv, 2), __builtin_amdgcn_readlane(dv, 3)));
      asm volatile("" ::: "memory");
      bool progress = false, all = true;
#pragma unroll
      for (int s = 0; s < NS; ++s) {
        const int d = j + s * NLD;
        while (nxt[s] < 8 && nxt[s] < (int)dmin + RR) {
          const int i = nxt[s];
          LAS unsigned char* dst = L + ((i & (RR - 1)) * 8 + d) * PIECE;
          if (d == 0) {
            const unsigned char* src = wown + (size_t)i * 64 * NN;
            __builtin_amdgcn_global_load_lds((const GAS u32*)(src), (LAS u32*)(dst), 16, 0, 0);
            __builtin_amdgcn_global_load_lds((const GAS u32*)(src + 32 * NN), (LAS u32*)(dst + 1024), 16, 0, 0);
            __builtin_amdgcn_global_load_lds((const GAS u32*)(src + 32), (LAS u32*)(dst + 2048), 16, 0, 0);
            __builtin_amdgcn_global_load_lds((const GAS u32*)(src + 32 * NN + 32), (LAS u32*)(dst + 3072), 16, 0, 0);
          } else {
            if (!dead) {
              if (__builtin_amdgcn_readlane(fv, s * 4 + ((d - 1 - i) & 3)) < fbase + i + 1) break;
              const unsigned char* src = wb + (size_t)(d * 64 * 8 + i) * PIECE;
              __builtin_amdgcn_global_load_lds((const GAS u32*)(src), (LAS u32*)(dst), 16, 0, 17);
              __builtin_amdgcn_global_load_lds((const GAS u32*)(src), (LAS u32*)(dst), 16, 1024, 17);
              __builtin_amdgcn_global_load_lds((const GAS u32*)(src), (LAS u32*)(dst), 16, 2048, 17);
              __builtin_amdgcn_global_load_lds((const GAS u32*)(src), (LAS u32*)(dst), 16, 3072, 17);
            }
          }
          nxt[s] = i + 1;
          progress = true;
        }
        all = all && nxt[s] == 8;
      }
      if (progress) {
        asm volatile("s_waitcnt vmcnt(0)" ::: "memory");
#pragma unroll
        for (int s = 0; s < NS; ++s) {
          const int d = j + s * NLD;
          for (int i = pub[s]; i < nxt[s]; ++i) {
            if (lane == 0) READY[(i & (RR - 1)) * 8 + d] = (u32)(i + 1);
          }
          pub[s] = nxt[s];
        }
        if (all) break;
        t0 = rt();
      } else {
        if (!dead && (tmo_load(ws) == call || rt() - t0 > DEADLINE_TICKS)) { tmo_store(ws, call); dead = true; }
        __builtin_amdgcn_s_sleep(2);
      }
    }
    STAMP(3);
  } else {
    const int g = mt;
    u32x4 D[8][4];
    const unsigned char* wown = P.w + (size_t)(lane >> 1) * NN + u * 64 + (lane & 1) * 16;
#pragma unroll
    for (int i = 0; i < 8; ++i) {
      const unsigned char* src = wown + (size_t)i * 64 * NN;
      D[i][0] = *(const GAS u32x4*)(src);
      D[i][1] = *(const GAS u32x4*)(src + 32 * NN);
      D[i][2] = *(const GAS u32x4*)(src + 32);
      D[i][3] = *(const GAS u32x4*)(src + 32 * NN + 32);
    }
    const u32 call = __builtin_amdgcn_readfirstlane(cnt_old) + 1;
    if (lane == 0) __hip_atomic_store(cntp, call, __ATOMIC_RELAXED, __HIP_MEMORY_SCOPE_AGENT);
    const u32 par = call & 1;
    __builtin_amdgcn_s_barrier();
#pragma unroll
    for (int i = 0; i < 8; ++i)
      asm volatile("" : "+v"(D[i][0]), "+v"(D[i][1]), "+v"(D[i][2]), "+v"(D[i][3]));
    STAMP(1);
    const u32 vo = lane * 16;
    const size_t woff = WS_WB + par * WB_PAR + (size_t)u * 8 * PIECE;
    const size_t foff = WS_FW + (size_t)u * 8 * 128 + g * 4;
    unsigned char* pend = nullptr;
    u32 pendv = 0;
    auto slot = [&](u32x4 (&V)[4], int i, int d, bool have) {
      unsigned char* pd = P.peer[d];
      unsigned char* dst = pd + woff + (size_t)(d * 64 * 8 + i) * PIECE;
      ST16(dst, vo, V[0], 0);
      ST16(dst, vo, V[1], 1024);
      ST16(dst, vo, V[2], 2048);
      ST16(dst, vo, V[3], 3072);
      if (have) {
        asm volatile("s_waitcnt vmcnt(4)" ::: "memory");
        if (lane == 0) flag_store(pend, pendv);
      }
      pend = pd + foff + d * 128;
      pendv = call * 16 + i + 1;
    };
#pragma unroll
    for (int i = 0; i < 8; ++i) {
      const int d0 = ((g + i) & 3) + 1;
      slot(D[i], i, d0, i > 0);
      if (d0 < 4) slot(D[i], i, d0 + 4, true);
      STAMP(8 + i);
    }
    asm volatile("s_waitcnt vmcnt(0)" ::: "memory");
    if (lane == 0) flag_store(pend, pendv);
    STAMP(3);
  }
}

}

extern "C" void kernel_launch(int rank, int nranks, ncclComm_t comm,
                              hipStream_t stream,
                              void* const* d_in, const int* in_sizes, int n_in,
                              void* d_out, int out_size,
                              const int* in_dtypes, int out_dtype,
                              void* const* d_out_all,
                              void* const* d_ws_all, size_t ws_bytes) {
  (void)comm; (void)in_dtypes; (void)out_dtype; (void)d_out_all;
  if (nranks != R || (unsigned)rank >= (unsigned)R || n_in != 4 || ws_bytes < WS_END ||
      in_sizes[0] != 4096 * KS || in_sizes[1] != KS * NN || out_size != MS * NN) {
    fprintf(stderr, "kernel_launch: unexpected problem shape\n");
    return;
  }
  HIP_CHECK(hipFuncSetAttribute((const void*)fused_kernel, hipFuncAttributeMaxDynamicSharedMemorySize, LDS_BYTES));
  Params P;
  P.x = (const unsigned char*)d_in[0];
  P.w = (const unsigned char*)d_in[1];
  P.sx = (const float*)d_in[2];
  P.sw = (const float*)d_in[3];
  P.out = (float*)d_out;
  for (int d = 0; d < R; ++d) P.peer[d] = (unsigned char*)d_ws_all[(rank + d) % R];
  P.rank = rank;
  P.pad = 0;
  hipLaunchKernelGGL(fused_kernel, dim3(NWG), dim3(NTHREADS), LDS_BYTES, stream, P);
  HIP_CHECK(hipGetLastError());
}
```

```cpp
#include <hip/hip_runtime.h>
#include <rccl/rccl.h>
#include <cstdio>

#define HIP_CHECK(expr) do { hipError_t _e = (expr); if (_e != hipSuccess) { \
    fprintf(stderr, "HIP error %s at %s:%d\n", hipGetErrorString(_e), __FILE__, __LINE__); \
    return; } } while (0)

#define EPI 2
#define EPI_BITS "sc1"
#define NAGG_ 4
#define NPW 28

namespace {

typedef unsigned u32;
typedef unsigned long long u64;
typedef unsigned u32x4 __attribute__((ext_vector_type(4)));
typedef int i32x2 __attribute__((ext_vector_type(2)));
typedef int i32x8 __attribute__((ext_vector_type(8)));
typedef float f32x4 __attribute__((ext_vector_type(4)));
typedef float f32x16 __attribute__((ext_vector_type(16)));
#define LAS __attribute__((address_space(3)))
#define GAS __attribute__((address_space(1)))

constexpr int R = 8;
constexpr int MS = 512;
constexpr int KS = 512;
constexpr int NN = 4096;
constexpr int NWG = 256;
constexpr int NWAVES = 12;
constexpr int NTHREADS = NWAVES * 64;
constexpr int RR = 4;
constexpr int PIECE = 4096;
constexpr int SLOT = 5120;
constexpr int MAXEXC = 510;
constexpr int LDS_OWN = RR * 7 * PIECE;
constexpr int LDS_CTRL = LDS_OWN + 8 * PIECE;
constexpr int LDS_READY = LDS_CTRL;
constexpr int LDS_DONE = LDS_CTRL + 192;
constexpr int LDS_BAD = LDS_READY + (7 & (RR - 1)) * 32;
constexpr int LDS_GO = LDS_CTRL + 232;
constexpr int LDS_ENC = LDS_CTRL + 256;
constexpr int LDS_BYTES = LDS_ENC + 4096 + 1024 + 128;
static_assert(LDS_BYTES <= 160 * 1024, "LDS");

constexpr size_t WS_CNT = 0;
constexpr size_t WS_TMO = 32768;
constexpr size_t WS_FX = 65536;
constexpr size_t WS_TK = 36864;
constexpr size_t WS_RC = 40960;
constexpr size_t WS_FR = 49152;
constexpr size_t WS_FW = 131072;
constexpr u32 NAGG = NAGG_;
constexpr size_t WS_DBG = 262144;
constexpr size_t WS_XB = 1ull << 20;
constexpr size_t XB_PAR = 2ull << 20;
constexpr size_t WB_SRC = (size_t)512 * SLOT;
constexpr size_t WS_WB = 8ull << 20;
constexpr size_t WB_PAR = 8 * WB_SRC;
constexpr size_t WS_END = WS_WB + 2 * WB_PAR;

constexpr u64 DEADLINE_TICKS = 40000000ull;

struct Params {
  const unsigned char* x;
  const unsigned char* w;
  const float* sx;
  const float* sw;
  float* out;
  unsigned char* peer[R];
  int rank;
  int pad;
};

__device__ __forceinline__ u64 rt() { return __builtin_amdgcn_s_memrealtime(); }

#define STAMP(slot) do {} while (0)

#define ST16(base, voff, val, imm) \
  asm volatile("global_store_dwordx4 %0, %1, %2 offset:%3 sc0 sc1\n\ts_nop 1" :: "v"(voff), "v"(val), "s"(base), "i"(imm) : "memory")
#define LD16(val, base, voff, imm) \
  asm volatile("global_load_dwordx4 %0, %1, %2 offset:%3" : "+v"(val) : "v"(voff), "s"(base), "i"(imm) : "memory")
#define LD16S(val, base, voff, imm) \
  asm volatile("global_load_dwordx4 %0, %1, %2 offset:%3 sc0 sc1" : "+v"(val) : "v"(voff), "s"(base), "i"(imm) : "memory")
#define VMWAIT(n) asm volatile("s_waitcnt vmcnt(%0)" :: "i"(n) : "memory")

__device__ __forceinline__ void flag_store(unsigned char* p, u32 v) {
  __hip_atomic_store((GAS u32*)p, v, __ATOMIC_RELAXED, __HIP_MEMORY_SCOPE_SYSTEM);
}
__device__ __forceinline__ u32 flag_load(const unsigned char* p) {
  return __hip_atomic_load((GAS u32*)p, __ATOMIC_RELAXED, __HIP_MEMORY_SCOPE_SYSTEM);
}
__device__ __forceinline__ u32 ag_load(const unsigned char* p) {
  return __hip_atomic_load((GAS u32*)p, __ATOMIC_RELAXED, __HIP_MEMORY_SCOPE_AGENT);
}
__device__ __forceinline__ void ag_store(unsigned char* p, u32 v) {
  __hip_atomic_store((GAS u32*)p, v, __ATOMIC_RELAXED, __HIP_MEMORY_SCOPE_AGENT);
}

template <bool SYS>
__device__ __forceinline__ bool wait_all(const unsigned char* fp, u32 want, unsigned char* ws, u32 call) {
  const u64 t0 = rt();
  for (u32 spin = 1;; ++spin) {
    const u32 v = SYS ? flag_load(fp) : ag_load(fp);
    if (__all(v >= want)) break;
    if ((spin & 15) == 0 && (__builtin_amdgcn_readfirstlane(ag_load(ws + WS_TMO)) == call || rt() - t0 > DEADLINE_TICKS)) { ag_store(ws + WS_TMO, call); return false; }
    __builtin_amdgcn_s_sleep(2);
  }
  asm volatile("" ::: "memory");
  return true;
}

__device__ __forceinline__ u32 wave_sum(u32 v) {
  v += __builtin_amdgcn_update_dpp(0u, v, 0x111, 0xf, 0xf, true);
  v += __builtin_amdgcn_update_dpp(0u, v, 0x112, 0xf, 0xf, true);
  v += __builtin_amdgcn_update_dpp(0u, v, 0x114, 0xf, 0xf, true);
  v += __builtin_amdgcn_update_dpp(0u, v, 0x118, 0xf, 0xf, true);
  return __builtin_amdgcn_readlane(v, 15) + __builtin_amdgcn_readlane(v, 31) +
         __builtin_amdgcn_readlane(v, 47) + __builtin_amdgcn_readlane(v, 63);
}

__device__ __forceinline__ void read_bfrag(i32x8& r, u32 addr, int imm) {
  LAS unsigned char* b = (LAS unsigned char*)(size_t)addr;
  i32x2 r0 = __builtin_amdgcn_ds_read_tr8_b64_v2i32((LAS i32x2*)(b + imm));
  i32x2 r1 = __builtin_amdgcn_ds_read_tr8_b64_v2i32((LAS i32x2*)(b + imm + 256));
  i32x2 r2 = __builtin_amdgcn_ds_read_tr8_b64_v2i32((LAS i32x2*)(b + imm + 512));
  i32x2 r3 = __builtin_amdgcn_ds_read_tr8_b64_v2i32((LAS i32x2*)(b + imm + 768));
  r[0] = r0.x; r[1] = r0.y; r[2] = r1.x; r[3] = r1.y; r[4] = r2.x; r[5] = r2.y; r[6] = r3.x; r[7] = r3.y;
}

__device__ __forceinline__ u32 miss_pattern(u32 m  , u32 cneg, u32 lo4, u32& code) {
  code = (m + cneg) & 0x1f1f1f1fu;
  return (code + lo4) ^ m;
}

__device__ __forceinline__ u32 wave_scan(u32 v) {
  v += __builtin_amdgcn_update_dpp(0u, v, 0x111, 0xf, 0xf, true);
  v += __builtin_amdgcn_update_dpp(0u, v, 0x112, 0xf, 0xf, true);
  v += __builtin_amdgcn_update_dpp(0u, v, 0x114, 0xf, 0xf, true);
  v += __builtin_amdgcn_update_dpp(0u, v, 0x118, 0xf, 0xf, true);
  v += __builtin_amdgcn_update_dpp(0u, v, 0x142, 0xa, 0xf, false);
  v += __builtin_amdgcn_update_dpp(0u, v, 0x143, 0xc, 0xf, false);
  return v;
}

__device__ __forceinline__ int choose_window(const u32x4 (&V)[4]) {
  u32 sum = 0;
#pragma unroll
  for (int t = 0; t < 4; ++t) sum = __builtin_amdgcn_sad_u8(V[t][3 - t] & 0x7c7c7c7cu, 0u, sum);
  int lo = (int)((wave_sum(sum) + 1638u) >> 12) - 6;
  lo = lo < 0 ? 0 : (lo > 22 ? 22 : lo);
  u32 cnts = 0;
#pragma unroll
  for (int k = 0; k < 3; ++k) {
    const u32 lo4 = (u32)(lo + k) * 0x04040404u, cneg = (u32)((-4 * (lo + k)) & 31) * 0x01010101u;
    u32 n = 0, code;
#pragma unroll
    for (int t = 0; t < 4; ++t)
      n += __builtin_popcount((miss_pattern(V[t][t] & 0x7f7f7f7fu, cneg, lo4, code) + 0x7c7c7c7cu) & 0x80808080u);
    cnts |= min(n, 15u) << (10 * k);
  }
  const u32 tot = wave_sum(cnts);
  const u32 n0 = tot & 1023, n1 = (tot >> 10) & 1023, n2 = tot >> 20;
  return lo + ((n1 <= n0 && n1 <= n2) ? 1 : (n0 <= n2 ? 0 : 2));
}

__device__ __forceinline__ u32 encode_piece(const u32x4 (&V)[4], int lo, u32x4 (&D)[4], u32x4& T, LAS unsigned char* scr, int lane) {
  const u32 lo4 = (u32)lo * 0x04040404u, cneg = (u32)((-4 * lo) & 31) * 0x01010101u;
  const u32 vo = lane * 16;
  u32x4 C[4];
  u32 M[2];
#pragma unroll
  for (int t = 0; t < 4; ++t) {
    u32x4 X;
    u32 f[4];
#pragma unroll
    for (int e = 0; e < 4; ++e) {
      u32 code;
      X[e] = miss_pattern(V[t][e] & 0x7f7f7f7fu, cneg, lo4, code);
      C[t][e] = code | (V[t][e] & 0x80808080u);
      f[e] = (X[e] + 0x7c7c7c7cu) & 0x80808080u;
    }
    *(LAS u32x4*)(scr + t * 1024 + vo) = X;
    const u32 b0 = __builtin_amdgcn_udot4(f[1], 0x80402010u, __builtin_amdgcn_udot4(f[0], 0x08040201u, 0u, false), false);
    const u32 b1 = __builtin_amdgcn_udot4(f[3], 0x80402010u, __builtin_amdgcn_udot4(f[2], 0x08040201u, 0u, false), false);
    const u32 m16 = (b0 >> 7) | (b1 << 1);
    if (t & 1) M[t >> 1] |= m16 << 16; else M[t >> 1] = m16;
  }
  const u32 n = __builtin_popcount(M[0]) | (__builtin_popcount(M[1]) << 16);
  const u32 inc = wave_scan(n);
  const u32 tot = __builtin_amdgcn_readlane(inc, 63);
  const u32 nhalf = tot & 0xffffu, nexc = nhalf + (tot >> 16);
  LAS unsigned char* recs = scr + 4096;
  if (nexc > MAXEXC) {
    D[0] = V[0]; D[1] = V[1]; D[2] = V[2]; T = V[3];
    D[3] = u32x4{0x80000000u, 0u, 0u, 0u};
    return nexc;
  }
  u32 wa[2] = {4 + 2 * ((inc - n) & 0xffffu), 4 + 2 * (nhalf + ((inc - n) >> 16))};
  const u32 dummy = 1024 + 2 * lane;
  if (lane == 0) *(LAS u32*)recs = nhalf | (nexc << 10) | ((u32)lo << 20);
  do {
    u32 pos[2][3], x[2][3], ad[2][3];
#pragma unroll
    for (int r = 0; r < 3; ++r)
#pragma unroll
      for (int h = 0; h < 2; ++h) {
        const u32 bit = __builtin_ctz(M[h] | 0x80000000u);
        pos[h][r] = vo + (bit & 15) + ((bit & 16) << 6);
        x[h][r] = scr[h * 2048 + pos[h][r]];
        ad[h][r] = M[h] ? wa[h] : dummy;
        wa[h] += 2;
        M[h] &= M[h] - 1;
      }
#pragma unroll
    for (int r = 0; r < 3; ++r)
#pragma unroll
      for (int h = 0; h < 2; ++h)
        *(LAS unsigned short*)(recs + ad[h][r]) = (unsigned short)(pos[h][r] | (x[h][r] << 9));
  } while (__any((M[0] | M[1]) != 0));
#pragma unroll
  for (int e = 0; e < 4; ++e) {
    const u32 c3 = C[3][e];
    D[0][e] = C[0][e] | ((c3 << 5) & 0x60606060u);
    D[1][e] = C[1][e] | ((c3 << 3) & 0x60606060u);
    D[2][e] = C[2][e] | ((c3 << 1) & 0x20202020u) | ((c3 >> 1) & 0x40404040u);
  }
  D[3] = *(LAS u32x4*)(recs + vo);
  return nexc;
}

__device__ __forceinline__ void unpack_piece(const unsigned char* src, LAS unsigned char* dst, int lane) {
  LAS u32x4* o = (LAS u32x4*)(dst + lane * 16);
  u32 ev = *(LAS u32*)(dst + 3072 + lane * 4);
  u32x4 D[3] = {o[0], o[64], o[128]};
  asm volatile("" : "+v"(ev), "+v"(D[0]), "+v"(D[1]), "+v"(D[2]));
  const u32 hdr = __builtin_amdgcn_readfirstlane(ev);
  if (hdr >> 31) {
    u32x4 D3;
    asm volatile("global_load_dwordx4 %0, %1, %2 sc0 sc1\n\ts_waitcnt vmcnt(0)" : "=&v"(D3) : "v"(lane * 16), "s"(src + 4096) : "memory");
    o[192] = D3;
    return;
  }
  const u32 lo4 = ((hdr >> 20) & 31) * 0x04040404u, nhalf = hdr & 1023, nexc = (hdr >> 10) & 1023;
  u32x4 O0, O1, O2, O3;
#pragma unroll
  for (int e = 0; e < 4; ++e) {
    const u32 d0 = D[0][e], d1 = D[1][e], d2 = D[2][e];
    u32 s0 = d0 & 0x60606060u, s1 = d1 & 0x60606060u, s2 = d2 & 0x60606060u;
    asm("" : "+v"(s0), "+v"(s1), "+v"(s2));
    O0[e] = (d0 ^ s0) + lo4; O1[e] = (d1 ^ s1) + lo4; O2[e] = (d2 ^ s2) + lo4;
    const u32 t = (s0 >> 5) | (s1 >> 3) | (s2 >> 1), z = t & 0x20202020u;
    O3[e] = t + lo4 + 3 * z;
  }
  o[0] = O0; o[64] = O1; o[128] = O2; o[192] = O3;
  for (u32 base = 0;;) {
#pragma unroll
    for (int h = 0; h < 2; ++h) {
      const u32 rec = h ? ev >> 16 : ev & 0xffffu;
      const u32 idx = base + 2 * lane - 2 + h;
      const u32 off = (rec & 0x7ffu) | (idx >= nhalf ? 0x800u : 0u);
      if (idx < nexc)
        __hip_atomic_fetch_xor((LAS u32*)(dst + (off & ~3u)), ((rec >> 11) << 2) << (8 * (off & 3)), __ATOMIC_RELAXED, __HIP_MEMORY_SCOPE_WORKGROUP);
    }
    base += 128;
    if (base >= nexc + 2) break;
    asm volatile("global_load_dword %0, %1, %2 offset:3072 sc0 sc1\n\ts_waitcnt vmcnt(0)" : "=&v"(ev) : "v"(lane * 4 + 2 * base), "s"(src) : "memory");
  }
}

__global__ void __launch_bounds__(NTHREADS, 3) __attribute__((amdgpu_num_vgpr(128))) fused_kernel(Params P) {
  extern __shared__ __attribute__((aligned(16))) unsigned char lds_raw[];
  LAS unsigned char* L = (LAS unsigned char*)lds_raw;
  volatile LAS u32* READY = (volatile LAS u32*)(L + LDS_READY);
  volatile LAS u32* DONE = (volatile LAS u32*)(L + LDS_DONE);
  volatile LAS u32* BAD = (volatile LAS u32*)(L + LDS_BAD);
  volatile LAS u32* GO = (volatile LAS u32*)(L + LDS_GO);

  const int c = blockIdx.x;
  const int lane = threadIdx.x & 63;
  const int wave = __builtin_amdgcn_readfirstlane(threadIdx.x >> 6);
  const int u = (c & 7) * 8 + ((c >> 3) & 7);
  const int mt = c >> 6;
  unsigned char* const ws = P.peer[0];
  STAMP(0);

  unsigned char* cntp = ws + WS_CNT + c * 128 + wave * 4;
#define START_BARRIER() do { __builtin_amdgcn_sched_barrier(0); __builtin_amdgcn_s_barrier(); __builtin_amdgcn_sched_barrier(0); } while (0)
  const u32 w_lane = (u32)(lane >> 1) * NN + (lane & 1) * 16;
  const u32 vo = lane * 16;

  if (wave == NWAVES - 1) {
    if (c < 224) {
      const int d = 1 + c % 7, j = c / 7;
      const unsigned char* xs = P.x + (size_t)(((P.rank + d) & 7) * MS + (j >> 1) * 32) * KS + (j & 1) * 256;
      const u32 xo = (u32)(lane & 31) * KS + 32 * (lane >> 5);
      const u32 wl2 = w_lane + 32 * NN;
#define ACLOB "v128", "v129", "v130", "v131", "v132", "v133", "v134", "v135", "v136", "v137", "v138", "v139", "v140", "v141", "v142", "v143", "v144", "v145", "v146", "v147", \
              "v148", "v149", "v150", "v151", "v152", "v153", "v154", "v155", "v156", "v157", "v158", "v159", "v160", "v161", "v162", "memory"
#define UTILE(t) ((((t) & 7) << 3) | (((t) >> 3) & 7))
#define LOADW(t) do { const unsigned char* s_ = P.w + (size_t)((t) >> 6) * 64 * NN + UTILE(t) * 64; \
        asm volatile("global_load_dwordx4 v[128:131], %0, %2\n\tglobal_load_dwordx4 v[132:135], %1, %2\n\t" \
                     "global_load_dwordx4 v[136:139], %0, %2 offset:32\n\tglobal_load_dwordx4 v[140:143], %1, %2 offset:32" \
                     :: "v"(w_lane), "v"(wl2), "s"(s_) : ACLOB); } while (0)
      asm volatile("global_load_dword v160, %0, %1 sc1\n\t"
                   "global_load_dwordx4 v[128:131], %2, %3\n\tglobal_load_dwordx4 v[132:135], %2, %3 offset:16\n\t"
                   "global_load_dwordx4 v[136:139], %2, %3 offset:64\n\tglobal_load_dwordx4 v[140:143], %2, %3 offset:80\n\t"
                   "global_load_dwordx4 v[144:147], %2, %3 offset:128\n\tglobal_load_dwordx4 v[148:151], %2, %3 offset:144\n\t"
                   "global_load_dwordx4 v[152:155], %2, %3 offset:192\n\tglobal_load_dwordx4 v[156:159], %2, %3 offset:208\n\t"
                   "v_mov_b32 v162, 1\n\tv_mov_b32 v161, 0"
                   :: "v"(0u), "s"(cntp), "v"(xo), "s"(xs) : ACLOB);
      START_BARRIER();
      __builtin_amdgcn_s_setprio(2);
      u32 cnt;
      asm volatile("s_waitcnt vmcnt(8)\n\tv_mov_b32 %0, v160" : "=v"(cnt) :: "memory");
      const u32 call = __builtin_amdgcn_readfirstlane(cnt) + 1;
      unsigned char* pd = P.peer[d];
      unsigned char* dst = pd + WS_XB + (call & 1) * XB_PAR + (size_t)(d * 32 + j) * 8192;
      asm volatile("s_waitcnt vmcnt(4)\n\t"
                   "global_store_dwordx4 %0, v[128:131], %1 sc0 sc1\n\tglobal_store_dwordx4 %0, v[132:135], %1 offset:1024 sc0 sc1\n\t"
                   "global_store_dwordx4 %0, v[136:139], %1 offset:2048 sc0 sc1\n\tglobal_store_dwordx4 %0, v[140:143], %1 offset:3072 sc0 sc1\n\ts_nop 1"
                   :: "v"(vo), "s"(dst) : "memory");
      LOADW(j);
      dst += 4096;
      VMWAIT(8);
      if (lane == 0) ag_store(cntp, call);
      asm volatile("global_store_dwordx4 %0, v[144:147], %1 sc0 sc1\n\tglobal_store_dwordx4 %0, v[148:151], %1 offset:1024 sc0 sc1\n\t"
                   "global_store_dwordx4 %0, v[152:155], %1 offset:2048 sc0 sc1\n\tglobal_store_dwordx4 %0, v[156:159], %1 offset:3072 sc0 sc1\n\ts_nop 1"
                   :: "v"(vo), "s"(dst) : "memory");
      GO[0] = 1u;
      STAMP(1);
      unsigned char* const wdst = pd + WS_WB + (call & 1) * WB_PAR + d * WB_SRC;
      unsigned char* const fw = pd + WS_FW + d * 64 * 128;
      unsigned char* const fr = pd + WS_FR + d * 128;
      unsigned char* const rc = ws + WS_RC + d * 8 * 128;
      unsigned char* const fx = pd + WS_FX + (d * 16 + (j >> 1)) * 128 + (j & 1) * 4;
      const unsigned char* tkp = ws + WS_TK + d * 128;
      const u32 tbase = (call - 1) * 512u - 2u * NPW;
      LAS unsigned char* scr = L + LDS_ENC;
      int lo = 0;
      u32 nexc = 1000, cur = j;
      asm volatile("v_mov_b32 v160, %0" :: "v"(tbase + NPW + j) : ACLOB);
      u32 p1 = 1025;
      u32 cr;
#define LANDED(t) do { if (lane == 0) { \
          if ((t) < NAGG * 64u) { unsigned char* a_ = rc + ((t) >> 6) * 128; \
            asm volatile("global_atomic_add v161, %0, v162, %1 sc0 sc1" :: "v"(0u), "s"(a_) : ACLOB); } \
          else flag_store((t) < 512u ? fw + UTILE(t) * 128 + ((t) >> 6) * 4 : fx, call); } \
        cr = (t) >> 6; } while (0)
#define PUBLISH(n) do { u32 cv; asm volatile("s_waitcnt vmcnt(" #n ")\n\tv_mov_b32 %0, v161" : "=v"(cv) :: "memory"); \
        if (cr < NAGG && (__builtin_amdgcn_readfirstlane(cv) & 63u) == 63u && lane == 0) flag_store(fr + cr * 4, call); } while (0)
      cr = 8;
#pragma unroll 1
      while (j < NPW) {
        u32x4 D[4], T, V[4];
        u32 tk;
        {
          u32 v0, v1, v2, v3, v4, v5, v6, v7, v8, v9, v10, v11, v12, v13, v14, v15;
          asm volatile("s_waitcnt vmcnt(5)\n\tv_mov_b32 %16, v160\n\t"
                       "v_mov_b32 %0, v128\n\tv_mov_b32 %1, v129\n\tv_mov_b32 %2, v130\n\tv_mov_b32 %3, v131\n\t"
                       "v_mov_b32 %4, v132\n\tv_mov_b32 %5, v133\n\tv_mov_b32 %6, v134\n\tv_mov_b32 %7, v135\n\t"
                       "v_mov_b32 %8, v136\n\tv_mov_b32 %9, v137\n\tv_mov_b32 %10, v138\n\tv_mov_b32 %11, v139\n\t"
                       "v_mov_b32 %12, v140\n\tv_mov_b32 %13, v141\n\tv_mov_b32 %14, v142\n\tv_mov_b32 %15, v143"
                       : "=v"(v0), "=v"(v1), "=v"(v2), "=v"(v3), "=v"(v4), "=v"(v5), "=v"(v6), "=v"(v7), "=v"(v8), "=v"(v9), "=v"(v10),
                         "=v"(v11), "=v"(v12), "=v"(v13), "=v"(v14), "=v"(v15), "=v"(tk) :: "memory");
          V[0] = u32x4{v0, v1, v2, v3}; V[1] = u32x4{v4, v5, v6, v7}; V[2] = u32x4{v8, v9, v10, v11}; V[3] = u32x4{v12, v13, v14, v15};
        }
        const u32 nxt = __builtin_amdgcn_readfirstlane(tk) - tbase;
        if (lane == 0) asm volatile("global_atomic_add v160, %0, v162, %1 sc0 sc1" :: "v"(0u), "s"(tkp) : ACLOB);
        const u32 nl = nxt < 512u ? nxt : cur;
        LOADW(nl);
        if (nexc > 96) lo = choose_window(V);
        nexc = encode_piece(V, lo, D, T, scr, lane);
        PUBLISH(9);
        VMWAIT(5);
        LANDED(p1);
        dst = wdst + (size_t)((cur & ~63u) + UTILE(cur)) * SLOT;
        ST16(dst, vo, D[0], 0); ST16(dst, vo, D[1], 1024); ST16(dst, vo, D[2], 2048);
        if ((vo & ~63u) < 4 + 2 * nexc) ST16(dst, vo, D[3], 3072);
        if (nexc > MAXEXC) { unsigned char* d2 = dst + 4096; ST16(d2, vo, T, 0); }
        p1 = cur;
        if ((cur & 32) == 0) STAMP(8 + (cur >> 6));
        if (nxt >= 512u) break;
        cur = nxt;
      }
      PUBLISH(0);
      LANDED(p1);
      STAMP(3);
      PUBLISH(0);
#undef LANDED
#undef PUBLISH
#undef LOADW
#undef ACLOB
#undef UTILE
    } else {
      START_BARRIER();
      GO[0] = 1u;
    }
  } else if (wave < 4) {
    const int mymb = mt * 4 + wave;
    const u32 cnt_old = ag_load(cntp);
    START_BARRIER();
    const u32 call = __builtin_amdgcn_readfirstlane(cnt_old) + 1;
    const u32 par = call & 1;
    STAMP(1);
    if (lane == 0) ag_store(cntp, call);
    const float scale = P.sx[0] * P.sw[0];
    const u32 b_lane = (u32)(size_t)L + ((lane >> 5) * 32 + ((lane & 15) >> 1)) * 32 + ((lane >> 4) & 1) * 16 + (lane & 1) * 8;
    f32x16 acc0 = {}, acc1 = {};
    u32 rv;
    u32x4 AL[8] = {}, AH[8] = {};
#define AWAIT() asm volatile("s_waitcnt vmcnt(0)" : "+v"(AL[0]), "+v"(AL[1]), "+v"(AL[2]), "+v"(AL[3]), "+v"(AL[4]), "+v"(AL[5]), "+v"(AL[6]), "+v"(AL[7]), \
                             "+v"(AH[0]), "+v"(AH[1]), "+v"(AH[2]), "+v"(AH[3]), "+v"(AH[4]), "+v"(AH[5]), "+v"(AH[6]), "+v"(AH[7]) :: "memory")
#define MMA(p, off) do { \
          i32x8 b0, b1; read_bfrag(b0, baddr, off); read_bfrag(b1, baddr, off + 2048); \
          const i32x8 a = {(int)AL[p].x, (int)AL[p].y, (int)AL[p].z, (int)AL[p].w, (int)AH[p].x, (int)AH[p].y, (int)AH[p].z, (int)AH[p].w}; \
          __builtin_amdgcn_sched_barrier(0); \
          acc0 = __builtin_amdgcn_mfma_scale_f32_32x32x64_f8f6f4(b0, a, acc0, 1, 0, 0, 0, 0, 0); \
          acc1 = __builtin_amdgcn_mfma_scale_f32_32x32x64_f8f6f4(b1, a, acc1, 1, 0, 0, 0, 0, 0); } while (0)
#define POLL(row, want) do { while (true) { \
          rv = READY[(row) * 8 + (lane & 7)]; \
          m = (u32)__ballot(rv >= (u32)(want)) & rem; \
          if (m) break; \
          __builtin_amdgcn_s_sleep(1); } \
        asm volatile("" ::: "memory"); rem &= ~m; } while (0)
    while (GO[0] == 0u) __builtin_amdgcn_s_sleep(4);
    {
      const unsigned char* xs = P.x + (size_t)(P.rank * MS + mymb * 32) * KS;
      const u32 xo = (u32)(lane & 31) * KS + 32 * (lane >> 5);
#define LA(p) LD16(AL[p], xs, xo, p * 64); LD16(AH[p], xs, xo, p * 64 + 16);
      LA(0) LA(1) LA(2) LA(3) LA(4) LA(5) LA(6) LA(7)
#undef LA
      AWAIT();
      const u32 baddr = b_lane + LDS_OWN;
      u32 rem = 0xFFu, m;
      do {
        POLL(RR, 1);
#define STEP(p) if (m & (1u << p)) MMA(p, p * PIECE);
        STEP(0) STEP(1) STEP(2) STEP(3) STEP(4) STEP(5) STEP(6) STEP(7)
#undef STEP
      } while (rem);
    }
    STAMP(7);
    if (!wait_all<true>(ws + WS_FX + ((1 + (lane & 7) % 7) * 16 + mymb) * 128 + ((lane >> 3) & 1) * 4, call, ws, call)) BAD[0] = 1u;
    STAMP(2);
    const unsigned char* xb = ws + WS_XB + par * XB_PAR + (size_t)mymb * 16384;
#define LA(p) do { const unsigned char* s_ = xb + p * 262144; LD16S(AL[p], s_, vo, 0); LD16S(AH[p], s_, vo, 1024); } while (0)
    LA(1); LA(2); LA(3); LA(4); LA(5); LA(6); LA(7);
#pragma unroll 1
    for (int i = 0; i < 8; ++i) {
      const int row = i & (RR - 1);
      const u32 baddr = b_lane + row * 7 * PIECE;
      xb += 2048;
      AWAIT();
      u32 rem = 0xFEu, m;
      do {
        POLL(row, i + 1);
#define STEP(p) if (m & (1u << p)) { MMA(p, (p - 1) * PIECE); if (i < 7) LA(p); }
        STEP(1) STEP(2) STEP(3) STEP(4) STEP(5) STEP(6) STEP(7)
#undef STEP
      } while (rem);
      asm volatile("s_waitcnt lgkmcnt(0)" ::: "memory");
      if (lane == 0) DONE[wave] = (u32)(i + 1);
      STAMP(8 + i);
    }
#undef LA
    AWAIT();
    const bool bad = __builtin_amdgcn_readfirstlane(rv) != 0u;
    u32x4 o[8];
    const float sc = bad ? __builtin_nanf("") : scale;
    acc0 *= sc; acc1 *= sc;
#pragma unroll
    for (int q = 0; q < 4; ++q)
#pragma unroll
      for (int e = 0; e < 4; ++e) {
        o[q][e] = __builtin_bit_cast(u32, __builtin_elementwise_maximum(acc0[4 * q + e], 0.f));
        o[4 + q][e] = __builtin_bit_cast(u32, __builtin_elementwise_maximum(acc1[4 * q + e], 0.f));
      }
    const unsigned char* ob = (const unsigned char*)(P.out + (size_t)mymb * 32 * NN + u * 64);
#define OST(val, voff, imm) asm volatile("global_store_dwordx4 %0, %1, %2 offset:%3 " EPI_BITS "\n\ts_nop 1" :: "v"(voff), "v"(val), "s"(ob), "i"(imm) : "memory")
    {
      LAS unsigned char* tb = L + LDS_OWN + wave * 8192;
      const int m = lane & 31, h = lane >> 5;
#pragma unroll
      for (int q = 0; q < 8; ++q) *(LAS u32x4*)(tb + m * 256 + (((2 * q + h) ^ (m & 15)) << 4)) = o[q];
      const int r0 = lane >> 4, ch = lane & 15;
#pragma unroll
      for (int t = 0; t < 8; ++t) o[t] = *(LAS u32x4*)(tb + (4 * t + r0) * 256 + ((ch ^ ((4 * t + r0) & 15)) << 4));
      u32 oo = (u32)r0 * NN * 4 + ch * 16;
#pragma unroll
      for (int t = 0; t < 8; ++t) { OST(o[t], oo, 0); oo += 4 * NN * 4; }
    }
#undef OST
    STAMP(3);
  } else {
    const int d = wave - 3;
    const u32 cnt_old = ag_load(cntp);
    if (wave == 5) asm volatile("ds_write_b32 %0, %1\n\ts_waitcnt lgkmcnt(0)" :: "v"((u32)(size_t)L + LDS_CTRL + lane * 4), "v"(0u) : "memory");
    START_BARRIER();
    while (GO[0] == 0u) __builtin_amdgcn_s_sleep(4);
    {
      const unsigned char* src = P.w + u * 64 + w_lane;
      for (int i = d - 1; i < 8; i += 7) {
        const unsigned char* s = src + (size_t)i * 64 * NN;
        LAS unsigned char* dst = L + LDS_OWN + i * PIECE;
        __builtin_amdgcn_global_load_lds((const GAS u32*)(s), (LAS u32*)(dst), 16, 0, 0);
        __builtin_amdgcn_global_load_lds((const GAS u32*)(s + 32 * NN), (LAS u32*)(dst + 1024), 16, 0, 0);
        __builtin_amdgcn_global_load_lds((const GAS u32*)(s + 32), (LAS u32*)(dst + 2048), 16, 0, 0);
        __builtin_amdgcn_global_load_lds((const GAS u32*)(s + 32 * NN + 32), (LAS u32*)(dst + 3072), 16, 0, 0);
      }
    }
    VMWAIT(0);
    if (lane == 0) { READY[RR * 8 + d - 1] = 1u; if (d == 1) READY[RR * 8 + 7] = 1u; }
    const u32 call = __builtin_amdgcn_readfirstlane(cnt_old) + 1;
    if (lane == 0) ag_store(cntp, call);
    const unsigned char* fp = ws + ((lane & 7) < NAGG ? WS_FR + d * 128 : WS_FW + (d * 64 + u) * 128) + (lane & 7) * 4;
    const unsigned char* src = ws + WS_WB + (call & 1) * WB_PAR + d * WB_SRC + (size_t)u * SLOT;
    u32 seen = 0;
    bool dead = false, pre = false;
    const unsigned char* tmop = ws + WS_TMO;
    const u32 l4 = lane * 4;
#define FETCH(i_, s_) do { u32 keep_; const u32 la_ = (u32)(size_t)L + ((((i_) & (RR - 1)) * 7 + d - 1) * PIECE); \
      asm volatile("s_mov_b32 %0, m0\n\ts_mov_b32 m0, %4\n\ts_nop 0\n\t" \
                   "global_load_lds_dwordx4 %1, %3 sc0 sc1\n\t" \
                   "global_load_lds_dwordx4 %1, %3 offset:1024 sc0 sc1\n\t" \
                   "global_load_lds_dwordx4 %1, %3 offset:2048 sc0 sc1\n\t" \
                   "global_load_lds_dword %2, %3 offset:3072 sc0 sc1\n\t" \
                   "s_mov_b32 m0, %0" : "=&s"(keep_) : "v"(vo), "v"(l4), "s"(s_), "s"(la_) : "memory"); } while (0)
#pragma unroll 1
    for (int i = 0; i < 8; ++i, src += 64 * SLOT) {
      if (!pre) {
        if (i >= RR) {
          while (!__all(DONE[lane & 3] + RR > (u32)i)) __builtin_amdgcn_s_sleep(2);
          asm volatile("" ::: "memory");
        }
        if (!((seen >> i) & 1) && !dead) {
          const u64 t0 = rt();
          for (u32 spin = 1;; ++spin) {
            seen = (u32)__ballot(flag_load(fp) >= call);
            if ((seen >> i) & 1) break;
            if ((spin & 63) == 0 && (__builtin_amdgcn_readfirstlane(ag_load(tmop)) == call || rt() - t0 > DEADLINE_TICKS)) { dead = true; break; }
            if (i < 7) __builtin_amdgcn_s_sleep(2);
          }
          if (dead) ag_store(ws + WS_TMO, call);
          asm volatile("" ::: "memory");
        }
        if (i == 7) STAMP(4);
        if (!dead) FETCH(i, src);
      }
      pre = !dead && i < 7 && ((seen >> (i + 1)) & 1) && __all(DONE[lane & 3] + RR > (u32)i + 1);
      if (pre) {
        const unsigned char* s2 = src + 64 * SLOT;
        FETCH(i + 1, s2);
        VMWAIT(4);
      } else {
        if (i == 7) dead = dead || __builtin_amdgcn_readfirstlane(ag_load(tmop)) == call;
        VMWAIT(0);
      }
      if (i == 7) STAMP(5);
      const int row = i & (RR - 1);
      if (!dead) unpack_piece(src, L + (row * 7 + d - 1) * PIECE, lane); else BAD[0] = 1u;
      asm volatile("s_waitcnt lgkmcnt(0)" ::: "memory");
      READY[row * 8 + d] = (u32)(i + 1);
      STAMP(8 + i);
    }
#undef FETCH
    STAMP(3);
  }
}

}

extern "C" void kernel_launch(int rank, int nranks, ncclComm_t comm,
                              hipStream_t stream,
                              void* const* d_in, const int* in_sizes, int n_in,
                              void* d_out, int out_size,
                              const int* in_dtypes, int out_dtype,
                              void* const* d_out_all,
                              void* const* d_ws_all, size_t ws_bytes) {
  (void)comm; (void)in_dtypes; (void)out_dtype; (void)d_out_all;
  if (nranks != R || (unsigned)rank >= (unsigned)R || n_in != 4 || ws_bytes < WS_END ||
      in_sizes[0] != 4096 * KS || in_sizes[1] != KS * NN || out_size != MS * NN) {
    fprintf(stderr, "kernel_launch: unexpected problem shape\n");
    return;
  }
  HIP_CHECK(hipFuncSetAttribute((const void*)fused_kernel, hipFuncAttributeMaxDynamicSharedMemorySize, LDS_BYTES));
  Params P;
  P.x = (const unsigned char*)d_in[0];
  P.w = (const unsigned char*)d_in[1];
  P.sx = (const float*)d_in[2];
  P.sw = (const float*)d_in[3];
  P.out = (float*)d_out;
  for (int d = 0; d < R; ++d) P.peer[d] = (unsigned char*)d_ws_all[(rank + d) % R];
  P.rank = rank;
  P.pad = 0;
  hipLaunchKernelGGL(fused_kernel, dim3(NWG), dim3(NTHREADS), LDS_BYTES, stream, P);
  HIP_CHECK(hipGetLastError());
}
```

```cpp
#include <hip/hip_runtime.h>
#include <rccl/rccl.h>
#include <cstdio>

#define HIP_CHECK(expr) do { hipError_t _e = (expr); if (_e != hipSuccess) { \
    fprintf(stderr, "HIP error %s at %s:%d\n", hipGetErrorString(_e), __FILE__, __LINE__); \
    return; } } while (0)

#define EPI 2
#define EPI_BITS "sc1"
#define NAGG_ 3
#define EARLY 1
#define BEND 448
#define NPW 22

namespace {

typedef unsigned u32;
typedef unsigned long long u64;
typedef unsigned u32x4 __attribute__((ext_vector_type(4)));
typedef int i32x2 __attribute__((ext_vector_type(2)));
typedef int i32x8 __attribute__((ext_vector_type(8)));
typedef float f32x4 __attribute__((ext_vector_type(4)));
typedef float f32x16 __attribute__((ext_vector_type(16)));
#define LAS __attribute__((address_space(3)))
#define GAS __attribute__((address_space(1)))

constexpr int R = 8;
constexpr int MS = 512;
constexpr int KS = 512;
constexpr int NN = 4096;
constexpr int NWG = 256;
constexpr int NWAVES = 12;
constexpr int NTHREADS = NWAVES * 64;
constexpr int RR = 4;
constexpr int PIECE = 4096;
constexpr int SLOT = 5120;
constexpr int MAXEXC = 510;
constexpr int LDS_OWN = RR * 7 * PIECE;
constexpr int LDS_CTRL = LDS_OWN + 8 * PIECE;
constexpr int LDS_READY = LDS_CTRL;
constexpr int LDS_DONE = LDS_CTRL + 192;
constexpr int LDS_BAD = LDS_READY + (7 & (RR - 1)) * 32;
constexpr int LDS_GO = LDS_CTRL + 232;
constexpr int LDS_ENCD = LDS_CTRL + 240;
constexpr int LDS_LUT = LDS_CTRL + 256;
constexpr int LDS_SCR = 7 * PIECE;
constexpr int LDS_ENC = LDS_CTRL + 512;
constexpr int LDS_BYTES = LDS_ENC + 4096 + 1024 + 128;
static_assert(LDS_BYTES <= 160 * 1024, "LDS");

constexpr size_t WS_CNT = 0;
constexpr size_t WS_TMO = 32768;
constexpr size_t WS_FX = 65536;
constexpr size_t WS_TK = 36864;
constexpr size_t WS_RC = 40960;
constexpr size_t WS_FR = 49152;
constexpr size_t WS_FW = 131072;
constexpr u32 NAGG = NAGG_;
constexpr u32 U0 = 64, U1 = BEND, U2 = U0 + 32 * 11;
constexpr u32 DRAWS = 512 - 2 * (32 - NPW);
constexpr size_t WS_SC = 53248;
constexpr size_t WS_DBG = 262144;
constexpr size_t WS_XB = 1ull << 20;
constexpr size_t XB_PAR = 2ull << 20;
constexpr size_t WB_SRC = (size_t)512 * SLOT;
constexpr size_t WS_ST = 5ull << 20;
constexpr size_t WS_WB = 8ull << 20;
constexpr size_t WB_PAR = 8 * WB_SRC;
constexpr size_t WS_END = WS_WB + 2 * WB_PAR;
static_assert(WS_ST + WB_SRC <= WS_WB, "staging");

constexpr u64 DEADLINE_TICKS = 40000000ull;

struct Params {
  const unsigned char* x;
  const unsigned char* w;
  const float* sx;
  const float* sw;
  float* out;
  unsigned char* peer[R];
  int rank;
  int pad;
};

#define UTILE(t) ((((t) & 7) << 3) | (((t) >> 3) & 7))
#define FMT_B(t) ((u32)(t) - U0 < U1 - U0)

__device__ __forceinline__ u64 rt() { return __builtin_amdgcn_s_memrealtime(); }

#define STAMP(slot) do {} while (0)
#define STAMP_OUT() do {} while (0)

#define ST16(base, voff, val, imm) \
  asm volatile("global_store_dwordx4 %0, %1, %2 offset:%3 sc0 sc1\n\ts_nop 1" :: "v"(voff), "v"(val), "s"(base), "i"(imm) : "memory")
#define ST16A(base, voff, val, imm) \
  asm volatile("global_store_dwordx4 %0, %1, %2 offset:%3 sc1\n\ts_nop 1" :: "v"(voff), "v"(val), "s"(base), "i"(imm) : "memory")
#define LD16(val, base, voff, imm) \
  asm volatile("global_load_dwordx4 %0, %1, %2 offset:%3" : "+v"(val) : "v"(voff), "s"(base), "i"(imm) : "memory")
#define LD16S(val, base, voff, imm) \
  asm volatile("global_load_dwordx4 %0, %1, %2 offset:%3 sc0 sc1" : "+v"(val) : "v"(voff), "s"(base), "i"(imm) : "memory")
#define VMWAIT(n) asm volatile("s_waitcnt vmcnt(%0)" :: "i"(n) : "memory")

__device__ __forceinline__ void flag_store(unsigned char* p, u32 v) {
  __hip_atomic_store((GAS u32*)p, v, __ATOMIC_RELAXED, __HIP_MEMORY_SCOPE_SYSTEM);
}
__device__ __forceinline__ u32 flag_load(const unsigned char* p) {
  return __hip_atomic_load((GAS u32*)p, __ATOMIC_RELAXED, __HIP_MEMORY_SCOPE_SYSTEM);
}
__device__ __forceinline__ u32 ag_load(const unsigned char* p) {
  return __hip_atomic_load((GAS u32*)p, __ATOMIC_RELAXED, __HIP_MEMORY_SCOPE_AGENT);
}
__device__ __forceinline__ void ag_store(unsigned char* p, u32 v) {
  __hip_atomic_store((GAS u32*)p, v, __ATOMIC_RELAXED, __HIP_MEMORY_SCOPE_AGENT);
}

template <bool SYS>
__device__ __forceinline__ bool wait_all(const unsigned char* fp, u32 want, unsigned char* ws, u32 call) {
  const u64 t0 = rt();
  for (u32 spin = 1;; ++spin) {
    const u32 v = SYS ? flag_load(fp) : ag_load(fp);
    if (__all(v >= want)) break;
    if ((spin & 15) == 0 && (__builtin_amdgcn_readfirstlane(ag_load(ws + WS_TMO)) == call || rt() - t0 > DEADLINE_TICKS)) { ag_store(ws + WS_TMO, call); return false; }
    __builtin_amdgcn_s_sleep(2);
  }
  asm volatile("" ::: "memory");
  return true;
}

__device__ __forceinline__ u32 wave_sum(u32 v) {
  v += __builtin_amdgcn_update_dpp(0u, v, 0x111, 0xf, 0xf, true);
  v += __builtin_amdgcn_update_dpp(0u, v, 0x112, 0xf, 0xf, true);
  v += __builtin_amdgcn_update_dpp(0u, v, 0x114, 0xf, 0xf, true);
  v += __builtin_amdgcn_update_dpp(0u, v, 0x118, 0xf, 0xf, true);
  return __builtin_amdgcn_readlane(v, 15) + __builtin_amdgcn_readlane(v, 31) +
         __builtin_amdgcn_readlane(v, 47) + __builtin_amdgcn_readlane(v, 63);
}

__device__ __forceinline__ void read_bfrag(i32x8& r, u32 addr, int imm) {
  LAS unsigned char* b = (LAS unsigned char*)(size_t)addr;
  i32x2 r0 = __builtin_amdgcn_ds_read_tr8_b64_v2i32((LAS i32x2*)(b + imm));
  i32x2 r1 = __builtin_amdgcn_ds_read_tr8_b64_v2i32((LAS i32x2*)(b + imm + 256));
  i32x2 r2 = __builtin_amdgcn_ds_read_tr8_b64_v2i32((LAS i32x2*)(b + imm + 512));
  i32x2 r3 = __builtin_amdgcn_ds_read_tr8_b64_v2i32((LAS i32x2*)(b + imm + 768));
  r[0] = r0.x; r[1] = r0.y; r[2] = r1.x; r[3] = r1.y; r[4] = r2.x; r[5] = r2.y; r[6] = r3.x; r[7] = r3.y;
}

__device__ __forceinline__ u32 miss_pattern(u32 m  , u32 cneg, u32 lo4, u32& code) {
  code = (m + cneg) & 0x1f1f1f1fu;
  return (code + lo4) ^ m;
}

__device__ __forceinline__ u32 wave_scan(u32 v) {
  v += __builtin_amdgcn_update_dpp(0u, v, 0x111, 0xf, 0xf, true);
  v += __builtin_amdgcn_update_dpp(0u, v, 0x112, 0xf, 0xf, true);
  v += __builtin_amdgcn_update_dpp(0u, v, 0x114, 0xf, 0xf, true);
  v += __builtin_amdgcn_update_dpp(0u, v, 0x118, 0xf, 0xf, true);
  v += __builtin_amdgcn_update_dpp(0u, v, 0x142, 0xa, 0xf, false);
  v += __builtin_amdgcn_update_dpp(0u, v, 0x143, 0xc, 0xf, false);
  return v;
}

__device__ __forceinline__ u32 gather16(const u32x4& X) {
  u32 f[4];
#pragma unroll
  for (int e = 0; e < 4; ++e) f[e] = (X[e] + 0x7c7c7c7cu) & 0x80808080u;
  const u32 b0 = __builtin_amdgcn_udot4(f[1], 0x80402010u, __builtin_amdgcn_udot4(f[0], 0x08040201u, 0u, false), false);
  const u32 b1 = __builtin_amdgcn_udot4(f[3], 0x80402010u, __builtin_amdgcn_udot4(f[2], 0x08040201u, 0u, false), false);
  return (b0 >> 7) | (b1 << 1);
}

template <int RECR>
__device__ __forceinline__ void write_records(u32 (&M)[2], u32 wa0, u32 wa1, LAS unsigned char* xs, LAS unsigned char* recs, u32 dummy, u32 vo) {
  u32 wa[2] = {wa0, wa1};
  do {
    u32 pos[2][RECR], x[2][RECR], ad[2][RECR];
#pragma unroll
    for (int r = 0; r < RECR; ++r)
#pragma unroll
      for (int h = 0; h < 2; ++h) {
        const u32 bit = __builtin_ctz(M[h] | 0x80000000u);
        pos[h][r] = vo + (bit & 15) + ((bit & 16) << 6);
        x[h][r] = xs[h * 2048 + pos[h][r]];
        ad[h][r] = M[h] ? wa[h] : dummy;
        wa[h] += 2;
        M[h] &= M[h] - 1;
      }
#pragma unroll
    for (int r = 0; r < RECR; ++r)
#pragma unroll
      for (int h = 0; h < 2; ++h)
        *(LAS unsigned short*)(recs + ad[h][r]) = (unsigned short)(pos[h][r] | (x[h][r] << 9));
  } while (__any((M[0] | M[1]) != 0));
}

__device__ __forceinline__ void apply_records(u32 ev, u32 first, u32 lim, u32 nhalf, LAS unsigned char* dst, int lane) {
#pragma unroll
  for (int h = 0; h < 2; ++h) {
    const u32 rec = h ? ev >> 16 : ev & 0xffffu;
    const u32 idx = first + 2 * lane + h;
    const u32 off = (rec & 0x7ffu) | (idx >= nhalf ? 0x800u : 0u);
    if (idx < lim)
      __hip_atomic_fetch_xor((LAS u32*)(dst + (off & ~3u)), ((rec >> 11) << 2) << (8 * (off & 3)), __ATOMIC_RELAXED, __HIP_MEMORY_SCOPE_WORKGROUP);
  }
}

__device__ __forceinline__ int choose_window(const u32x4 (&V)[4]) {
  u32 sum = 0;
#pragma unroll
  for (int t = 0; t < 4; ++t) sum = __builtin_amdgcn_sad_u8(V[t][3 - t] & 0x7c7c7c7cu, 0u, sum);
  int lo = (int)((wave_sum(sum) + 1638u) >> 12) - 6;
  lo = lo < 0 ? 0 : (lo > 22 ? 22 : lo);
  u32 cnts = 0;
#pragma unroll
  for (int k = 0; k < 3; ++k) {
    const u32 lo4 = (u32)(lo + k) * 0x04040404u, cneg = (u32)((-4 * (lo + k)) & 31) * 0x01010101u;
    u32 n = 0, code;
#pragma unroll
    for (int t = 0; t < 4; ++t)
      n += __builtin_popcount((miss_pattern(V[t][t] & 0x7f7f7f7fu, cneg, lo4, code) + 0x7c7c7c7cu) & 0x80808080u);
    cnts |= min(n, 15u) << (10 * k);
  }
  const u32 tot = wave_sum(cnts);
  const u32 n0 = tot & 1023, n1 = (tot >> 10) & 1023, n2 = tot >> 20;
  return lo + ((n1 <= n0 && n1 <= n2) ? 1 : (n0 <= n2 ? 0 : 2));
}

__device__ __forceinline__ u32 encode_piece(const u32x4 (&V)[4], int lo, u32x4 (&D)[4], u32x4& T, LAS unsigned char* scr, int lane) {
  const u32 lo4 = (u32)lo * 0x04040404u, cneg = (u32)((-4 * lo) & 31) * 0x01010101u;
  const u32 vo = lane * 16;
  u32x4 C[4];
  u32 M[2];
#pragma unroll
  for (int t = 0; t < 4; ++t) {
    u32x4 X;
#pragma unroll
    for (int e = 0; e < 4; ++e) {
      u32 code;
      X[e] = miss_pattern(V[t][e] & 0x7f7f7f7fu, cneg, lo4, code);
      C[t][e] = code | (V[t][e] & 0x80808080u);
    }
    *(LAS u32x4*)(scr + t * 1024 + vo) = X;
    const u32 m16 = gather16(X);
    if (t & 1) M[t >> 1] |= m16 << 16; else M[t >> 1] = m16;
  }
  const u32 n = __builtin_popcount(M[0]) | (__builtin_popcount(M[1]) << 16);
  const u32 inc = wave_scan(n);
  const u32 tot = __builtin_amdgcn_readlane(inc, 63);
  const u32 nhalf = tot & 0xffffu, nexc = nhalf + (tot >> 16);
  LAS unsigned char* recs = scr + 4096;
  if (nexc > MAXEXC) {
    D[0] = V[0]; D[1] = V[1]; D[2] = V[2]; T = V[3];
    D[3] = u32x4{0x80000000u, 0u, 0u, 0u};
    return nexc;
  }
  if (lane == 0) *(LAS u32*)recs = nhalf | (nexc << 10) | ((u32)lo << 20);
  write_records<3>(M, 4 + 2 * ((inc - n) & 0xffffu), 4 + 2 * (nhalf + ((inc - n) >> 16)), scr, recs, 1024 + 2 * lane, vo);
#pragma unroll
  for (int e = 0; e < 4; ++e) {
    const u32 c3 = C[3][e];
    D[0][e] = C[0][e] | ((c3 << 5) & 0x60606060u);
    D[1][e] = C[1][e] | ((c3 << 3) & 0x60606060u);
    D[2][e] = C[2][e] | ((c3 << 1) & 0x20202020u) | ((c3 >> 1) & 0x40404040u);
  }
  D[3] = *(LAS u32x4*)(recs + vo);
  return nexc;
}

__device__ __forceinline__ void unpack_piece(const unsigned char* src, LAS unsigned char* dst, int lane) {
  LAS u32x4* o = (LAS u32x4*)(dst + lane * 16);
  u32 ev = *(LAS u32*)(dst + 3072 + lane * 4);
  u32x4 D[3] = {o[0], o[64], o[128]};
  asm volatile("" : "+v"(ev), "+v"(D[0]), "+v"(D[1]), "+v"(D[2]));
  const u32 hdr = __builtin_amdgcn_readfirstlane(ev);
  if (hdr >> 31) {
    u32x4 D3;
    asm volatile("global_load_dwordx4 %0, %1, %2 sc0 sc1\n\ts_waitcnt vmcnt(0)" : "=&v"(D3) : "v"(lane * 16), "s"(src + 4096) : "memory");
    o[192] = D3;
    return;
  }
  const u32 lo4 = ((hdr >> 20) & 31) * 0x04040404u, nhalf = hdr & 1023, nexc = (hdr >> 10) & 1023;
  u32x4 O0, O1, O2, O3;
#pragma unroll
  for (int e = 0; e < 4; ++e) {
    const u32 d0 = D[0][e], d1 = D[1][e], d2 = D[2][e];
    u32 s0 = d0 & 0x60606060u, s1 = d1 & 0x60606060u, s2 = d2 & 0x60606060u;
    asm("" : "+v"(s0), "+v"(s1), "+v"(s2));
    O0[e] = (d0 ^ s0) + lo4; O1[e] = (d1 ^ s1) + lo4; O2[e] = (d2 ^ s2) + lo4;
    const u32 t = (s0 >> 5) | (s1 >> 3) | (s2 >> 1), z = t & 0x20202020u;
    O3[e] = t + lo4 + 3 * z;
  }
  o[0] = O0; o[64] = O1; o[128] = O2; o[192] = O3;
  for (u32 base = 0;;) {
    apply_records(ev, base - 2, nexc, nhalf, dst, lane);
    base += 128;
    if (base >= nexc + 2) break;
    asm volatile("global_load_dword %0, %1, %2 offset:3072 sc0 sc1\n\ts_waitcnt vmcnt(0)" : "=&v"(ev) : "v"(lane * 4 + 2 * base), "s"(src) : "memory");
  }
}

constexpr int B_HDR = 2560, B_STR = 2576;
__device__ __forceinline__ u64 tier2_exponents(int b) {
  if (b >= 10) return (u64)(4 * b) * 0x0101010101010101ull + 12 - 0x1c1814100c080400ull;
  u64 r = (u64)(4 * (b + 3));
#pragma unroll
  for (int k = 1; k < 8; ++k) r |= (u64)(4 * max(b - k, 3)) << (8 * k);
  return r;
}
__device__ __forceinline__ int choose_base(const u32x4 (&V)[4]) {
  u32 sum = 0;
#pragma unroll
  for (int t = 0; t < 4; ++t) sum = __builtin_amdgcn_sad_u8(V[t][3 - t] & 0x7c7c7c7cu, 0u, sum);
  int b = (int)((wave_sum(sum) + 1638u) >> 12) - 2;
  b = b < 0 ? 0 : (b > 26 ? 26 : b);
  u32 cnts = 0;
#pragma unroll
  for (int k = 0; k < 3; ++k) {
    const u32 c0 = (u32)(0x80 - 4 * (b + k)) * 0x01010101u;
    u32 n = 0;
#pragma unroll
    for (int t = 0; t < 4; ++t) {
      const u32 q = (V[t][t] & 0x7f7f7f7fu) + c0;
      n += __builtin_popcount(q & ~(q - 0x0c0c0c0cu) & 0x80808080u);
    }
    cnts |= min(n, 15u) << (10 * k);
  }
  const u32 tot = wave_sum(cnts);
  const u32 n0 = tot & 1023, n1 = (tot >> 10) & 1023, n2 = tot >> 20;
  return b + ((n1 >= n0 && n1 >= n2) ? 1 : (n0 >= n2 ? 0 : 2));
}

__device__ __forceinline__ u32 encode_b(const u32x4 (&V)[4], int b, u32x4 (&N)[2], LAS unsigned char* scr, LAS unsigned char* lut, int lane) {
  const u32 vo = lane * 16;
  LAS unsigned char* tail = scr + 4096;
  *(LAS u32x4*)(tail + vo) = u32x4{0u, 0u, 0u, 0u};
  *(LAS u32x4*)(tail + 1024 + vo) = u32x4{0u, 0u, 0u, 0u};
  const u32 c1 = (u32)(0x83 + 4 * (b + 3)) * 0x01010101u;
  const u64 ex = tier2_exponents(b);
  const u32 exlo = (u32)ex, exhi = (u32)(ex >> 32);
  u32 M[2], S[2] = {0u, 0u}, o[4];
  u64 acc[4];
#pragma unroll
  for (int t = 0; t < 4; ++t) {
    u32x4 X;
    u32 n1[4], k[4], e1[4];
#pragma unroll
    for (int e = 0; e < 4; ++e) {
      const u32 v = V[t][e], m = v & 0x7f7f7f7fu;
      const u32 q = c1 - m;
      const u32 F = ((((q ^ 0x80808080u) & 0xf0f0f0f0u) >> 1) + 0x78787878u) & 0x80808080u;
      const u32 f4 = F >> 4, f5 = F >> 5;
      n1[e] = ~q | f4 | f5;
      e1[e] = (n1[e] >> 3) & (n1[e] >> 2) & 0x01010101u;
      k[e] = ((q - 0x0c0c0c0cu) >> 2) & (f4 - (F >> 7));
      X[e] = (__builtin_amdgcn_perm(exhi, exlo, k[e]) ^ m) & (F - f5);
      S[t >> 1] |= (v >> (4 * (t & 1) + e)) & (0x80808080u >> (4 * (t & 1) + e));
    }
    *(LAS u32x4*)(scr + t * 1024 + vo) = X;
    const u32 m16 = gather16(X);
    if (t & 1) M[t >> 1] |= m16 << 16; else M[t >> 1] = m16;
    N[t >> 1][2 * (t & 1)] = (n1[0] & 0x0f0f0f0fu) | ((n1[1] << 4) & 0xf0f0f0f0u);
    N[t >> 1][2 * (t & 1) + 1] = (n1[2] & 0x0f0f0f0fu) | ((n1[3] << 4) & 0xf0f0f0f0u);
    u32 ob = 0;
    u64 a = 0;
#pragma unroll
    for (int e = 0; e < 4; ++e) {
      const i32x2 wm = *(LAS i32x2*)(lut + __builtin_amdgcn_udot4(e1[e], 0x40201008u, 0u, false));
      const u32 c12 = __builtin_amdgcn_udot4(k[e], (u32)wm.x, (k[e] >> 15) & (u32)wm.y, false);
      a |= (u64)c12 << ob;
      ob = __builtin_amdgcn_udot4(e1[e], 0x03030303u, ob, false);
    }
    acc[t] = a; o[t] = ob;
  }
  const u32 nb = o[0] + o[1] + o[2] + o[3];
  const u32 nr = __builtin_popcount(M[0]) | (__builtin_popcount(M[1]) << 16);
  const u32 ib = wave_scan(nb), ir = wave_scan(nr);
  const u32 bits = __builtin_amdgcn_readlane(ib, 63), totr = __builtin_amdgcn_readlane(ir, 63);
  const u32 nhalf = totr & 0xffffu, nexc = nhalf + (totr >> 16);
  const u32 rec0 = B_STR + ((bits + 31) >> 5) * 4;
  const u32 len = rec0 + 2 * nexc;
  if (bits > 3 * 2040 || len > 4032 || nexc > MAXEXC) return 0;
  *(LAS i32x2*)(tail + lane * 8) = i32x2{(int)S[0], (int)S[1]};
  if (lane == 0) *(LAS u32x4*)(tail + B_HDR - 2048) = u32x4{nhalf | (nexc << 10) | ((u32)b << 20), bits | (rec0 << 16), len, 0u};
  u32 bp = ib - nb + (B_STR - 2048) * 8;
#pragma unroll
  for (int t = 0; t < 4; ++t) {
    LAS u32* w = (LAS u32*)(tail + ((bp >> 5) << 2));
    const u32 sh = bp & 31, hi = (u32)(acc[t] >> 32);
    const u64 x = acc[t] << sh;
    __hip_atomic_fetch_or(w, (u32)x, __ATOMIC_RELAXED, __HIP_MEMORY_SCOPE_WORKGROUP);
    __hip_atomic_fetch_or(w + 1, (u32)(x >> 32), __ATOMIC_RELAXED, __HIP_MEMORY_SCOPE_WORKGROUP);
    __hip_atomic_fetch_or(w + 2, (hi >> 1) >> (31 - sh), __ATOMIC_RELAXED, __HIP_MEMORY_SCOPE_WORKGROUP);
    bp += o[t];
  }
  write_records<1>(M, rec0 - 2048 + 2 * ((ir - nr) & 0xffffu), rec0 - 2048 + 2 * (nhalf + ((ir - nr) >> 16)), scr, tail, 2048 + 2 * lane, vo);
  return len;
}

__device__ __forceinline__ void unpack_b(const unsigned char* src, LAS unsigned char* dst, LAS unsigned char* lut, int lane) {
  LAS u32x4* o = (LAS u32x4*)(dst + lane * 16);
  u32x4 N[2] = {o[0], o[64]};
  i32x2 S = *(LAS i32x2*)(dst + 2048 + lane * 8);
  i32x2 H = *(LAS i32x2*)(dst + B_HDR);
  const u32 so = B_STR + ((6 * lane) & ~3);
  u32 c0 = *(LAS u32*)(dst + so), c1 = *(LAS u32*)(dst + so + 4);
  asm volatile("" : "+v"(N[0]), "+v"(N[1]), "+v"(S), "+v"(H), "+v"(c0), "+v"(c1));
  const u32 hdr = __builtin_amdgcn_readfirstlane(H.x), h1 = __builtin_amdgcn_readfirstlane(H.y);
  if (hdr >> 31) {
    u32x4 D3, D4;
    asm volatile("global_load_dwordx4 %0, %2, %3 offset:3072 sc0 sc1\n\tglobal_load_dwordx4 %1, %4, %3 sc0 sc1\n\ts_waitcnt vmcnt(0)"
                 : "=&v"(D3), "=&v"(D4) : "v"(lane * 16), "s"(src), "v"(4096u) : "memory");
    o[192] = D3;
    *(LAS u32x4*)(dst + B_HDR) = D4;
    return;
  }
  const int b = (hdr >> 20) & 31;
  const u32 nhalf = hdr & 1023, nexc = (hdr >> 10) & 1023, bits = h1 & 0xffffu, rec0 = h1 >> 16;
  if (rec0 + 2 * nexc > 3072) {
    u32x4 D3;
    asm volatile("global_load_dwordx4 %0, %1, %2 offset:3072 sc0 sc1\n\ts_waitcnt vmcnt(0)" : "=&v"(D3) : "v"(lane * 16), "s"(src) : "memory");
    o[192] = D3;
  }
  u32 ev = *(LAS u32*)(dst + min(rec0 + lane * 4, 4092u));
  u32 c2 = 0, c3 = 0;
  if (bits > 3072) { c2 = *(LAS u32*)(dst + so + 384); c3 = *(LAS u32*)(dst + so + 388); }
  const u64 ex = tier2_exponents(b) - 0x0c0c0c0c0c0c0c0cull;
  const u32 exlo = (u32)ex, exhi = (u32)(ex >> 32);
  auto expand = [&](u32 a0, u32 a1, int p) {
    const u32 sh = (lane & 1) * 16;
    const u32 x = __builtin_amdgcn_alignbit(a1, a0, sh), y = a1 >> sh;
    const u32 g[4] = {x & 0xfffu, (x >> 12) & 0xfffu, __builtin_amdgcn_alignbit(y, x, 24) & 0xfffu, (y >> 4) & 0xfffu};
    u32x4 r;
#pragma unroll
    for (int i = 0; i < 4; ++i) {
      const u32 z = (g[i] | (g[i] << 10)) & 0x003f003fu;
      r[i] = __builtin_amdgcn_perm(exhi, exlo, (z | (z << 5)) & 0x07070707u);
    }
    o[p * 64] = r;
    o[128 + p * 64] = u32x4{r[1], r[2], r[3], (u32)__builtin_amdgcn_update_dpp(0u, r[0], 0x130, 0xf, 0xf, true)};
    if (p && lane == 0) *(LAS u32*)(dst + 2044 + 1024) = r[0];
  };
  expand(c0, c1, 0);
  if (bits > 3072) expand(c2, c3, 1);
  u32 G[8], n = 0;
#pragma unroll
  for (int i = 0; i < 8; ++i) {
    const u32 x = N[i >> 2][i & 3];
    G[i] = x & (x >> 1) & 0x44444444u;
    n += __builtin_popcount(G[i]);
  }
  u32 wa = (u32)(size_t)dst + wave_scan(n) - n;
  const u32 b4 = (u32)(4 * b) * 0x01010101u;
  u32 wad[16], sel[16];
  i32x2 W[16];
#pragma unroll
  for (int j = 0; j < 16; ++j) {
    const u32 f = ((j & 1) ? G[j >> 1] >> 4 : G[j >> 1]) & 0x04040404u;
    sel[j] = *(LAS u32*)(lut + __builtin_amdgcn_udot4(f, 0x08040201u, 0u, false));
    W[j] = *(LAS i32x2*)__builtin_assume_aligned((LAS void*)(size_t)((wa + (wa & 4u) * 511u) & ~3u), 8);
    wad[j] = wa;
    wa += __builtin_popcount(f);
  }
  asm volatile("" : "+v"(W[0]), "+v"(W[1]), "+v"(W[2]), "+v"(W[3]), "+v"(W[4]), "+v"(W[5]), "+v"(W[6]), "+v"(W[7]),
               "+v"(sel[0]), "+v"(sel[1]), "+v"(sel[2]), "+v"(sel[3]), "+v"(sel[4]), "+v"(sel[5]), "+v"(sel[6]), "+v"(sel[7]));
  asm volatile("" : "+v"(W[8]), "+v"(W[9]), "+v"(W[10]), "+v"(W[11]), "+v"(W[12]), "+v"(W[13]), "+v"(W[14]), "+v"(W[15]),
               "+v"(sel[8]), "+v"(sel[9]), "+v"(sel[10]), "+v"(sel[11]), "+v"(sel[12]), "+v"(sel[13]), "+v"(sel[14]), "+v"(sel[15]));
#pragma unroll
  for (int t = 0; t < 4; ++t) {
    u32x4 O;
#pragma unroll
    for (int e = 0; e < 4; ++e) {
      const int j = 4 * t + e;
      const u32 x = N[j >> 3][(j >> 1) & 3];
      const u32 nib = ((j & 1) ? x >> 4 : x) & 0x0f0f0f0fu;
      const u32 sg = ((u32)S[t >> 1] << (j & 7)) & 0x80808080u;
      O[e] = (nib | sg) + __builtin_amdgcn_perm(b4, __builtin_amdgcn_alignbyte((u32)W[j].y, (u32)W[j].x, wad[j]), sel[j]);
    }
    o[t * 64] = O;
  }
  for (u32 base = 0;;) {
    apply_records(ev, base, nexc, nhalf, dst, lane);
    base += 128;
    if (base >= nexc) break;
    asm volatile("global_load_dword %0, %1, %2 sc0 sc1\n\ts_waitcnt vmcnt(0)" : "=&v"(ev) : "v"(rec0 + lane * 4 + 2 * base), "s"(src) : "memory");
  }
}

__device__ __forceinline__ void init_luts(LAS unsigned char* lut, int lane) {
  if (lane < 16) {
    u32 w = 0, sel = 0, r = 0;
#pragma unroll
    for (int i = 0; i < 4; ++i) {
      const bool on = (lane >> i) & 1;
      w |= (on && r < 3 ? 1u << (3 * r) : 0u) << (8 * i);
      sel |= (on ? r : 4u) << (8 * i);
      r += on;
    }
    *(LAS i32x2*)(lut + lane * 8) = i32x2{(int)w, lane == 15 ? 0xe00 : 0};
    *(LAS u32*)(lut + 128 + lane * 4) = sel;
  }
}

__global__ void __launch_bounds__(NTHREADS, 3) __attribute__((amdgpu_num_vgpr(128))) fused_kernel(Params P) {
  extern __shared__ __attribute__((aligned(16))) unsigned char lds_raw[];
  LAS unsigned char* L = (LAS unsigned char*)lds_raw;
  volatile LAS u32* READY = (volatile LAS u32*)(L + LDS_READY);
  volatile LAS u32* DONE = (volatile LAS u32*)(L + LDS_DONE);
  volatile LAS u32* BAD = (volatile LAS u32*)(L + LDS_BAD);
  volatile LAS u32* GO = (volatile LAS u32*)(L + LDS_GO);

  const int c = blockIdx.x;
  const int lane = threadIdx.x & 63;
  const int wave = __builtin_amdgcn_readfirstlane(threadIdx.x >> 6);
  const int u = (c & 7) * 8 + ((c >> 3) & 7);
  const int mt = c >> 6;
  unsigned char* const ws = P.peer[0];
  STAMP(0);

  unsigned char* cntp = ws + WS_CNT + c * 128 + wave * 4;
#define START_BARRIER() do { __builtin_amdgcn_sched_barrier(0); __builtin_amdgcn_s_barrier(); __builtin_amdgcn_sched_barrier(0); } while (0)
  const u32 w_lane = (u32)(lane >> 1) * NN + (lane & 1) * 16;
  const u32 vo = lane * 16;

  if (wave == NWAVES - 1) {
    __builtin_amdgcn_s_waitcnt(0x0F70);
    if (c < 224) {
      const int d = 1 + c % 7, j = c / 7;
      const unsigned char* xs = P.x + (size_t)(((P.rank + d) & 7) * MS + (j >> 1) * 32) * KS + (j & 1) * 256;
      const u32 xo = (u32)(lane & 31) * KS + 32 * (lane >> 5);
      const u32 wl2 = w_lane + 32 * NN;
#define ACLOB "v128", "v129", "v130", "v131", "v132", "v133", "v134", "v135", "v136", "v137", "v138", "v139", "v140", "v141", "v142", "v143", "v144", "v145", "v146", "v147", \
              "v148", "v149", "v150", "v151", "v152", "v153", "v154", "v155", "v156", "v157", "v158", "v159", "v160", "v161", "v162", "memory"
#define LOADW(t) do { const unsigned char* s_ = P.w + (size_t)((t) >> 6) * 64 * NN + UTILE(t) * 64; \
        asm volatile("global_load_dwordx4 v[128:131], %0, %2\n\tglobal_load_dwordx4 v[132:135], %1, %2\n\t" \
                     "global_load_dwordx4 v[136:139], %0, %2 offset:32\n\tglobal_load_dwordx4 v[140:143], %1, %2 offset:32" \
                     :: "v"(w_lane), "v"(wl2), "s"(s_) : ACLOB); } while (0)
      asm volatile("global_load_dword v160, %0, %1 sc1\n\t"
                   "global_load_dwordx4 v[128:131], %2, %3\n\tglobal_load_dwordx4 v[132:135], %2, %3 offset:16\n\t"
                   "global_load_dwordx4 v[136:139], %2, %3 offset:64\n\tglobal_load_dwordx4 v[140:143], %2, %3 offset:80\n\t"
                   "global_load_dwordx4 v[144:147], %2, %3 offset:128\n\tglobal_load_dwordx4 v[148:151], %2, %3 offset:144\n\t"
                   "global_load_dwordx4 v[152:155], %2, %3 offset:192\n\tglobal_load_dwordx4 v[156:159], %2, %3 offset:208\n\t"
                   "v_mov_b32 v162, 1\n\tv_mov_b32 v161, 0"
                   :: "v"(0u), "s"(cntp), "v"(xo), "s"(xs) : ACLOB);
      START_BARRIER();
      __builtin_amdgcn_s_setprio(2);
      u32 cnt;
      asm volatile("s_waitcnt vmcnt(8)\n\tv_mov_b32 %0, v160" : "=v"(cnt) :: "memory");
      const u32 call = __builtin_amdgcn_readfirstlane(cnt) + 1;
      unsigned char* pd = P.peer[d];
      unsigned char* dst = pd + WS_XB + (call & 1) * XB_PAR + (size_t)(d * 32 + j) * 8192;
      asm volatile("s_waitcnt vmcnt(4)\n\t"
                   "global_store_dwordx4 %0, v[128:131], %1 sc0 sc1\n\tglobal_store_dwordx4 %0, v[132:135], %1 offset:1024 sc0 sc1\n\t"
                   "global_store_dwordx4 %0, v[136:139], %1 offset:2048 sc0 sc1\n\tglobal_store_dwordx4 %0, v[140:143], %1 offset:3072 sc0 sc1\n\ts_nop 1"
                   :: "v"(vo), "s"(dst) : "memory");
      LOADW(j);
      dst += 4096;
      VMWAIT(8);
      if (lane == 0) ag_store(cntp, call);
      asm volatile("global_store_dwordx4 %0, v[144:147], %1 sc0 sc1\n\tglobal_store_dwordx4 %0, v[148:151], %1 offset:1024 sc0 sc1\n\t"
                   "global_store_dwordx4 %0, v[152:155], %1 offset:2048 sc0 sc1\n\tglobal_store_dwordx4 %0, v[156:159], %1 offset:3072 sc0 sc1\n\ts_nop 1"
                   :: "v"(vo), "s"(dst) : "memory");
      GO[0] = 1u;
      STAMP(1);
      unsigned char* const wdst = pd + WS_WB + (call & 1) * WB_PAR + d * WB_SRC;
      unsigned char* const fw = pd + WS_FW + d * 64 * 128;
      unsigned char* const fr = pd + WS_FR + d * 128;
      unsigned char* const rc = ws + WS_RC + d * 8 * 128;
      unsigned char* const fx = pd + WS_FX + (d * 16 + (j >> 1)) * 128 + (j & 1) * 4;
      const unsigned char* tkp = ws + WS_TK + d * 128;
      const u32 tbase = (call - 1) * DRAWS - 64u;
      LAS unsigned char* scr = L + LDS_ENC;
      int lo = 0;
      u32 nexc = 1000, cur = j;
      asm volatile("v_mov_b32 v160, %0" :: "v"(tbase + 32 + j) : ACLOB);
      u32 p1 = 1024;
      bool xdone = false;
      u32 cr;
#define LANDED(t) do { if (lane == 0) { \
          if ((t) < NAGG * 64u) { unsigned char* a_ = rc + ((t) >> 6) * 128; \
            asm volatile("global_atomic_add v161, %0, v162, %1 sc0 sc1" :: "v"(0u), "s"(a_) : ACLOB); } \
          else flag_store((t) < 512u ? fw + UTILE(t) * 128 + ((t) >> 6) * 4 : ws + WS_CNT + c * 128 + 64, call); } \
        cr = (t) >> 6; } while (0)
#define PUBLISH(n) do { if (cr < NAGG) { u32 cv; asm volatile("s_waitcnt vmcnt(" #n ")\n\tv_mov_b32 %0, v161" : "=v"(cv) :: "memory"); \
        if ((__builtin_amdgcn_readfirstlane(cv) & 63u) == 63u && lane == 0) flag_store(fr + cr * 4, call); } } while (0)
      cr = 8;
      u32 staged = U0;
#pragma unroll 1
      while (true) {
        u32x4 D[4], T, V[4];
        u32 tk;
        {
          u32 v0, v1, v2, v3, v4, v5, v6, v7, v8, v9, v10, v11, v12, v13, v14, v15;
          if (FMT_B(p1)) VMWAIT(4); else VMWAIT(5);
          asm volatile("v_mov_b32 %16, v160\n\t"
                       "v_mov_b32 %0, v128\n\tv_mov_b32 %1, v129\n\tv_mov_b32 %2, v130\n\tv_mov_b32 %3, v131\n\t"
                       "v_mov_b32 %4, v132\n\tv_mov_b32 %5, v133\n\tv_mov_b32 %6, v134\n\tv_mov_b32 %7, v135\n\t"
                       "v_mov_b32 %8, v136\n\tv_mov_b32 %9, v137\n\tv_mov_b32 %10, v138\n\tv_mov_b32 %11, v139\n\t"
                       "v_mov_b32 %12, v140\n\tv_mov_b32 %13, v141\n\tv_mov_b32 %14, v142\n\tv_mov_b32 %15, v143"
                       : "=v"(v0), "=v"(v1), "=v"(v2), "=v"(v3), "=v"(v4), "=v"(v5), "=v"(v6), "=v"(v7), "=v"(v8), "=v"(v9), "=v"(v10),
                         "=v"(v11), "=v"(v12), "=v"(v13), "=v"(v14), "=v"(v15), "=v"(tk) :: "memory");
          V[0] = u32x4{v0, v1, v2, v3}; V[1] = u32x4{v4, v5, v6, v7}; V[2] = u32x4{v8, v9, v10, v11}; V[3] = u32x4{v12, v13, v14, v15};
        }
        const u32 nxt = __builtin_amdgcn_readfirstlane(tk) - tbase;
        const u32 nl = nxt < 512u ? nxt : cur;
        if (nl >= staged) {
          if (staged == U0) (void)wait_all<false>(ws + WS_SC + (lane & 7) * 128, call * 44u, ws, call);
          if (nl >= U2) (void)wait_all<false>(ws + WS_SC + 1024 + (lane & 7) * 128, call * 12u, ws, call);
          staged = nl >= U2 ? 512u : U2;
        }
        if (j < NPW) {
          if (lane == 0) asm volatile("global_atomic_add v160, %0, v162, %1 sc0 sc1" :: "v"(0u), "s"(tkp) : ACLOB);
        } else {
          asm volatile("global_load_dword v144, %0, %1 sc1\n\tv_mov_b32 v160, %2" :: "v"(0u), "s"(tkp), "v"(tbase + 512u) : ACLOB);
        }
        if (nl < U0) {
          LOADW(nl);
        } else {
          const unsigned char* s_ = ws + WS_ST + (size_t)nl * SLOT;
          asm volatile("global_load_dwordx4 v[128:131], %0, %1 sc1\n\tglobal_load_dwordx4 v[132:135], %0, %1 offset:1024 sc1\n\t"
                       "global_load_dwordx4 v[136:139], %0, %1 offset:2048 sc1\n\tglobal_load_dwordx4 v[140:143], %0, %1 offset:3072 sc1"
                       :: "v"(vo), "s"(s_) : ACLOB);
        }
        u32 len;
        if (cur < U0) {
          if (nexc > 96) lo = choose_window(V);
          nexc = encode_piece(V, lo, D, T, scr, lane);
          len = nexc > MAXEXC ? 5120u : 3076u + 2 * nexc;
        } else {
          D[0] = V[0]; D[1] = V[1]; D[2] = V[2]; D[3] = V[3];
          const u32 ha = __builtin_amdgcn_readfirstlane(V[3].x);
          len = cur < U1 ? __builtin_amdgcn_readlane(V[2].z, 32) : ((ha >> 31) ? 5120u : 3076u + 2 * ((ha >> 10) & 1023));
          if (len > 4096u)
            asm volatile("global_load_dwordx4 %0, %1, %2 sc1\n\ts_waitcnt vmcnt(0)" : "=&v"(T) : "v"(vo), "s"(ws + WS_ST + (size_t)cur * SLOT + 4096) : "memory");
        }
        PUBLISH(8);
        if (p1 != 1024u || !EARLY) {
          VMWAIT(5);
          if (!xdone) { if (lane == 0) flag_store(fx, call); xdone = true; }
        }
        LANDED(p1);
        dst = wdst + (size_t)((cur & ~63u) + UTILE(cur)) * SLOT;
        const u32 blk = vo & ~63u;
        ST16(dst, vo, D[0], 0); ST16(dst, vo, D[1], 1024);
        if (2048 + blk < len) ST16(dst, vo, D[2], 2048);
        if (3072 + blk < len) ST16(dst, vo, D[3], 3072);
        if (4096 + blk < len) { unsigned char* d2 = dst + 4096; ST16(d2, vo, T, 0); }
        p1 = cur;
        if ((cur & 32) == 0) STAMP(8 + (cur >> 6));
        if (nxt >= 512u) break;
        cur = nxt;
      }
      VMWAIT(0);
      PUBLISH(0);
      if (!xdone && lane == 0) flag_store(fx, call);
      LANDED(p1);
      STAMP(3);
      PUBLISH(0);
#undef LANDED
#undef PUBLISH
#undef LOADW
#undef ACLOB
    } else {
      START_BARRIER();
      GO[0] = 1u;
    }
    STAMP_OUT();
    return;
  }
  const u32 cnt_old = ag_load(cntp);
  if (wave == 5) asm volatile("ds_write_b32 %0, %1\n\ts_waitcnt lgkmcnt(0)" :: "v"((u32)(size_t)L + LDS_CTRL + lane * 4), "v"(0u) : "memory");
  if (wave == 6) { init_luts(L + LDS_LUT, lane); asm volatile("s_waitcnt lgkmcnt(0)" ::: "memory"); }
  u32 call;
  if (c >= 224) {
    START_BARRIER();
    call = __builtin_amdgcn_readfirstlane(cnt_old) + 1;
    LAS unsigned char* scr = L + LDS_SCR + wave * 6400;
#pragma unroll 1
    for (u32 e = U0 + (c - 224) + 32 * wave; e < 512u; e += 32 * 11) {
      const unsigned char* src = P.w + (size_t)(e >> 6) * 64 * NN + UTILE(e) * 64 + w_lane;
      u32x4 V[4], D[4], T;
      V[0] = *(const GAS u32x4*)(src);
      V[1] = *(const GAS u32x4*)(src + 32 * NN);
      V[2] = *(const GAS u32x4*)(src + 32);
      V[3] = *(const GAS u32x4*)(src + 32 * NN + 32);
      u32 len;
      if (e >= U1) {
        const u32 nexc = encode_piece(V, choose_window(V), D, T, scr, lane);
        len = nexc > MAXEXC ? 5120u : 3076u + 2 * nexc;
      } else {
        u32x4 N[2];
        len = encode_b(V, choose_base(V), N, scr, L + LDS_LUT, lane);
        if (len) {
          D[0] = N[0]; D[1] = N[1];
          D[2] = *(LAS u32x4*)(scr + 4096 + vo); D[3] = *(LAS u32x4*)(scr + 5120 + vo);
        } else {
          len = 4112;
          D[0] = V[0]; D[1] = V[1]; D[3] = V[3];
          D[2] = lane == 32 ? u32x4{0x80000000u, 0u, 4112u, 0u} : V[2];
#pragma unroll
          for (int q = 0; q < 4; ++q) T[q] = __builtin_amdgcn_readlane(V[2][q], 32);
        }
      }
      unsigned char* dst = ws + WS_ST + (size_t)e * SLOT;
      ST16A(dst, vo, D[0], 0); ST16A(dst, vo, D[1], 1024); ST16A(dst, vo, D[2], 2048);
      if (3072 + (vo & ~63u) < len) ST16A(dst, vo, D[3], 3072);
      if (4096 + (vo & ~63u) < len) { unsigned char* d2 = dst + 4096; ST16A(d2, vo, T, 0); }
      if (e < U2) STAMP(6);
      VMWAIT(0);
      if (lane == 0) __hip_atomic_fetch_add((GAS u32*)(ws + WS_SC + (e >= U2 ? 1024 : 0) + (e & 7) * 128), 1u, __ATOMIC_RELAXED, __HIP_MEMORY_SCOPE_AGENT);
      STAMP(5);
    }
    if (lane == 0) __hip_atomic_fetch_add((LAS u32*)(L + LDS_ENCD), 1u, __ATOMIC_RELAXED, __HIP_MEMORY_SCOPE_WORKGROUP);
    while (*(volatile LAS u32*)(L + LDS_ENCD) < 11u) __builtin_amdgcn_s_sleep(2);
    asm volatile("" ::: "memory");
  } else {
    START_BARRIER();
    call = __builtin_amdgcn_readfirstlane(cnt_old) + 1;
  }
  if (lane == 0) ag_store(cntp, call);
  while (GO[0] == 0u) __builtin_amdgcn_s_sleep(4);

  if (wave < 4) {
    const int mymb = mt * 4 + wave;
    const u32 par = call & 1;
    const float scale = P.sx[0] * P.sw[0];
    const u32 b_lane = (u32)(size_t)L + ((lane >> 5) * 32 + ((lane & 15) >> 1)) * 32 + ((lane >> 4) & 1) * 16 + (lane & 1) * 8;
    f32x16 acc0 = {}, acc1 = {};
    u32 rv;
    u32x4 AL[8] = {}, AH[8] = {};
#define AWAIT() asm volatile("s_waitcnt vmcnt(0)" : "+v"(AL[0]), "+v"(AL[1]), "+v"(AL[2]), "+v"(AL[3]), "+v"(AL[4]), "+v"(AL[5]), "+v"(AL[6]), "+v"(AL[7]), \
                             "+v"(AH[0]), "+v"(AH[1]), "+v"(AH[2]), "+v"(AH[3]), "+v"(AH[4]), "+v"(AH[5]), "+v"(AH[6]), "+v"(AH[7]) :: "memory")
#define MMA(p, off) do { \
          i32x8 b0, b1; read_bfrag(b0, baddr, off); read_bfrag(b1, baddr, off + 2048); \
          const i32x8 a = {(int)AL[p].x, (int)AL[p].y, (int)AL[p].z, (int)AL[p].w, (int)AH[p].x, (int)AH[p].y, (int)AH[p].z, (int)AH[p].w}; \
          __builtin_amdgcn_sched_barrier(0); \
          acc0 = __builtin_amdgcn_mfma_scale_f32_32x32x64_f8f6f4(b0, a, acc0, 1, 0, 0, 0, 0, 0); \
          acc1 = __builtin_amdgcn_mfma_scale_f32_32x32x64_f8f6f4(b1, a, acc1, 1, 0, 0, 0, 0, 0); } while (0)
#define POLL(row, want) do { while (true) { \
          rv = READY[(row) * 8 + (lane & 7)]; \
          m = (u32)__ballot(rv >= (u32)(want)) & rem; \
          if (m) break; \
          __builtin_amdgcn_s_sleep(1); } \
        asm volatile("" ::: "memory"); rem &= ~m; } while (0)
    {
      const unsigned char* xs = P.x + (size_t)(P.rank * MS + mymb * 32) * KS;
      const u32 xo = (u32)(lane & 31) * KS + 32 * (lane >> 5);
#define LA(p) LD16(AL[p], xs, xo, p * 64); LD16(AH[p], xs, xo, p * 64 + 16);
      LA(0) LA(1) LA(2) LA(3) LA(4) LA(5) LA(6) LA(7)
#undef LA
      AWAIT();
      const u32 baddr = b_lane + LDS_OWN;
      u32 rem = 0xFFu, m;
      do {
        POLL(RR, 1);
#define STEP(p) if (m & (1u << p)) MMA(p, p * PIECE);
        STEP(0) STEP(1) STEP(2) STEP(3) STEP(4) STEP(5) STEP(6) STEP(7)
#undef STEP
      } while (rem);
    }
    STAMP(7);
    if (!wait_all<true>(ws + WS_FX + ((1 + (lane & 7) % 7) * 16 + mymb) * 128 + ((lane >> 3) & 1) * 4, call, ws, call)) BAD[0] = 1u;
    STAMP(2);
    const unsigned char* xb = ws + WS_XB + par * XB_PAR + (size_t)mymb * 16384;
#define LA(p) do { const unsigned char* s_ = xb + p * 262144; LD16S(AL[p], s_, vo, 0); LD16S(AH[p], s_, vo, 1024); } while (0)
    LA(1); LA(2); LA(3); LA(4); LA(5); LA(6); LA(7);
#pragma unroll 1
    for (int i = 0; i < 8; ++i) {
      const int row = i & (RR - 1);
      const u32 baddr = b_lane + row * 7 * PIECE;
      xb += 2048;
      AWAIT();
      u32 rem = 0xFEu, m;
      do {
        POLL(row, i + 1);
#define STEP(p) if (m & (1u << p)) { MMA(p, (p - 1) * PIECE); if (i < 7) LA(p); }
        STEP(1) STEP(2) STEP(3) STEP(4) STEP(5) STEP(6) STEP(7)
#undef STEP
      } while (rem);
      asm volatile("s_waitcnt lgkmcnt(0)" ::: "memory");
      if (lane == 0) DONE[wave] = (u32)(i + 1);
      STAMP(8 + i);
    }
#undef LA
    AWAIT();
    const bool bad = __builtin_amdgcn_readfirstlane(rv) != 0u;
    u32x4 o[8];
    const float sc = bad ? __builtin_nanf("") : scale;
    acc0 *= sc; acc1 *= sc;
#pragma unroll
    for (int q = 0; q < 4; ++q)
#pragma unroll
      for (int e = 0; e < 4; ++e) {
        o[q][e] = __builtin_bit_cast(u32, __builtin_elementwise_maximum(acc0[4 * q + e], 0.f));
        o[4 + q][e] = __builtin_bit_cast(u32, __builtin_elementwise_maximum(acc1[4 * q + e], 0.f));
      }
    const unsigned char* ob = (const unsigned char*)(P.out + (size_t)mymb * 32 * NN + u * 64);
#define OST(val, voff, imm) asm volatile("global_store_dwordx4 %0, %1, %2 offset:%3 " EPI_BITS "\n\ts_nop 1" :: "v"(voff), "v"(val), "s"(ob), "i"(imm) : "memory")
    {
      LAS unsigned char* tb = L + LDS_OWN + wave * 8192;
      const int m = lane & 31, h = lane >> 5;
#pragma unroll
      for (int q = 0; q < 8; ++q) *(LAS u32x4*)(tb + m * 256 + (((2 * q + h) ^ (m & 15)) << 4)) = o[q];
      const int r0 = lane >> 4, ch = lane & 15;
#pragma unroll
      for (int t = 0; t < 8; ++t) o[t] = *(LAS u32x4*)(tb + (4 * t + r0) * 256 + ((ch ^ ((4 * t + r0) & 15)) << 4));
      u32 oo = (u32)r0 * NN * 4 + ch * 16;
#pragma unroll
      for (int t = 0; t < 8; ++t) { OST(o[t], oo, 0); oo += 4 * NN * 4; }
    }
#undef OST
    STAMP(3);
  } else {
    const int d = wave - 3;
    {
      const unsigned char* src = P.w + u * 64 + w_lane;
      for (int i = d - 1; i < 8; i += 7) {
        const unsigned char* s = src + (size_t)i * 64 * NN;
        LAS unsigned char* dst = L + LDS_OWN + i * PIECE;
        __builtin_amdgcn_global_load_lds((const GAS u32*)(s), (LAS u32*)(dst), 16, 0, 0);
        __builtin_amdgcn_global_load_lds((const GAS u32*)(s + 32 * NN), (LAS u32*)(dst + 1024), 16, 0, 0);
        __builtin_amdgcn_global_load_lds((const GAS u32*)(s + 32), (LAS u32*)(dst + 2048), 16, 0, 0);
        __builtin_amdgcn_global_load_lds((const GAS u32*)(s + 32 * NN + 32), (LAS u32*)(dst + 3072), 16, 0, 0);
      }
    }
    VMWAIT(0);
    if (lane == 0) { READY[RR * 8 + d - 1] = 1u; if (d == 1) READY[RR * 8 + 7] = 1u; }
    const unsigned char* fp = ws + ((lane & 7) < NAGG ? WS_FR + d * 128 : WS_FW + (d * 64 + u) * 128) + (lane & 7) * 4;
    const unsigned char* src = ws + WS_WB + (call & 1) * WB_PAR + d * WB_SRC + (size_t)u * SLOT;
    u32 seen = 0;
    bool dead = false, pre = false;
    const unsigned char* tmop = ws + WS_TMO;
    const u32 l4 = lane * 4;
#define FMT_A(i_) (!FMT_B((i_) * 64 + UTILE(u)))
#define FETCH(i_, s_) do { u32 keep_; const u32 la_ = (u32)(size_t)L + ((((i_) & (RR - 1)) * 7 + d - 1) * PIECE); \
      asm volatile("s_mov_b32 %0, m0\n\ts_mov_b32 m0, %3\n\ts_nop 0\n\t" \
                   "global_load_lds_dwordx4 %1, %2 sc0 sc1\n\t" \
                   "global_load_lds_dwordx4 %1, %2 offset:1024 sc0 sc1\n\t" \
                   "global_load_lds_dwordx4 %1, %2 offset:2048 sc0 sc1\n\t" \
                   "s_mov_b32 m0, %0" : "=&s"(keep_) : "v"(vo), "s"(s_), "s"(la_) : "memory"); \
      if (FMT_A(i_)) \
        asm volatile("s_mov_b32 %0, m0\n\ts_mov_b32 m0, %3\n\ts_nop 0\n\tglobal_load_lds_dword %1, %2 offset:3072 sc0 sc1\n\ts_mov_b32 m0, %0" \
                     : "=&s"(keep_) : "v"(l4), "s"(s_), "s"(la_) : "memory"); } while (0)
#pragma unroll 1
    for (int i = 0; i < 8; ++i, src += 64 * SLOT) {
      if (!pre) {
        if (i >= RR) {
          while (!__all(DONE[lane & 3] + RR > (u32)i)) __builtin_amdgcn_s_sleep(2);
          asm volatile("" ::: "memory");
        }
        if (!((seen >> i) & 1) && !dead) {
          const u64 t0 = rt();
          for (u32 spin = 1;; ++spin) {
            seen = (u32)__ballot(flag_load(fp) >= call);
            if ((seen >> i) & 1) break;
            if ((spin & 63) == 0 && (__builtin_amdgcn_readfirstlane(ag_load(tmop)) == call || rt() - t0 > DEADLINE_TICKS)) { dead = true; break; }
            if (i < 7) __builtin_amdgcn_s_sleep(2);
          }
          if (dead) ag_store(ws + WS_TMO, call);
          asm volatile("" ::: "memory");
        }
        if (i == 7) STAMP(4);
        if (!dead) FETCH(i, src);
      }
      pre = !dead && i < 7 && ((seen >> (i + 1)) & 1) && __all(DONE[lane & 3] + RR > (u32)i + 1);
      if (pre) {
        const unsigned char* s2 = src + 64 * SLOT;
        FETCH(i + 1, s2);
        VMWAIT(3);
      } else {
        if (i == 7) dead = dead || __builtin_amdgcn_readfirstlane(ag_load(tmop)) == call;
        VMWAIT(0);
      }
      if (i == 7) STAMP(5);
      const int row = i & (RR - 1);
      if (dead) BAD[0] = 1u;
      else if (FMT_A(i)) unpack_piece(src, L + (row * 7 + d - 1) * PIECE, lane);
      else unpack_b(src, L + (row * 7 + d - 1) * PIECE, L + LDS_LUT + 128, lane);
      asm volatile("s_waitcnt lgkmcnt(0)" ::: "memory");
      READY[row * 8 + d] = (u32)(i + 1);
      STAMP(8 + i);
    }
#undef FETCH
#undef FMT_A
    STAMP(3);
  }
  STAMP_OUT();
}

}

extern "C" void kernel_launch(int rank, int nranks, ncclComm_t comm,
                              hipStream_t stream,
                              void* const* d_in, const int* in_sizes, int n_in,
                              void* d_out, int out_size,
                              const int* in_dtypes, int out_dtype,
                              void* const* d_out_all,
                              void* const* d_ws_all, size_t ws_bytes) {
  (void)comm; (void)in_dtypes; (void)out_dtype; (void)d_out_all;
  if (nranks != R || (unsigned)rank >= (unsigned)R || n_in != 4 || ws_bytes < WS_END ||
      in_sizes[0] != 4096 * KS || in_sizes[1] != KS * NN || out_size != MS * NN) {
    fprintf(stderr, "kernel_launch: unexpected problem shape\n");
    return;
  }
  HIP_CHECK(hipFuncSetAttribute((const void*)fused_kernel, hipFuncAttributeMaxDynamicSharedMemorySize, LDS_BYTES));
  Params P;
  P.x = (const unsigned char*)d_in[0];
  P.w = (const unsigned char*)d_in[1];
  P.sx = (const float*)d_in[2];
  P.sw = (const float*)d_in[3];
  P.out = (float*)d_out;
  for (int d = 0; d < R; ++d) P.peer[d] = (unsigned char*)d_ws_all[(rank + d) % R];
  P.rank = rank;
  P.pad = 0;
  hipLaunchKernelGGL(fused_kernel, dim3(NWG), dim3(NTHREADS), LDS_BYTES, stream, P);
  HIP_CHECK(hipGetLastError());
}
```

```cpp
#include <hip/hip_runtime.h>
#include <rccl/rccl.h>
#include <cstdio>

#define HIP_CHECK(expr) do { hipError_t _e = (expr); if (_e != hipSuccess) { \
    fprintf(stderr, "HIP error %s at %s:%d\n", hipGetErrorString(_e), __FILE__, __LINE__); \
    return; } } while (0)

#define BEND 416
#define NPW 19

namespace {

typedef unsigned u32;
typedef unsigned long long u64;
typedef unsigned u32x4 __attribute__((ext_vector_type(4)));
typedef int i32x2 __attribute__((ext_vector_type(2)));
typedef int i32x8 __attribute__((ext_vector_type(8)));
typedef float f32x4 __attribute__((ext_vector_type(4)));
typedef float f32x16 __attribute__((ext_vector_type(16)));
#define LAS __attribute__((address_space(3)))
#define GAS __attribute__((address_space(1)))

constexpr int R = 8;
constexpr int MS = 512;
constexpr int KS = 512;
constexpr int NN = 4096;
constexpr int NWG = 256;
constexpr int NWAVES = 12;
constexpr int NTHREADS = NWAVES * 64;
constexpr int RR = 4;
constexpr int PIECE = 4096;
constexpr int SLOT = 5120;
constexpr int MAXEXC = 510;
constexpr int LDS_OWN = RR * 7 * PIECE;
constexpr int LDS_CTRL = LDS_OWN + 8 * PIECE;
constexpr int LDS_READY = LDS_CTRL;
constexpr int LDS_DONE = LDS_CTRL + 192;
constexpr int LDS_BAD = LDS_READY + (7 & (RR - 1)) * 32;
constexpr int LDS_GO = LDS_CTRL + 232;
constexpr int LDS_ENCD = LDS_CTRL + 240;
constexpr int LDS_STG = LDS_CTRL + 248;
constexpr int LDS_LUT = LDS_CTRL + 256;
constexpr int LDS_SCR = 7 * PIECE;
constexpr int LDS_ENC = LDS_CTRL + 512;
constexpr int LDS_BYTES = LDS_ENC + 4096 + 1024 + 128;
static_assert(LDS_BYTES <= 160 * 1024, "LDS");

constexpr size_t WS_CNT = 0;
constexpr size_t WS_TMO = 32768;
constexpr size_t WS_TK = 36864;
constexpr size_t WS_FX = 65536;
constexpr size_t WS_FW = 131072;
constexpr u32 U0 = 64, U1 = BEND, U2 = U0 + 32 * 11;
constexpr u32 DRAWS = 512 - 2 * (32 - NPW);
constexpr size_t WS_SC = 53248;
constexpr size_t WS_DBG = 524288;
constexpr size_t WS_XB = 1ull << 20;
constexpr size_t XB_PAR = 2ull << 20;
constexpr size_t WB_SRC = (size_t)512 * SLOT;
constexpr size_t WS_ST = 5ull << 20;
constexpr size_t WS_WB = 8ull << 20;
constexpr size_t WB_PAR = 8 * WB_SRC;
constexpr size_t WS_END = WS_WB + 2 * WB_PAR;
static_assert(WS_ST + WB_SRC <= WS_WB, "staging");

constexpr u64 DEADLINE_TICKS = 40000000ull;

struct Params {
  const unsigned char* x;
  const unsigned char* w;
  const float* sx;
  const float* sw;
  float* out;
  unsigned char* peer[R];
  int rank;
  int pad;
};

#define UTILE(t) ((((t) & 7) << 3) | (((t) >> 3) & 7))
#define FMT_B(t) ((u32)(t) - U0 < U1 - U0)

__device__ __forceinline__ u64 rt() { return __builtin_amdgcn_s_memrealtime(); }

#define STAMP(slot) do {} while (0)
#define STAMP_OUT() do {} while (0)

#define ST16(base, voff, val, imm) \
  asm volatile("global_store_dwordx4 %0, %1, %2 offset:%3 sc0 sc1\n\ts_nop 1" :: "v"(voff), "v"(val), "s"(base), "i"(imm) : "memory")
#define ST16A(base, voff, val, imm) \
  asm volatile("global_store_dwordx4 %0, %1, %2 offset:%3 sc1\n\ts_nop 1" :: "v"(voff), "v"(val), "s"(base), "i"(imm) : "memory")
#define LD16(val, base, voff, imm) \
  asm volatile("global_load_dwordx4 %0, %1, %2 offset:%3" : "+v"(val) : "v"(voff), "s"(base), "i"(imm) : "memory")
#define LD16S(val, base, voff, imm) \
  asm volatile("global_load_dwordx4 %0, %1, %2 offset:%3 sc0 sc1" : "+v"(val) : "v"(voff), "s"(base), "i"(imm) : "memory")
#define VMWAIT(n) asm volatile("s_waitcnt vmcnt(%0)" :: "i"(n) : "memory")

__device__ __forceinline__ void flag_store(unsigned char* p, u32 v) {
  __hip_atomic_store((GAS u32*)p, v, __ATOMIC_RELAXED, __HIP_MEMORY_SCOPE_SYSTEM);
}
__device__ __forceinline__ u32 flag_load(const unsigned char* p) {
  return __hip_atomic_load((GAS u32*)p, __ATOMIC_RELAXED, __HIP_MEMORY_SCOPE_SYSTEM);
}
__device__ __forceinline__ u32 ag_load(const unsigned char* p) {
  return __hip_atomic_load((GAS u32*)p, __ATOMIC_RELAXED, __HIP_MEMORY_SCOPE_AGENT);
}
__device__ __forceinline__ void ag_store(unsigned char* p, u32 v) {
  __hip_atomic_store((GAS u32*)p, v, __ATOMIC_RELAXED, __HIP_MEMORY_SCOPE_AGENT);
}

template <bool SYS>
__device__ __forceinline__ bool wait_all(const unsigned char* fp, u32 want, unsigned char* ws, u32 call) {
  const u64 t0 = rt();
  for (u32 spin = 1;; ++spin) {
    const u32 v = SYS ? flag_load(fp) : ag_load(fp);
    if (__all(v >= want)) break;
    if ((spin & 15) == 0 && (__builtin_amdgcn_readfirstlane(ag_load(ws + WS_TMO)) == call || rt() - t0 > DEADLINE_TICKS)) { ag_store(ws + WS_TMO, call); return false; }
    __builtin_amdgcn_s_sleep(2);
  }
  asm volatile("" ::: "memory");
  return true;
}

__device__ __forceinline__ u32 wave_sum(u32 v) {
  v += __builtin_amdgcn_update_dpp(0u, v, 0x111, 0xf, 0xf, true);
  v += __builtin_amdgcn_update_dpp(0u, v, 0x112, 0xf, 0xf, true);
  v += __builtin_amdgcn_update_dpp(0u, v, 0x114, 0xf, 0xf, true);
  v += __builtin_amdgcn_update_dpp(0u, v, 0x118, 0xf, 0xf, true);
  return __builtin_amdgcn_readlane(v, 15) + __builtin_amdgcn_readlane(v, 31) +
         __builtin_amdgcn_readlane(v, 47) + __builtin_amdgcn_readlane(v, 63);
}

__device__ __forceinline__ void read_bfrag(i32x8& r, u32 addr, int imm) {
  LAS unsigned char* b = (LAS unsigned char*)(size_t)addr;
  i32x2 r0 = __builtin_amdgcn_ds_read_tr8_b64_v2i32((LAS i32x2*)(b + imm));
  i32x2 r1 = __builtin_amdgcn_ds_read_tr8_b64_v2i32((LAS i32x2*)(b + imm + 256));
  i32x2 r2 = __builtin_amdgcn_ds_read_tr8_b64_v2i32((LAS i32x2*)(b + imm + 512));
  i32x2 r3 = __builtin_amdgcn_ds_read_tr8_b64_v2i32((LAS i32x2*)(b + imm + 768));
  r[0] = r0.x; r[1] = r0.y; r[2] = r1.x; r[3] = r1.y; r[4] = r2.x; r[5] = r2.y; r[6] = r3.x; r[7] = r3.y;
}

__device__ __forceinline__ u32 miss_pattern(u32 m  , u32 cneg, u32 lo4, u32& code) {
  code = (m + cneg) & 0x1f1f1f1fu;
  return (code + lo4) ^ m;
}

__device__ __forceinline__ u32 wave_scan(u32 v) {
  v += __builtin_amdgcn_update_dpp(0u, v, 0x111, 0xf, 0xf, true);
  v += __builtin_amdgcn_update_dpp(0u, v, 0x112, 0xf, 0xf, true);
  v += __builtin_amdgcn_update_dpp(0u, v, 0x114, 0xf, 0xf, true);
  v += __builtin_amdgcn_update_dpp(0u, v, 0x118, 0xf, 0xf, true);
  v += __builtin_amdgcn_update_dpp(0u, v, 0x142, 0xa, 0xf, false);
  v += __builtin_amdgcn_update_dpp(0u, v, 0x143, 0xc, 0xf, false);
  return v;
}

__device__ __forceinline__ u32 gather16(const u32x4& X) {
  u32 f[4];
#pragma unroll
  for (int e = 0; e < 4; ++e) f[e] = (X[e] + 0x7c7c7c7cu) & 0x80808080u;
  const u32 b0 = __builtin_amdgcn_udot4(f[1], 0x80402010u, __builtin_amdgcn_udot4(f[0], 0x08040201u, 0u, false), false);
  const u32 b1 = __builtin_amdgcn_udot4(f[3], 0x80402010u, __builtin_amdgcn_udot4(f[2], 0x08040201u, 0u, false), false);
  return (b0 >> 7) | (b1 << 1);
}

template <int RECR>
__device__ __forceinline__ void write_records(u32 (&M)[2], u32 wa0, u32 wa1, LAS unsigned char* xs, LAS unsigned char* recs, u32 dummy, u32 vo) {
  u32 wa[2] = {wa0, wa1};
  do {
    u32 pos[2][RECR], x[2][RECR], ad[2][RECR];
#pragma unroll
    for (int r = 0; r < RECR; ++r)
#pragma unroll
      for (int h = 0; h < 2; ++h) {
        const u32 bit = __builtin_ctz(M[h] | 0x80000000u);
        pos[h][r] = vo + (bit & 15) + ((bit & 16) << 6);
        x[h][r] = xs[h * 2048 + pos[h][r]];
        ad[h][r] = M[h] ? wa[h] : dummy;
        wa[h] += 2;
        M[h] &= M[h] - 1;
      }
#pragma unroll
    for (int r = 0; r < RECR; ++r)
#pragma unroll
      for (int h = 0; h < 2; ++h)
        *(LAS unsigned short*)(recs + ad[h][r]) = (unsigned short)(pos[h][r] | (x[h][r] << 9));
  } while (__any((M[0] | M[1]) != 0));
}

__device__ __forceinline__ void apply_records(u32 ev, u32 first, u32 lim, u32 nhalf, LAS unsigned char* dst, int lane) {
#pragma unroll
  for (int h = 0; h < 2; ++h) {
    const u32 rec = h ? ev >> 16 : ev & 0xffffu;
    const u32 idx = first + 2 * lane + h;
    const u32 off = (rec & 0x7ffu) | (idx >= nhalf ? 0x800u : 0u);
    if (idx < lim)
      __hip_atomic_fetch_xor((LAS u32*)(dst + (off & ~3u)), ((rec >> 11) << 2) << (8 * (off & 3)), __ATOMIC_RELAXED, __HIP_MEMORY_SCOPE_WORKGROUP);
  }
}

__device__ __forceinline__ int choose_window(const u32x4 (&V)[4]) {
  u32 sum = 0;
#pragma unroll
  for (int t = 0; t < 4; ++t) sum = __builtin_amdgcn_sad_u8(V[t][3 - t] & 0x7c7c7c7cu, 0u, sum);
  int lo = (int)((wave_sum(sum) + 1638u) >> 12) - 6;
  lo = lo < 0 ? 0 : (lo > 22 ? 22 : lo);
  u32 cnts = 0;
#pragma unroll
  for (int k = 0; k < 3; ++k) {
    const u32 lo4 = (u32)(lo + k) * 0x04040404u, cneg = (u32)((-4 * (lo + k)) & 31) * 0x01010101u;
    u32 n = 0, code;
#pragma unroll
    for (int t = 0; t < 4; ++t)
      n += __builtin_popcount((miss_pattern(V[t][t] & 0x7f7f7f7fu, cneg, lo4, code) + 0x7c7c7c7cu) & 0x80808080u);
    cnts |= min(n, 15u) << (10 * k);
  }
  const u32 tot = wave_sum(cnts);
  const u32 n0 = tot & 1023, n1 = (tot >> 10) & 1023, n2 = tot >> 20;
  return lo + ((n1 <= n0 && n1 <= n2) ? 1 : (n0 <= n2 ? 0 : 2));
}

__device__ __forceinline__ u32 encode_piece(const u32x4 (&V)[4], int lo, u32x4 (&D)[4], u32x4& T, LAS unsigned char* scr, int lane) {
  const u32 lo4 = (u32)lo * 0x04040404u, cneg = (u32)((-4 * lo) & 31) * 0x01010101u;
  const u32 vo = lane * 16;
  u32x4 C[4];
  u32 M[2];
#pragma unroll
  for (int t = 0; t < 4; ++t) {
    u32x4 X;
#pragma unroll
    for (int e = 0; e < 4; ++e) {
      u32 code;
      X[e] = miss_pattern(V[t][e] & 0x7f7f7f7fu, cneg, lo4, code);
      C[t][e] = code | (V[t][e] & 0x80808080u);
    }
    *(LAS u32x4*)(scr + t * 1024 + vo) = X;
    const u32 m16 = gather16(X);
    if (t & 1) M[t >> 1] |= m16 << 16; else M[t >> 1] = m16;
  }
  const u32 n = __builtin_popcount(M[0]) | (__builtin_popcount(M[1]) << 16);
  const u32 inc = wave_scan(n);
  const u32 tot = __builtin_amdgcn_readlane(inc, 63);
  const u32 nhalf = tot & 0xffffu, nexc = nhalf + (tot >> 16);
  LAS unsigned char* recs = scr + 4096;
  if (nexc > MAXEXC) {
    D[0] = V[0]; D[1] = V[1]; D[2] = V[2]; T = V[3];
    D[3] = u32x4{0x80000000u, 0u, 0u, 0u};
    return nexc;
  }
  if (lane == 0) *(LAS u32*)recs = nhalf | (nexc << 10) | ((u32)lo << 20);
  write_records<3>(M, 4 + 2 * ((inc - n) & 0xffffu), 4 + 2 * (nhalf + ((inc - n) >> 16)), scr, recs, 1024 + 2 * lane, vo);
#pragma unroll
  for (int e = 0; e < 4; ++e) {
    const u32 c3 = C[3][e];
    D[0][e] = C[0][e] | ((c3 << 5) & 0x60606060u);
    D[1][e] = C[1][e] | ((c3 << 3) & 0x60606060u);
    D[2][e] = C[2][e] | ((c3 << 1) & 0x20202020u) | ((c3 >> 1) & 0x40404040u);
  }
  D[3] = *(LAS u32x4*)(recs + vo);
  return nexc;
}

__device__ __forceinline__ void unpack_piece(const unsigned char* src, LAS unsigned char* dst, int lane) {
  LAS u32x4* o = (LAS u32x4*)(dst + lane * 16);
  u32 ev = *(LAS u32*)(dst + 3072 + lane * 4);
  u32x4 D[3] = {o[0], o[64], o[128]};
  asm volatile("" : "+v"(ev), "+v"(D[0]), "+v"(D[1]), "+v"(D[2]));
  const u32 hdr = __builtin_amdgcn_readfirstlane(ev);
  if (hdr >> 31) {
    u32x4 D3;
    asm volatile("global_load_dwordx4 %0, %1, %2 sc0 sc1\n\ts_waitcnt vmcnt(0)" : "=&v"(D3) : "v"(lane * 16), "s"(src + 4096) : "memory");
    o[192] = D3;
    return;
  }
  const u32 lo4 = ((hdr >> 20) & 31) * 0x04040404u, nhalf = hdr & 1023, nexc = (hdr >> 10) & 1023;
  u32x4 O0, O1, O2, O3;
#pragma unroll
  for (int e = 0; e < 4; ++e) {
    const u32 d0 = D[0][e], d1 = D[1][e], d2 = D[2][e];
    u32 s0 = d0 & 0x60606060u, s1 = d1 & 0x60606060u, s2 = d2 & 0x60606060u;
    asm("" : "+v"(s0), "+v"(s1), "+v"(s2));
    O0[e] = (d0 ^ s0) + lo4; O1[e] = (d1 ^ s1) + lo4; O2[e] = (d2 ^ s2) + lo4;
    const u32 t = (s0 >> 5) | (s1 >> 3) | (s2 >> 1), z = t & 0x20202020u;
    O3[e] = t + lo4 + 3 * z;
  }
  o[0] = O0; o[64] = O1; o[128] = O2; o[192] = O3;
  for (u32 base = 0;;) {
    apply_records(ev, base - 2, nexc, nhalf, dst, lane);
    base += 128;
    if (base >= nexc + 2) break;
    asm volatile("global_load_dword %0, %1, %2 offset:3072 sc0 sc1\n\ts_waitcnt vmcnt(0)" : "=&v"(ev) : "v"(lane * 4 + 2 * base), "s"(src) : "memory");
  }
}

constexpr int B_HDR = 2560, B_STR = 2576;
__device__ __forceinline__ u64 tier2_exponents(int b) {
  if (b >= 10) return (u64)(4 * b) * 0x0101010101010101ull + 12 - 0x1c1814100c080400ull;
  u64 r = (u64)(4 * (b + 3));
#pragma unroll
  for (int k = 1; k < 8; ++k) r |= (u64)(4 * max(b - k, 3)) << (8 * k);
  return r;
}
__device__ __forceinline__ int choose_base(const u32x4 (&V)[4]) {
  u32 sum = 0;
#pragma unroll
  for (int t = 0; t < 4; ++t) sum = __builtin_amdgcn_sad_u8(V[t][3 - t] & 0x7c7c7c7cu, 0u, sum);
  int b = (int)((wave_sum(sum) + 1638u) >> 12) - 2;
  b = b < 0 ? 0 : (b > 26 ? 26 : b);
  u32 cnts = 0;
#pragma unroll
  for (int k = 0; k < 3; ++k) {
    const u32 c0 = (u32)(0x80 - 4 * (b + k)) * 0x01010101u;
    u32 n = 0;
#pragma unroll
    for (int t = 0; t < 4; ++t) {
      const u32 q = (V[t][t] & 0x7f7f7f7fu) + c0;
      n += __builtin_popcount(q & ~(q - 0x0c0c0c0cu) & 0x80808080u);
    }
    cnts |= min(n, 15u) << (10 * k);
  }
  const u32 tot = wave_sum(cnts);
  const u32 n0 = tot & 1023, n1 = (tot >> 10) & 1023, n2 = tot >> 20;
  return b + ((n1 >= n0 && n1 >= n2) ? 1 : (n0 >= n2 ? 0 : 2));
}

__device__ __forceinline__ u32 encode_b(const u32x4 (&V)[4], int b, u32x4 (&N)[2], LAS unsigned char* scr, LAS unsigned char* lut, int lane) {
  const u32 vo = lane * 16;
  LAS unsigned char* tail = scr + 4096;
  *(LAS u32x4*)(tail + vo) = u32x4{0u, 0u, 0u, 0u};
  *(LAS u32x4*)(tail + 1024 + vo) = u32x4{0u, 0u, 0u, 0u};
  const u32 c1 = (u32)(0x83 + 4 * (b + 3)) * 0x01010101u;
  const u64 ex = tier2_exponents(b);
  const u32 exlo = (u32)ex, exhi = (u32)(ex >> 32);
  u32 M[2], S[2] = {0u, 0u}, o[4];
  u64 acc[4];
#pragma unroll
  for (int t = 0; t < 4; ++t) {
    u32x4 X;
    u32 n1[4], k[4], e1[4];
#pragma unroll
    for (int e = 0; e < 4; ++e) {
      const u32 v = V[t][e], m = v & 0x7f7f7f7fu;
      const u32 q = c1 - m;
      const u32 F = ((((q ^ 0x80808080u) & 0xf0f0f0f0u) >> 1) + 0x78787878u) & 0x80808080u;
      const u32 f4 = F >> 4, f5 = F >> 5;
      n1[e] = ~q | f4 | f5;
      e1[e] = (n1[e] >> 3) & (n1[e] >> 2) & 0x01010101u;
      k[e] = ((q - 0x0c0c0c0cu) >> 2) & (f4 - (F >> 7));
      X[e] = (__builtin_amdgcn_perm(exhi, exlo, k[e]) ^ m) & (F - f5);
      S[t >> 1] |= (v >> (4 * (t & 1) + e)) & (0x80808080u >> (4 * (t & 1) + e));
    }
    *(LAS u32x4*)(scr + t * 1024 + vo) = X;
    const u32 m16 = gather16(X);
    if (t & 1) M[t >> 1] |= m16 << 16; else M[t >> 1] = m16;
    N[t >> 1][2 * (t & 1)] = (n1[0] & 0x0f0f0f0fu) | ((n1[1] << 4) & 0xf0f0f0f0u);
    N[t >> 1][2 * (t & 1) + 1] = (n1[2] & 0x0f0f0f0fu) | ((n1[3] << 4) & 0xf0f0f0f0u);
    u32 ob = 0;
    u64 a = 0;
#pragma unroll
    for (int e = 0; e < 4; ++e) {
      const i32x2 wm = *(LAS i32x2*)(lut + __builtin_amdgcn_udot4(e1[e], 0x40201008u, 0u, false));
      const u32 c12 = __builtin_amdgcn_udot4(k[e], (u32)wm.x, (k[e] >> 15) & (u32)wm.y, false);
      a |= (u64)c12 << ob;
      ob = __builtin_amdgcn_udot4(e1[e], 0x03030303u, ob, false);
    }
    acc[t] = a; o[t] = ob;
  }
  const u32 nb = o[0] + o[1] + o[2] + o[3];
  const u32 nr = __builtin_popcount(M[0]) | (__builtin_popcount(M[1]) << 16);
  const u32 ib = wave_scan(nb), ir = wave_scan(nr);
  const u32 bits = __builtin_amdgcn_readlane(ib, 63), totr = __builtin_amdgcn_readlane(ir, 63);
  const u32 nhalf = totr & 0xffffu, nexc = nhalf + (totr >> 16);
  const u32 rec0 = B_STR + ((bits + 31) >> 5) * 4;
  const u32 len = rec0 + 2 * nexc;
  if (bits > 3 * 2040 || len > 4032 || nexc > MAXEXC) return 0;
  *(LAS i32x2*)(tail + lane * 8) = i32x2{(int)S[0], (int)S[1]};
  if (lane == 0) *(LAS u32x4*)(tail + B_HDR - 2048) = u32x4{nhalf | (nexc << 10) | ((u32)b << 20), bits | (rec0 << 16), len, 0u};
  u32 bp = ib - nb + (B_STR - 2048) * 8;
#pragma unroll
  for (int t = 0; t < 4; ++t) {
    LAS u32* w = (LAS u32*)(tail + ((bp >> 5) << 2));
    const u32 sh = bp & 31, hi = (u32)(acc[t] >> 32);
    const u64 x = acc[t] << sh;
    __hip_atomic_fetch_or(w, (u32)x, __ATOMIC_RELAXED, __HIP_MEMORY_SCOPE_WORKGROUP);
    __hip_atomic_fetch_or(w + 1, (u32)(x >> 32), __ATOMIC_RELAXED, __HIP_MEMORY_SCOPE_WORKGROUP);
    __hip_atomic_fetch_or(w + 2, (hi >> 1) >> (31 - sh), __ATOMIC_RELAXED, __HIP_MEMORY_SCOPE_WORKGROUP);
    bp += o[t];
  }
  write_records<1>(M, rec0 - 2048 + 2 * ((ir - nr) & 0xffffu), rec0 - 2048 + 2 * (nhalf + ((ir - nr) >> 16)), scr, tail, 2048 + 2 * lane, vo);
  return len;
}

__device__ __forceinline__ void unpack_b(const unsigned char* src, LAS unsigned char* dst, LAS unsigned char* lut, int lane) {
  LAS u32x4* o = (LAS u32x4*)(dst + lane * 16);
  u32x4 N[2] = {o[0], o[64]};
  i32x2 S = *(LAS i32x2*)(dst + 2048 + lane * 8);
  i32x2 H = *(LAS i32x2*)(dst + B_HDR);
  const u32 so = B_STR + ((6 * lane) & ~3);
  u32 c0 = *(LAS u32*)(dst + so), c1 = *(LAS u32*)(dst + so + 4);
  asm volatile("" : "+v"(N[0]), "+v"(N[1]), "+v"(S), "+v"(H), "+v"(c0), "+v"(c1));
  const u32 hdr = __builtin_amdgcn_readfirstlane(H.x), h1 = __builtin_amdgcn_readfirstlane(H.y);
  if (hdr >> 31) {
    u32x4 D3, D4;
    asm volatile("global_load_dwordx4 %0, %2, %3 offset:3072 sc0 sc1\n\tglobal_load_dwordx4 %1, %4, %3 sc0 sc1\n\ts_waitcnt vmcnt(0)"
                 : "=&v"(D3), "=&v"(D4) : "v"(lane * 16), "s"(src), "v"(4096u) : "memory");
    o[192] = D3;
    *(LAS u32x4*)(dst + B_HDR) = D4;
    return;
  }
  const int b = (hdr >> 20) & 31;
  const u32 nhalf = hdr & 1023, nexc = (hdr >> 10) & 1023, bits = h1 & 0xffffu, rec0 = h1 >> 16;
  if (rec0 + 2 * nexc > 3072) {
    u32x4 D3;
    asm volatile("global_load_dwordx4 %0, %1, %2 offset:3072 sc0 sc1\n\ts_waitcnt vmcnt(0)" : "=&v"(D3) : "v"(lane * 16), "s"(src) : "memory");
    o[192] = D3;
  }
  u32 ev = *(LAS u32*)(dst + min(rec0 + lane * 4, 4092u));
  u32 c2 = 0, c3 = 0;
  if (bits > 3072) { c2 = *(LAS u32*)(dst + so + 384); c3 = *(LAS u32*)(dst + so + 388); }
  const u64 ex = tier2_exponents(b) - 0x0c0c0c0c0c0c0c0cull;
  const u32 exlo = (u32)ex, exhi = (u32)(ex >> 32);
  auto expand = [&](u32 a0, u32 a1, int p) {
    const u32 sh = (lane & 1) * 16;
    const u32 x = __builtin_amdgcn_alignbit(a1, a0, sh), y = a1 >> sh;
    const u32 g[4] = {x & 0xfffu, (x >> 12) & 0xfffu, __builtin_amdgcn_alignbit(y, x, 24) & 0xfffu, (y >> 4) & 0xfffu};
    u32x4 r;
#pragma unroll
    for (int i = 0; i < 4; ++i) {
      const u32 z = (g[i] | (g[i] << 10)) & 0x003f003fu;
      r[i] = __builtin_amdgcn_perm(exhi, exlo, (z | (z << 5)) & 0x07070707u);
    }
    o[p * 64] = r;
    o[128 + p * 64] = u32x4{r[1], r[2], r[3], (u32)__builtin_amdgcn_update_dpp(0u, r[0], 0x130, 0xf, 0xf, true)};
    if (p && lane == 0) *(LAS u32*)(dst + 2044 + 1024) = r[0];
  };
  expand(c0, c1, 0);
  if (bits > 3072) expand(c2, c3, 1);
  u32 G[8], n = 0;
#pragma unroll
  for (int i = 0; i < 8; ++i) {
    const u32 x = N[i >> 2][i & 3];
    G[i] = x & (x >> 1) & 0x44444444u;
    n += __builtin_popcount(G[i]);
  }
  u32 wa = (u32)(size_t)dst + wave_scan(n) - n;
  const u32 b4 = (u32)(4 * b) * 0x01010101u;
  u32 wad[16], sel[16];
  i32x2 W[16];
#pragma unroll
  for (int j = 0; j < 16; ++j) {
    const u32 f = ((j & 1) ? G[j >> 1] >> 4 : G[j >> 1]) & 0x04040404u;
    sel[j] = *(LAS u32*)(lut + __builtin_amdgcn_udot4(f, 0x08040201u, 0u, false));
    W[j] = *(LAS i32x2*)__builtin_assume_aligned((LAS void*)(size_t)((wa + (wa & 4u) * 511u) & ~3u), 8);
    wad[j] = wa;
    wa += __builtin_popcount(f);
  }
  asm volatile("" : "+v"(W[0]), "+v"(W[1]), "+v"(W[2]), "+v"(W[3]), "+v"(W[4]), "+v"(W[5]), "+v"(W[6]), "+v"(W[7]),
               "+v"(sel[0]), "+v"(sel[1]), "+v"(sel[2]), "+v"(sel[3]), "+v"(sel[4]), "+v"(sel[5]), "+v"(sel[6]), "+v"(sel[7]));
  asm volatile("" : "+v"(W[8]), "+v"(W[9]), "+v"(W[10]), "+v"(W[11]), "+v"(W[12]), "+v"(W[13]), "+v"(W[14]), "+v"(W[15]),
               "+v"(sel[8]), "+v"(sel[9]), "+v"(sel[10]), "+v"(sel[11]), "+v"(sel[12]), "+v"(sel[13]), "+v"(sel[14]), "+v"(sel[15]));
#pragma unroll
  for (int t = 0; t < 4; ++t) {
    u32x4 O;
#pragma unroll
    for (int e = 0; e < 4; ++e) {
      const int j = 4 * t + e;
      const u32 x = N[j >> 3][(j >> 1) & 3];
      const u32 nib = ((j & 1) ? x >> 4 : x) & 0x0f0f0f0fu;
      const u32 sg = ((u32)S[t >> 1] << (j & 7)) & 0x80808080u;
      O[e] = (nib | sg) + __builtin_amdgcn_perm(b4, __builtin_amdgcn_alignbyte((u32)W[j].y, (u32)W[j].x, wad[j]), sel[j]);
    }
    o[t * 64] = O;
  }
  for (u32 base = 0;;) {
    apply_records(ev, base, nexc, nhalf, dst, lane);
    base += 128;
    if (base >= nexc) break;
    asm volatile("global_load_dword %0, %1, %2 sc0 sc1\n\ts_waitcnt vmcnt(0)" : "=&v"(ev) : "v"(rec0 + lane * 4 + 2 * base), "s"(src) : "memory");
  }
}

__device__ __forceinline__ void init_luts(LAS unsigned char* lut, int lane) {
  if (lane < 16) {
    u32 w = 0, sel = 0, r = 0;
#pragma unroll
    for (int i = 0; i < 4; ++i) {
      const bool on = (lane >> i) & 1;
      w |= (on && r < 3 ? 1u << (3 * r) : 0u) << (8 * i);
      sel |= (on ? r : 4u) << (8 * i);
      r += on;
    }
    *(LAS i32x2*)(lut + lane * 8) = i32x2{(int)w, lane == 15 ? 0xe00 : 0};
    *(LAS u32*)(lut + 128 + lane * 4) = sel;
  }
}

__global__ void __launch_bounds__(NTHREADS, 3) __attribute__((amdgpu_num_vgpr(128))) fused_kernel(Params P) {
  extern __shared__ __attribute__((aligned(16))) unsigned char lds_raw[];
  LAS unsigned char* L = (LAS unsigned char*)lds_raw;
  volatile LAS u32* READY = (volatile LAS u32*)(L + LDS_READY);
  volatile LAS u32* DONE = (volatile LAS u32*)(L + LDS_DONE);
  volatile LAS u32* BAD = (volatile LAS u32*)(L + LDS_BAD);
  volatile LAS u32* GO = (volatile LAS u32*)(L + LDS_GO);

  const int c = blockIdx.x;
  const int lane = threadIdx.x & 63;
  const int wave = __builtin_amdgcn_readfirstlane(threadIdx.x >> 6);
  const int u = (c & 7) * 8 + ((c >> 3) & 7);
  const int mt = c >> 6;
  unsigned char* const ws = P.peer[0];
  STAMP(0);

  unsigned char* cntp = ws + WS_CNT + c * 128 + wave * 4;
#define START_BARRIER() do { __builtin_amdgcn_sched_barrier(0); __builtin_amdgcn_s_barrier(); __builtin_amdgcn_sched_barrier(0); } while (0)
  const u32 w_lane = (u32)(lane >> 1) * NN + (lane & 1) * 16;
  const u32 vo = lane * 16;

  if (wave == NWAVES - 1) {
    __builtin_amdgcn_s_waitcnt(0x0F70);
    if (c < 224) {
      const int d = 1 + c % 7, j = c / 7;
      const unsigned char* xs = P.x + (size_t)(((P.rank + d) & 7) * MS + (j >> 1) * 32) * KS + (j & 1) * 256;
      const u32 xo = (u32)(lane & 31) * KS + 32 * (lane >> 5);
      const u32 wl2 = w_lane + 32 * NN;
#define ACLOB "v128", "v129", "v130", "v131", "v132", "v133", "v134", "v135", "v136", "v137", "v138", "v139", "v140", "v141", "v142", "v143", "v144", "v145", "v146", "v147", \
              "v148", "v149", "v150", "v151", "v152", "v153", "v154", "v155", "v156", "v157", "v158", "v159", "v160", "v161", "memory"
#define LOADW(t) do { const unsigned char* s_ = P.w + (size_t)((t) >> 6) * 64 * NN + UTILE(t) * 64; \
        asm volatile("global_load_dwordx4 v[128:131], %0, %2\n\tglobal_load_dwordx4 v[132:135], %1, %2\n\t" \
                     "global_load_dwordx4 v[136:139], %0, %2 offset:32\n\tglobal_load_dwordx4 v[140:143], %1, %2 offset:32" \
                     :: "v"(w_lane), "v"(wl2), "s"(s_) : ACLOB); } while (0)
      asm volatile("global_load_dword v160, %0, %1 sc1\n\t"
                   "global_load_dwordx4 v[128:131], %2, %3\n\tglobal_load_dwordx4 v[132:135], %2, %3 offset:16\n\t"
                   "global_load_dwordx4 v[136:139], %2, %3 offset:64\n\tglobal_load_dwordx4 v[140:143], %2, %3 offset:80\n\t"
                   "global_load_dwordx4 v[144:147], %2, %3 offset:128\n\tglobal_load_dwordx4 v[148:151], %2, %3 offset:144\n\t"
                   "global_load_dwordx4 v[152:155], %2, %3 offset:192\n\tglobal_load_dwordx4 v[156:159], %2, %3 offset:208\n\t"
                   "v_mov_b32 v161, 1"
                   :: "v"(0u), "s"(cntp), "v"(xo), "s"(xs) : ACLOB);
      START_BARRIER();
      __builtin_amdgcn_s_setprio(2);
      u32 cnt;
      asm volatile("s_waitcnt vmcnt(8)\n\tv_mov_b32 %0, v160" : "=v"(cnt) :: "memory");
      const u32 call = __builtin_amdgcn_readfirstlane(cnt) + 1;
      unsigned char* pd = P.peer[d];
      unsigned char* dst = pd + WS_XB + (call & 1) * XB_PAR + (size_t)(d * 32 + j) * 8192;
      asm volatile("s_waitcnt vmcnt(4)\n\t"
                   "global_store_dwordx4 %0, v[128:131], %1 sc0 sc1\n\tglobal_store_dwordx4 %0, v[132:135], %1 offset:1024 sc0 sc1\n\t"
                   "global_store_dwordx4 %0, v[136:139], %1 offset:2048 sc0 sc1\n\tglobal_store_dwordx4 %0, v[140:143], %1 offset:3072 sc0 sc1\n\ts_nop 1"
                   :: "v"(vo), "s"(dst) : "memory");
      LOADW(j);
      dst += 4096;
      VMWAIT(8);
      if (lane == 0) ag_store(cntp, call);
      asm volatile("global_store_dwordx4 %0, v[144:147], %1 sc0 sc1\n\tglobal_store_dwordx4 %0, v[148:151], %1 offset:1024 sc0 sc1\n\t"
                   "global_store_dwordx4 %0, v[152:155], %1 offset:2048 sc0 sc1\n\tglobal_store_dwordx4 %0, v[156:159], %1 offset:3072 sc0 sc1\n\ts_nop 1"
                   :: "v"(vo), "s"(dst) : "memory");
      GO[0] = 1u;
      STAMP(1);
      unsigned char* const wdst = pd + WS_WB + (call & 1) * WB_PAR + d * WB_SRC;
      unsigned char* const fw = pd + WS_FW + d * 64 * 512;
      unsigned char* flag = pd + WS_FX + (d * 32 + j) * 64;
      const unsigned char* tkp = ws + WS_TK + d * 128;
      const u32 tbase = (call - 1) * DRAWS - 64u;
      LAS unsigned char* scr = L + LDS_ENC;
      int lo = 0;
      u32 nexc = 1000, cur = j;
      asm volatile("v_mov_b32 v160, %0" :: "v"(tbase + 32 + j) : ACLOB);
      u32 p1 = 0;
      u32 staged = U0;
#pragma unroll 1
      while (true) {
        u32x4 D[4], T, V[4];
        u32 tk;
        {
          u32 v0, v1, v2, v3, v4, v5, v6, v7, v8, v9, v10, v11, v12, v13, v14, v15;
          if (FMT_B(p1)) VMWAIT(4); else VMWAIT(5);
          asm volatile("v_mov_b32 %16, v160\n\t"
                       "v_mov_b32 %0, v128\n\tv_mov_b32 %1, v129\n\tv_mov_b32 %2, v130\n\tv_mov_b32 %3, v131\n\t"
                       "v_mov_b32 %4, v132\n\tv_mov_b32 %5, v133\n\tv_mov_b32 %6, v134\n\tv_mov_b32 %7, v135\n\t"
                       "v_mov_b32 %8, v136\n\tv_mov_b32 %9, v137\n\tv_mov_b32 %10, v138\n\tv_mov_b32 %11, v139\n\t"
                       "v_mov_b32 %12, v140\n\tv_mov_b32 %13, v141\n\tv_mov_b32 %14, v142\n\tv_mov_b32 %15, v143"
                       : "=v"(v0), "=v"(v1), "=v"(v2), "=v"(v3), "=v"(v4), "=v"(v5), "=v"(v6), "=v"(v7), "=v"(v8), "=v"(v9), "=v"(v10),
                         "=v"(v11), "=v"(v12), "=v"(v13), "=v"(v14), "=v"(v15), "=v"(tk) :: "memory");
          V[0] = u32x4{v0, v1, v2, v3}; V[1] = u32x4{v4, v5, v6, v7}; V[2] = u32x4{v8, v9, v10, v11}; V[3] = u32x4{v12, v13, v14, v15};
        }
        const u32 nxt = __builtin_amdgcn_readfirstlane(tk) - tbase;
        const u32 nl = nxt < 512u ? nxt : cur;
        while (nl >= staged) {
          staged = __builtin_amdgcn_readfirstlane(__hip_atomic_load((LAS u32*)(L + LDS_STG), __ATOMIC_ACQUIRE, __HIP_MEMORY_SCOPE_WORKGROUP));
          if (nl >= staged) __builtin_amdgcn_s_sleep(2);
        }
        if (j < NPW) {
          if (lane == 0) asm volatile("global_atomic_add v160, %0, v161, %1 sc0 sc1" :: "v"(0u), "s"(tkp) : ACLOB);
        } else {
          asm volatile("global_load_dword v144, %0, %1 sc1\n\tv_mov_b32 v160, %2" :: "v"(0u), "s"(tkp), "v"(tbase + 512u) : ACLOB);
        }
        if (nl < U0) {
          LOADW(nl);
        } else {
          const unsigned char* s_ = ws + WS_ST + (size_t)nl * SLOT;
          asm volatile("global_load_dwordx4 v[128:131], %0, %1 sc1\n\tglobal_load_dwordx4 v[132:135], %0, %1 offset:1024 sc1\n\t"
                       "global_load_dwordx4 v[136:139], %0, %1 offset:2048 sc1\n\tglobal_load_dwordx4 v[140:143], %0, %1 offset:3072 sc1"
                       :: "v"(vo), "s"(s_) : ACLOB);
        }
        u32 len;
        if (cur < U0) {
          if (nexc > 96) lo = choose_window(V);
          nexc = encode_piece(V, lo, D, T, scr, lane);
          len = nexc > MAXEXC ? 5120u : 3076u + 2 * nexc;
        } else {
          D[0] = V[0]; D[1] = V[1]; D[2] = V[2]; D[3] = V[3];
          const u32 ha = __builtin_amdgcn_readfirstlane(V[3].x);
          len = cur < U1 ? __builtin_amdgcn_readlane(V[2].z, 32) : ((ha >> 31) ? 5120u : 3076u + 2 * ((ha >> 10) & 1023));
          if (len > 4096u)
            asm volatile("global_load_dwordx4 %0, %1, %2 sc1\n\ts_waitcnt vmcnt(0)" : "=&v"(T) : "v"(vo), "s"(ws + WS_ST + (size_t)cur * SLOT + 4096) : "memory");
        }
        VMWAIT(5);
        if (lane < 16) flag_store(flag + lane * 4, call);
        dst = wdst + (size_t)((cur & ~63u) + UTILE(cur)) * SLOT;
        const u32 blk = vo & ~63u;
        ST16(dst, vo, D[0], 0); ST16(dst, vo, D[1], 1024);
        if (2048 + blk < len) ST16(dst, vo, D[2], 2048);
        if (3072 + blk < len) ST16(dst, vo, D[3], 3072);
        if (4096 + blk < len) { unsigned char* d2 = dst + 4096; ST16(d2, vo, T, 0); }
        p1 = cur;
        flag = fw + UTILE(cur) * 512 + (cur >> 6) * 64;
        if ((cur & 32) == 0) STAMP(8 + (cur >> 6));
        if (nxt >= 512u) break;
        cur = nxt;
      }
      VMWAIT(0);
      if (lane < 16) flag_store(flag + lane * 4, call);
      STAMP(3);
#undef LOADW
#undef ACLOB
    } else {
      START_BARRIER();
      GO[0] = 1u;
    }
    STAMP_OUT();
    return;
  }
  const u32 cnt_old = ag_load(cntp);
  if (wave == 5) asm volatile("ds_write_b32 %0, %1\n\ts_waitcnt lgkmcnt(0)" :: "v"((u32)(size_t)L + LDS_CTRL + lane * 4), "v"(0u) : "memory");
  if (wave == 6) { init_luts(L + LDS_LUT, lane); asm volatile("s_waitcnt lgkmcnt(0)" ::: "memory"); }
  u32 call;
  if (c >= 224) {
    START_BARRIER();
    call = __builtin_amdgcn_readfirstlane(cnt_old) + 1;
    LAS unsigned char* scr = L + LDS_SCR + wave * 6400;
#pragma unroll 1
    for (u32 e = U0 + (c - 224) + 32 * wave; e < 512u; e += 32 * 11) {
      const unsigned char* src = P.w + (size_t)(e >> 6) * 64 * NN + UTILE(e) * 64 + w_lane;
      u32x4 V[4], D[4], T;
      V[0] = *(const GAS u32x4*)(src);
      V[1] = *(const GAS u32x4*)(src + 32 * NN);
      V[2] = *(const GAS u32x4*)(src + 32);
      V[3] = *(const GAS u32x4*)(src + 32 * NN + 32);
      u32 len;
      if (e >= U1) {
        const u32 nexc = encode_piece(V, choose_window(V), D, T, scr, lane);
        len = nexc > MAXEXC ? 5120u : 3076u + 2 * nexc;
      } else {
        u32x4 N[2];
        len = encode_b(V, choose_base(V), N, scr, L + LDS_LUT, lane);
        if (len) {
          D[0] = N[0]; D[1] = N[1];
          D[2] = *(LAS u32x4*)(scr + 4096 + vo); D[3] = *(LAS u32x4*)(scr + 5120 + vo);
        } else {
          len = 4112;
          D[0] = V[0]; D[1] = V[1]; D[3] = V[3];
          D[2] = lane == 32 ? u32x4{0x80000000u, 0u, 4112u, 0u} : V[2];
#pragma unroll
          for (int q = 0; q < 4; ++q) T[q] = __builtin_amdgcn_readlane(V[2][q], 32);
        }
      }
      unsigned char* dst = ws + WS_ST + (size_t)e * SLOT;
      ST16A(dst, vo, D[0], 0); ST16A(dst, vo, D[1], 1024); ST16A(dst, vo, D[2], 2048);
      if (3072 + (vo & ~63u) < len) ST16A(dst, vo, D[3], 3072);
      if (4096 + (vo & ~63u) < len) { unsigned char* d2 = dst + 4096; ST16A(d2, vo, T, 0); }
      if (e < U2) STAMP(6);
      VMWAIT(0);
      if (lane == 0) __hip_atomic_fetch_add((GAS u32*)(ws + WS_SC + (e >= U2 ? 1024 : 0) + (e & 7) * 128), 1u, __ATOMIC_RELAXED, __HIP_MEMORY_SCOPE_AGENT);
      STAMP(5);
    }
    if (lane == 0) __hip_atomic_fetch_add((LAS u32*)(L + LDS_ENCD), 1u, __ATOMIC_RELAXED, __HIP_MEMORY_SCOPE_WORKGROUP);
    while (*(volatile LAS u32*)(L + LDS_ENCD) < 11u) __builtin_amdgcn_s_sleep(2);
    asm volatile("" ::: "memory");
  } else {
    START_BARRIER();
    call = __builtin_amdgcn_readfirstlane(cnt_old) + 1;
  }
  if (lane == 0) ag_store(cntp, call);
  while (GO[0] == 0u) __builtin_amdgcn_s_sleep(4);

  if (wave < 4) {
    const int mymb = mt * 4 + wave;
    const u32 par = call & 1;
    const float scale = P.sx[0] * P.sw[0];
    const u32 b_lane = (u32)(size_t)L + ((lane >> 5) * 32 + ((lane & 15) >> 1)) * 32 + ((lane >> 4) & 1) * 16 + (lane & 1) * 8;
    f32x16 acc0 = {}, acc1 = {};
    u32 rv;
    u32x4 AL[8] = {}, AH[8] = {};
#define AWAIT() asm volatile("s_waitcnt vmcnt(0)" : "+v"(AL[0]), "+v"(AL[1]), "+v"(AL[2]), "+v"(AL[3]), "+v"(AL[4]), "+v"(AL[5]), "+v"(AL[6]), "+v"(AL[7]), \
                             "+v"(AH[0]), "+v"(AH[1]), "+v"(AH[2]), "+v"(AH[3]), "+v"(AH[4]), "+v"(AH[5]), "+v"(AH[6]), "+v"(AH[7]) :: "memory")
#define MMA(p, off) do { \
          i32x8 b0, b1; read_bfrag(b0, baddr, off); read_bfrag(b1, baddr, off + 2048); \
          const i32x8 a = {(int)AL[p].x, (int)AL[p].y, (int)AL[p].z, (int)AL[p].w, (int)AH[p].x, (int)AH[p].y, (int)AH[p].z, (int)AH[p].w}; \
          __builtin_amdgcn_sched_barrier(0); \
          acc0 = __builtin_amdgcn_mfma_scale_f32_32x32x64_f8f6f4(b0, a, acc0, 1, 0, 0, 0, 0, 0); \
          acc1 = __builtin_amdgcn_mfma_scale_f32_32x32x64_f8f6f4(b1, a, acc1, 1, 0, 0, 0, 0, 0); } while (0)
#define POLL(row, want) do { while (true) { \
          rv = READY[(row) * 8 + (lane & 7)]; \
          m = (u32)__ballot(rv >= (u32)(want)) & rem; \
          if (m) break; \
          __builtin_amdgcn_s_sleep(1); } \
        asm volatile("" ::: "memory"); rem &= ~m; } while (0)
    {
      const unsigned char* xs = P.x + (size_t)(P.rank * MS + mymb * 32) * KS;
      const u32 xo = (u32)(lane & 31) * KS + 32 * (lane >> 5);
#define LA(p) LD16(AL[p], xs, xo, p * 64); LD16(AH[p], xs, xo, p * 64 + 16);
      LA(0) LA(1) LA(2) LA(3) LA(4) LA(5) LA(6) LA(7)
#undef LA
      AWAIT();
      const u32 baddr = b_lane + LDS_OWN;
      u32 rem = 0xFFu, m;
      do {
        POLL(RR, 1);
#define STEP(p) if (m & (1u << p)) MMA(p, p * PIECE);
        STEP(0) STEP(1) STEP(2) STEP(3) STEP(4) STEP(5) STEP(6) STEP(7)
#undef STEP
      } while (rem);
    }
    STAMP(7);
    if (!wait_all<true>(ws + WS_FX + ((1 + (lane & 7) % 7) * 16 + mymb) * 128 + ((lane >> 3) & 1) * 64, call, ws, call)) BAD[0] = 1u;
    STAMP(2);
    const unsigned char* xb = ws + WS_XB + par * XB_PAR + (size_t)mymb * 16384;
#define LA(p) do { const unsigned char* s_ = xb + p * 262144; LD16S(AL[p], s_, vo, 0); LD16S(AH[p], s_, vo, 1024); } while (0)
    LA(1); LA(2); LA(3); LA(4); LA(5); LA(6); LA(7);
#pragma unroll 1
    for (int i = 0; i < 8; ++i) {
      const int row = i & (RR - 1);
      const u32 baddr = b_lane + row * 7 * PIECE;
      xb += 2048;
      AWAIT();
      u32 rem = 0xFEu, m;
      do {
        POLL(row, i + 1);
#define STEP(p) if (m & (1u << p)) { MMA(p, (p - 1) * PIECE); if (i < 7) LA(p); }
        STEP(1) STEP(2) STEP(3) STEP(4) STEP(5) STEP(6) STEP(7)
#undef STEP
      } while (rem);
      asm volatile("s_waitcnt lgkmcnt(0)" ::: "memory");
      if (lane == 0) DONE[wave] = (u32)(i + 1);
      STAMP(8 + i);
    }
#undef LA
    AWAIT();
    const bool bad = __builtin_amdgcn_readfirstlane(rv) != 0u;
    u32x4 o[8];
    const float sc = bad ? __builtin_nanf("") : scale;
    acc0 *= sc; acc1 *= sc;
#pragma unroll
    for (int q = 0; q < 4; ++q)
#pragma unroll
      for (int e = 0; e < 4; ++e) {
        o[q][e] = __builtin_bit_cast(u32, __builtin_elementwise_maximum(acc0[4 * q + e], 0.f));
        o[4 + q][e] = __builtin_bit_cast(u32, __builtin_elementwise_maximum(acc1[4 * q + e], 0.f));
      }
    const unsigned char* ob = (const unsigned char*)(P.out + (size_t)mymb * 32 * NN + u * 64);
#define OST(val, voff) asm volatile("global_store_dwordx4 %0, %1, %2 sc1\n\ts_nop 1" :: "v"(voff), "v"(val), "s"(ob) : "memory")
    {
      LAS unsigned char* tb = L + LDS_OWN + wave * 8192;
      const int m = lane & 31, h = lane >> 5;
#pragma unroll
      for (int q = 0; q < 8; ++q) *(LAS u32x4*)(tb + m * 256 + (((2 * q + h) ^ (m & 15)) << 4)) = o[q];
      const int r0 = lane >> 4, ch = lane & 15;
#pragma unroll
      for (int t = 0; t < 8; ++t) o[t] = *(LAS u32x4*)(tb + (4 * t + r0) * 256 + ((ch ^ ((4 * t + r0) & 15)) << 4));
      u32 oo = (u32)r0 * NN * 4 + ch * 16;
#pragma unroll
      for (int t = 0; t < 8; ++t) { OST(o[t], oo); oo += 4 * NN * 4; }
    }
#undef OST
    STAMP(3);
  } else {
    const int d = wave - 3;
    {
      const unsigned char* src = P.w + u * 64 + w_lane;
      for (int i = d - 1; i < 8; i += 7) {
        const unsigned char* s = src + (size_t)i * 64 * NN;
        LAS unsigned char* dst = L + LDS_OWN + i * PIECE;
        __builtin_amdgcn_global_load_lds((const GAS u32*)(s), (LAS u32*)(dst), 16, 0, 0);
        __builtin_amdgcn_global_load_lds((const GAS u32*)(s + 32 * NN), (LAS u32*)(dst + 1024), 16, 0, 0);
        __builtin_amdgcn_global_load_lds((const GAS u32*)(s + 32), (LAS u32*)(dst + 2048), 16, 0, 0);
        __builtin_amdgcn_global_load_lds((const GAS u32*)(s + 32 * NN + 32), (LAS u32*)(dst + 3072), 16, 0, 0);
      }
    }
    VMWAIT(0);
    if (lane == 0) { READY[RR * 8 + d - 1] = 1u; if (d == 1) READY[RR * 8 + 7] = 1u; }
    if (d == 1 && c < 7 * NPW) {
      (void)wait_all<false>(ws + WS_SC + (lane & 7) * 128, call * 44u, ws, call);
      __hip_atomic_store((LAS u32*)(L + LDS_STG), U2, __ATOMIC_RELEASE, __HIP_MEMORY_SCOPE_WORKGROUP);
      (void)wait_all<false>(ws + WS_SC + 1024 + (lane & 7) * 128, call * 12u, ws, call);
      __hip_atomic_store((LAS u32*)(L + LDS_STG), 512u, __ATOMIC_RELEASE, __HIP_MEMORY_SCOPE_WORKGROUP);
    }
    const unsigned char* fp = ws + WS_FW + (d * 64 + u) * 512;
    const unsigned char* src = ws + WS_WB + (call & 1) * WB_PAR + d * WB_SRC + (size_t)u * SLOT;
    u32 seen = 0;
    bool dead = false, pre = false;
    const unsigned char* tmop = ws + WS_TMO;
    const u32 l4 = lane * 4;
#define FMT_A(i_) (!FMT_B((i_) * 64 + UTILE(u)))
#define FETCH(i_, s_) do { u32 keep_; const u32 la_ = (u32)(size_t)L + ((((i_) & (RR - 1)) * 7 + d - 1) * PIECE); \
      asm volatile("s_mov_b32 %0, m0\n\ts_mov_b32 m0, %3\n\ts_nop 0\n\t" \
                   "global_load_lds_dwordx4 %1, %2 sc0 sc1\n\t" \
                   "global_load_lds_dwordx4 %1, %2 offset:1024 sc0 sc1\n\t" \
                   "global_load_lds_dwordx4 %1, %2 offset:2048 sc0 sc1\n\t" \
                   "s_mov_b32 m0, %0" : "=&s"(keep_) : "v"(vo), "s"(s_), "s"(la_) : "memory"); \
      if (FMT_A(i_)) \
        asm volatile("s_mov_b32 %0, m0\n\ts_mov_b32 m0, %3\n\ts_nop 0\n\tglobal_load_lds_dword %1, %2 offset:3072 sc0 sc1\n\ts_mov_b32 m0, %0" \
                     : "=&s"(keep_) : "v"(l4), "s"(s_), "s"(la_) : "memory"); } while (0)
#pragma unroll 1
    for (int i = 0; i < 8; ++i, src += 64 * SLOT) {
      if (!pre) {
        if (i >= RR) {
          while (!__all(DONE[lane & 3] + RR > (u32)i)) __builtin_amdgcn_s_sleep(2);
          asm volatile("" ::: "memory");
        }
        if (!((seen >> i) & 1) && !dead) {
          const u64 t0 = rt();
          for (u32 spin = 1;; ++spin) {
            seen = (u32)__ballot(flag_load(fp + max(lane & 7, i) * 64) >= call);
            if ((seen >> i) & 1) break;
            if ((spin & 63) == 0 && (__builtin_amdgcn_readfirstlane(ag_load(tmop)) == call || rt() - t0 > DEADLINE_TICKS)) { dead = true; break; }
            if (i < 7) __builtin_amdgcn_s_sleep(2);
          }
          if (dead) ag_store(ws + WS_TMO, call);
          asm volatile("" ::: "memory");
        }
        if (i == 7) STAMP(4);
        if (!dead) FETCH(i, src);
      }
      pre = !dead && i < 7 && ((seen >> (i + 1)) & 1) && __all(DONE[lane & 3] + RR > (u32)i + 1);
      if (pre) {
        const unsigned char* s2 = src + 64 * SLOT;
        FETCH(i + 1, s2);
        VMWAIT(3);
      } else {
        if (i == 7) dead = dead || __builtin_amdgcn_readfirstlane(ag_load(tmop)) == call;
        VMWAIT(0);
      }
      if (i == 7) STAMP(5);
      const int row = i & (RR - 1);
      if (dead) BAD[0] = 1u;
      else if (FMT_A(i)) unpack_piece(src, L + (row * 7 + d - 1) * PIECE, lane);
      else unpack_b(src, L + (row * 7 + d - 1) * PIECE, L + LDS_LUT + 128, lane);
      asm volatile("s_waitcnt lgkmcnt(0)" ::: "memory");
      READY[row * 8 + d] = (u32)(i + 1);
      STAMP(8 + i);
    }
#undef FETCH
#undef FMT_A
    STAMP(3);
  }
  STAMP_OUT();
}

}

extern "C" void kernel_launch(int rank, int nranks, ncclComm_t comm,
                              hipStream_t stream,
                              void* const* d_in, const int* in_sizes, int n_in,
                              void* d_out, int out_size,
                              const int* in_dtypes, int out_dtype,
                              void* const* d_out_all,
                              void* const* d_ws_all, size_t ws_bytes) {
  (void)comm; (void)in_dtypes; (void)out_dtype; (void)d_out_all;
  if (nranks != R || (unsigned)rank >= (unsigned)R || n_in != 4 || ws_bytes < WS_END ||
      in_sizes[0] != 4096 * KS || in_sizes[1] != KS * NN || out_size != MS * NN) {
    fprintf(stderr, "kernel_launch: unexpected problem shape\n");
    return;
  }
  HIP_CHECK(hipFuncSetAttribute((const void*)fused_kernel, hipFuncAttributeMaxDynamicSharedMemorySize, LDS_BYTES));
  Params P;
  P.x = (const unsigned char*)d_in[0];
  P.w = (const unsigned char*)d_in[1];
  P.sx = (const float*)d_in[2];
  P.sw = (const float*)d_in[3];
  P.out = (float*)d_out;
  for (int d = 0; d < R; ++d) P.peer[d] = (unsigned char*)d_ws_all[(rank + d) % R];
  P.rank = rank;
  P.pad = 0;
  hipLaunchKernelGGL(fused_kernel, dim3(NWG), dim3(NTHREADS), LDS_BYTES, stream, P);
  HIP_CHECK(hipGetLastError());
}
```

```cpp
#include <hip/hip_runtime.h>
#include <rccl/rccl.h>
#include <cstdio>

#define HIP_CHECK(expr) do { hipError_t _e = (expr); if (_e != hipSuccess) { \
    fprintf(stderr, "HIP error %s at %s:%d\n", hipGetErrorString(_e), __FILE__, __LINE__); \
    return; } } while (0)

#define TPOLL 400
#define XW 2
#define BEND 448
#define NPW 22

namespace {

typedef unsigned u32;
typedef unsigned long long u64;
typedef unsigned u32x4 __attribute__((ext_vector_type(4)));
typedef int i32x2 __attribute__((ext_vector_type(2)));
typedef int i32x8 __attribute__((ext_vector_type(8)));
typedef float f32x4 __attribute__((ext_vector_type(4)));
typedef float f32x16 __attribute__((ext_vector_type(16)));
#define LAS __attribute__((address_space(3)))
#define GAS __attribute__((address_space(1)))

constexpr int R = 8;
constexpr int MS = 512;
constexpr int KS = 512;
constexpr int NN = 4096;
constexpr int NWG = 256;
constexpr int NWAVES = 12;
constexpr int NTHREADS = NWAVES * 64;
constexpr int RR = 4;
constexpr int PIECE = 4096;
constexpr int SLOT = 5120;
constexpr int MAXEXC = 510;
constexpr int LDS_OWN = RR * 7 * PIECE;
constexpr int LDS_CTRL = LDS_OWN + 8 * PIECE;
constexpr int LDS_READY = LDS_CTRL;
constexpr int LDS_DONE = LDS_CTRL + 192;
constexpr int LDS_BAD = LDS_READY + (7 & (RR - 1)) * 32;
constexpr int LDS_GO = LDS_CTRL + 232;
constexpr int LDS_ENCD = LDS_CTRL + 240;
constexpr int LDS_GRP = LDS_CTRL + 208;
constexpr int LDS_STG = LDS_CTRL + 248;
constexpr int LDS_LUT = LDS_CTRL + 256;
constexpr int LDS_SCR = 7 * PIECE;
constexpr int LDS_BYTES = LDS_CTRL + 512;
static_assert(LDS_BYTES <= 160 * 1024, "LDS");

constexpr size_t WS_CNT = 0;
constexpr size_t WS_TMO = 32768;
constexpr size_t WS_TK = 36864;
constexpr size_t WS_FX = 65536;
constexpr size_t WS_FW = 131072;
constexpr u32 U1 = BEND;
constexpr size_t WS_FS = 57344;
constexpr size_t WS_DBG = 524288;
constexpr size_t WS_XB = 1ull << 20;
constexpr size_t XB_PAR = 2ull << 20;
constexpr size_t WB_SRC = (size_t)512 * SLOT;
constexpr size_t WS_WB = 8ull << 20;
constexpr size_t WB_PAR = 8 * WB_SRC;
constexpr size_t WS_ST = WS_WB + 2 * WB_PAR;
constexpr size_t WS_END = WS_ST + 2 * WB_SRC;

constexpr u64 DEADLINE_TICKS = 40000000ull;

struct Params {
  const unsigned char* x;
  const unsigned char* w;
  const float* sx;
  const float* sw;
  float* out;
  unsigned char* peer[R];
  int rank;
  int pad;
};

#define UTILE(t) ((((t) & 7) << 3) | (((t) >> 3) & 7))
#define FMT_B(t) ((u32)(t) < U1)

__device__ __forceinline__ u64 rt() { return __builtin_amdgcn_s_memrealtime(); }

#define STAMP(slot) do {} while (0)
#define STAMP_OUT() do {} while (0)

#define ST16(base, voff, val, imm) \
  asm volatile("global_store_dwordx4 %0, %1, %2 offset:%3 sc0 sc1\n\ts_nop 1" :: "v"(voff), "v"(val), "s"(base), "i"(imm) : "memory")
#define ST16A(base, voff, val, imm) \
  asm volatile("global_store_dwordx4 %0, %1, %2 offset:%3 sc1\n\ts_nop 1" :: "v"(voff), "v"(val), "s"(base), "i"(imm) : "memory")
#define LD16S(val, base, voff, imm) \
  asm volatile("global_load_dwordx4 %0, %1, %2 offset:%3 sc0 sc1" : "+v"(val) : "v"(voff), "s"(base), "i"(imm) : "memory")
#define VMWAIT(n) asm volatile("s_waitcnt vmcnt(%0)" :: "i"(n) : "memory")

__device__ __forceinline__ void flag_store(unsigned char* p, u32 v) {
  __hip_atomic_store((GAS u32*)p, v, __ATOMIC_RELAXED, __HIP_MEMORY_SCOPE_SYSTEM);
}
__device__ __forceinline__ u32 flag_load(const unsigned char* p) {
  return __hip_atomic_load((GAS u32*)p, __ATOMIC_RELAXED, __HIP_MEMORY_SCOPE_SYSTEM);
}
__device__ __forceinline__ u32 ag_load(const unsigned char* p) {
  return __hip_atomic_load((GAS u32*)p, __ATOMIC_RELAXED, __HIP_MEMORY_SCOPE_AGENT);
}
__device__ __forceinline__ void ag_store(unsigned char* p, u32 v) {
  __hip_atomic_store((GAS u32*)p, v, __ATOMIC_RELAXED, __HIP_MEMORY_SCOPE_AGENT);
}

template <bool SYS, int NAP = 4>
__device__ __forceinline__ bool wait_all(const unsigned char* fp, u32 want, unsigned char* ws, u32 call) {
  const u64 t0 = rt();
  for (u32 spin = 1;; ++spin) {
    const u32 v = SYS ? flag_load(fp) : ag_load(fp);
    if (__all(v >= want)) break;
    if ((spin & 15) == 0 && (__builtin_amdgcn_readfirstlane(ag_load(ws + WS_TMO)) == call || rt() - t0 > DEADLINE_TICKS)) { ag_store(ws + WS_TMO, call); return false; }
    __builtin_amdgcn_s_sleep(NAP);
  }
  asm volatile("" ::: "memory");
  return true;
}

__device__ __forceinline__ u32 wave_sum(u32 v) {
  v += __builtin_amdgcn_update_dpp(0u, v, 0x111, 0xf, 0xf, true);
  v += __builtin_amdgcn_update_dpp(0u, v, 0x112, 0xf, 0xf, true);
  v += __builtin_amdgcn_update_dpp(0u, v, 0x114, 0xf, 0xf, true);
  v += __builtin_amdgcn_update_dpp(0u, v, 0x118, 0xf, 0xf, true);
  return __builtin_amdgcn_readlane(v, 15) + __builtin_amdgcn_readlane(v, 31) +
         __builtin_amdgcn_readlane(v, 47) + __builtin_amdgcn_readlane(v, 63);
}

__device__ __forceinline__ void read_bfrag(i32x8& r, u32 addr, int imm) {
  LAS unsigned char* b = (LAS unsigned char*)(size_t)addr;
  i32x2 r0 = __builtin_amdgcn_ds_read_tr8_b64_v2i32((LAS i32x2*)(b + imm));
  i32x2 r1 = __builtin_amdgcn_ds_read_tr8_b64_v2i32((LAS i32x2*)(b + imm + 256));
  i32x2 r2 = __builtin_amdgcn_ds_read_tr8_b64_v2i32((LAS i32x2*)(b + imm + 512));
  i32x2 r3 = __builtin_amdgcn_ds_read_tr8_b64_v2i32((LAS i32x2*)(b + imm + 768));
  r[0] = r0.x; r[1] = r0.y; r[2] = r1.x; r[3] = r1.y; r[4] = r2.x; r[5] = r2.y; r[6] = r3.x; r[7] = r3.y;
}

__device__ __forceinline__ u32 miss_pattern(u32 m  , u32 cneg, u32 lo4, u32& code) {
  code = (m + cneg) & 0x1f1f1f1fu;
  return (code + lo4) ^ m;
}

__device__ __forceinline__ u32 wave_scan(u32 v) {
  v += __builtin_amdgcn_update_dpp(0u, v, 0x111, 0xf, 0xf, true);
  v += __builtin_amdgcn_update_dpp(0u, v, 0x112, 0xf, 0xf, true);
  v += __builtin_amdgcn_update_dpp(0u, v, 0x114, 0xf, 0xf, true);
  v += __builtin_amdgcn_update_dpp(0u, v, 0x118, 0xf, 0xf, true);
  v += __builtin_amdgcn_update_dpp(0u, v, 0x142, 0xa, 0xf, false);
  v += __builtin_amdgcn_update_dpp(0u, v, 0x143, 0xc, 0xf, false);
  return v;
}

__device__ __forceinline__ u32 gather16(const u32x4& X) {
  u32 f[4];
#pragma unroll
  for (int e = 0; e < 4; ++e) f[e] = (X[e] + 0x7c7c7c7cu) & 0x80808080u;
  const u32 b0 = __builtin_amdgcn_udot4(f[1], 0x80402010u, __builtin_amdgcn_udot4(f[0], 0x08040201u, 0u, false), false);
  const u32 b1 = __builtin_amdgcn_udot4(f[3], 0x80402010u, __builtin_amdgcn_udot4(f[2], 0x08040201u, 0u, false), false);
  return (b0 >> 7) | (b1 << 1);
}

template <int RECR>
__device__ __forceinline__ void write_records(u32 (&M)[2], u32 wa0, u32 wa1, LAS unsigned char* xs, LAS unsigned char* recs, u32 dummy, u32 vo) {
  u32 wa[2] = {wa0, wa1};
  do {
    u32 pos[2][RECR], x[2][RECR], ad[2][RECR];
#pragma unroll
    for (int r = 0; r < RECR; ++r)
#pragma unroll
      for (int h = 0; h < 2; ++h) {
        const u32 bit = __builtin_ctz(M[h] | 0x80000000u);
        pos[h][r] = vo + (bit & 15) + ((bit & 16) << 6);
        x[h][r] = xs[h * 2048 + pos[h][r]];
        ad[h][r] = M[h] ? wa[h] : dummy;
        wa[h] += 2;
        M[h] &= M[h] - 1;
      }
#pragma unroll
    for (int r = 0; r < RECR; ++r)
#pragma unroll
      for (int h = 0; h < 2; ++h)
        *(LAS unsigned short*)(recs + ad[h][r]) = (unsigned short)(pos[h][r] | (x[h][r] << 9));
  } while (__any((M[0] | M[1]) != 0));
}

__device__ __forceinline__ void apply_records(u32 ev, u32 first, u32 lim, u32 nhalf, LAS unsigned char* dst, int lane) {
#pragma unroll
  for (int h = 0; h < 2; ++h) {
    const u32 rec = h ? ev >> 16 : ev & 0xffffu;
    const u32 idx = first + 2 * lane + h;
    const u32 off = (rec & 0x7ffu) | (idx >= nhalf ? 0x800u : 0u);
    if (idx < lim)
      __hip_atomic_fetch_xor((LAS u32*)(dst + (off & ~3u)), ((rec >> 11) << 2) << (8 * (off & 3)), __ATOMIC_RELAXED, __HIP_MEMORY_SCOPE_WORKGROUP);
  }
}

__device__ __forceinline__ int choose_window(const u32x4 (&V)[4]) {
  u32 sum = 0;
#pragma unroll
  for (int t = 0; t < 4; ++t) sum = __builtin_amdgcn_sad_u8(V[t][3 - t] & 0x7c7c7c7cu, 0u, sum);
  int lo = (int)((wave_sum(sum) + 1638u) >> 12) - 6;
  lo = lo < 0 ? 0 : (lo > 22 ? 22 : lo);
  u32 cnts = 0;
#pragma unroll
  for (int k = 0; k < 3; ++k) {
    const u32 lo4 = (u32)(lo + k) * 0x04040404u, cneg = (u32)((-4 * (lo + k)) & 31) * 0x01010101u;
    u32 n = 0, code;
#pragma unroll
    for (int t = 0; t < 4; ++t)
      n += __builtin_popcount((miss_pattern(V[t][t] & 0x7f7f7f7fu, cneg, lo4, code) + 0x7c7c7c7cu) & 0x80808080u);
    cnts |= min(n, 15u) << (10 * k);
  }
  const u32 tot = wave_sum(cnts);
  const u32 n0 = tot & 1023, n1 = (tot >> 10) & 1023, n2 = tot >> 20;
  return lo + ((n1 <= n0 && n1 <= n2) ? 1 : (n0 <= n2 ? 0 : 2));
}

__device__ __forceinline__ u32 encode_piece(const u32x4 (&V)[4], int lo, u32x4 (&D)[4], u32x4& T, LAS unsigned char* scr, int lane) {
  const u32 lo4 = (u32)lo * 0x04040404u, cneg = (u32)((-4 * lo) & 31) * 0x01010101u;
  const u32 vo = lane * 16;
  u32x4 C[4];
  u32 M[2];
#pragma unroll
  for (int t = 0; t < 4; ++t) {
    u32x4 X;
#pragma unroll
    for (int e = 0; e < 4; ++e) {
      u32 code;
      X[e] = miss_pattern(V[t][e] & 0x7f7f7f7fu, cneg, lo4, code);
      C[t][e] = code | (V[t][e] & 0x80808080u);
    }
    *(LAS u32x4*)(scr + t * 1024 + vo) = X;
    const u32 m16 = gather16(X);
    if (t & 1) M[t >> 1] |= m16 << 16; else M[t >> 1] = m16;
  }
  const u32 n = __builtin_popcount(M[0]) | (__builtin_popcount(M[1]) << 16);
  const u32 inc = wave_scan(n);
  const u32 tot = __builtin_amdgcn_readlane(inc, 63);
  const u32 nhalf = tot & 0xffffu, nexc = nhalf + (tot >> 16);
  LAS unsigned char* recs = scr + 4096;
  if (nexc > MAXEXC) {
    D[0] = V[0]; D[1] = V[1]; D[2] = V[2]; T = V[3];
    D[3] = u32x4{0x80000000u, 0u, 0u, 0u};
    return nexc;
  }
  if (lane == 0) *(LAS u32*)recs = nhalf | (nexc << 10) | ((u32)lo << 20);
  write_records<3>(M, 4 + 2 * ((inc - n) & 0xffffu), 4 + 2 * (nhalf + ((inc - n) >> 16)), scr, recs, 1024 + 2 * lane, vo);
#pragma unroll
  for (int e = 0; e < 4; ++e) {
    const u32 c3 = C[3][e];
    D[0][e] = C[0][e] | ((c3 << 5) & 0x60606060u);
    D[1][e] = C[1][e] | ((c3 << 3) & 0x60606060u);
    D[2][e] = C[2][e] | ((c3 << 1) & 0x20202020u) | ((c3 >> 1) & 0x40404040u);
  }
  D[3] = *(LAS u32x4*)(recs + vo);
  return nexc;
}

__device__ __forceinline__ void unpack_piece(const unsigned char* src, LAS unsigned char* dst, int lane) {
  LAS u32x4* o = (LAS u32x4*)(dst + lane * 16);
  u32 ev = *(LAS u32*)(dst + 3072 + lane * 4);
  u32x4 D[3] = {o[0], o[64], o[128]};
  asm volatile("" : "+v"(ev), "+v"(D[0]), "+v"(D[1]), "+v"(D[2]));
  const u32 hdr = __builtin_amdgcn_readfirstlane(ev);
  if (hdr >> 31) {
    u32x4 D3;
    asm volatile("global_load_dwordx4 %0, %1, %2 sc0 sc1\n\ts_waitcnt vmcnt(0)" : "=&v"(D3) : "v"(lane * 16), "s"(src + 4096) : "memory");
    o[192] = D3;
    return;
  }
  const u32 lo4 = ((hdr >> 20) & 31) * 0x04040404u, nhalf = hdr & 1023, nexc = (hdr >> 10) & 1023;
  u32x4 O0, O1, O2, O3;
#pragma unroll
  for (int e = 0; e < 4; ++e) {
    const u32 d0 = D[0][e], d1 = D[1][e], d2 = D[2][e];
    u32 s0 = d0 & 0x60606060u, s1 = d1 & 0x60606060u, s2 = d2 & 0x60606060u;
    asm("" : "+v"(s0), "+v"(s1), "+v"(s2));
    O0[e] = (d0 ^ s0) + lo4; O1[e] = (d1 ^ s1) + lo4; O2[e] = (d2 ^ s2) + lo4;
    const u32 t = (s0 >> 5) | (s1 >> 3) | (s2 >> 1), z = t & 0x20202020u;
    O3[e] = t + lo4 + 3 * z;
  }
  o[0] = O0; o[64] = O1; o[128] = O2; o[192] = O3;
  for (u32 base = 0;;) {
    apply_records(ev, base - 2, nexc, nhalf, dst, lane);
    base += 128;
    if (base >= nexc + 2) break;
    asm volatile("global_load_dword %0, %1, %2 offset:3072 sc0 sc1\n\ts_waitcnt vmcnt(0)" : "=&v"(ev) : "v"(lane * 4 + 2 * base), "s"(src) : "memory");
  }
}

constexpr int B_HDR = 2560, B_STR = 2576;
__device__ __forceinline__ u64 tier2_exponents(int b) {
  if (b >= 10) return (u64)(4 * b) * 0x0101010101010101ull + 12 - 0x1c1814100c080400ull;
  u64 r = (u64)(4 * (b + 3));
#pragma unroll
  for (int k = 1; k < 8; ++k) r |= (u64)(4 * max(b - k, 3)) << (8 * k);
  return r;
}
__device__ __forceinline__ int choose_base(const u32x4 (&V)[4]) {
  u32 sum = 0;
#pragma unroll
  for (int t = 0; t < 4; ++t) sum = __builtin_amdgcn_sad_u8(V[t][3 - t] & 0x7c7c7c7cu, 0u, sum);
  int b = (int)((wave_sum(sum) + 1638u) >> 12) - 2;
  b = b < 0 ? 0 : (b > 26 ? 26 : b);
  u32 cnts = 0;
#pragma unroll
  for (int k = 0; k < 3; ++k) {
    const u32 c0 = (u32)(0x80 - 4 * (b + k)) * 0x01010101u;
    u32 n = 0;
#pragma unroll
    for (int t = 0; t < 4; ++t) {
      const u32 q = (V[t][t] & 0x7f7f7f7fu) + c0;
      n += __builtin_popcount(q & ~(q - 0x0c0c0c0cu) & 0x80808080u);
    }
    cnts |= min(n, 15u) << (10 * k);
  }
  const u32 tot = wave_sum(cnts);
  const u32 n0 = tot & 1023, n1 = (tot >> 10) & 1023, n2 = tot >> 20;
  return b + ((n1 >= n0 && n1 >= n2) ? 1 : (n0 >= n2 ? 0 : 2));
}

__device__ __forceinline__ u32 encode_b(const u32x4 (&V)[4], int b, u32x4 (&N)[2], LAS unsigned char* scr, LAS unsigned char* lut, int lane) {
  const u32 vo = lane * 16;
  LAS unsigned char* tail = scr + 4096;
  *(LAS u32x4*)(tail + vo) = u32x4{0u, 0u, 0u, 0u};
  *(LAS u32x4*)(tail + 1024 + vo) = u32x4{0u, 0u, 0u, 0u};
  const u32 c1 = (u32)(0x83 + 4 * (b + 3)) * 0x01010101u;
  const u64 ex = tier2_exponents(b);
  const u32 exlo = (u32)ex, exhi = (u32)(ex >> 32);
  u32 M[2], S[2] = {0u, 0u}, o[4];
  u64 acc[4];
#pragma unroll
  for (int t = 0; t < 4; ++t) {
    u32x4 X;
    u32 n1[4], k[4], e1[4];
#pragma unroll
    for (int e = 0; e < 4; ++e) {
      const u32 v = V[t][e], m = v & 0x7f7f7f7fu;
      const u32 q = c1 - m;
      const u32 F = ((((q ^ 0x80808080u) & 0xf0f0f0f0u) >> 1) + 0x78787878u) & 0x80808080u;
      const u32 f4 = F >> 4, f5 = F >> 5;
      n1[e] = ~q | f4 | f5;
      e1[e] = (n1[e] >> 3) & (n1[e] >> 2) & 0x01010101u;
      k[e] = ((q - 0x0c0c0c0cu) >> 2) & (f4 - (F >> 7));
      X[e] = (__builtin_amdgcn_perm(exhi, exlo, k[e]) ^ m) & (F - f5);
      S[t >> 1] |= (v >> (4 * (t & 1) + e)) & (0x80808080u >> (4 * (t & 1) + e));
    }
    *(LAS u32x4*)(scr + t * 1024 + vo) = X;
    const u32 m16 = gather16(X);
    if (t & 1) M[t >> 1] |= m16 << 16; else M[t >> 1] = m16;
    N[t >> 1][2 * (t & 1)] = (n1[0] & 0x0f0f0f0fu) | ((n1[1] << 4) & 0xf0f0f0f0u);
    N[t >> 1][2 * (t & 1) + 1] = (n1[2] & 0x0f0f0f0fu) | ((n1[3] << 4) & 0xf0f0f0f0u);
    u32 ob = 0;
    u64 a = 0;
#pragma unroll
    for (int e = 0; e < 4; ++e) {
      const i32x2 wm = *(LAS i32x2*)(lut + __builtin_amdgcn_udot4(e1[e], 0x40201008u, 0u, false));
      const u32 c12 = __builtin_amdgcn_udot4(k[e], (u32)wm.x, (k[e] >> 15) & (u32)wm.y, false);
      a |= (u64)c12 << ob;
      ob = __builtin_amdgcn_udot4(e1[e], 0x03030303u, ob, false);
    }
    acc[t] = a; o[t] = ob;
  }
  const u32 nb = o[0] + o[1] + o[2] + o[3];
  const u32 nr = __builtin_popcount(M[0]) | (__builtin_popcount(M[1]) << 16);
  const u32 ib = wave_scan(nb), ir = wave_scan(nr);
  const u32 bits = __builtin_amdgcn_readlane(ib, 63), totr = __builtin_amdgcn_readlane(ir, 63);
  const u32 nhalf = totr & 0xffffu, nexc = nhalf + (totr >> 16);
  const u32 rec0 = B_STR + ((bits + 31) >> 5) * 4;
  const u32 len = rec0 + 2 * nexc;
  if (bits > 3 * 2040 || len > 4032 || nexc > MAXEXC) return 0;
  *(LAS i32x2*)(tail + lane * 8) = i32x2{(int)S[0], (int)S[1]};
  if (lane == 0) *(LAS u32x4*)(tail + B_HDR - 2048) = u32x4{nhalf | (nexc << 10) | ((u32)b << 20), bits | (rec0 << 16), len, 0u};
  u32 bp = ib - nb + (B_STR - 2048) * 8;
#pragma unroll
  for (int t = 0; t < 4; ++t) {
    LAS u32* w = (LAS u32*)(tail + ((bp >> 5) << 2));
    const u32 sh = bp & 31, hi = (u32)(acc[t] >> 32);
    const u64 x = acc[t] << sh;
    __hip_atomic_fetch_or(w, (u32)x, __ATOMIC_RELAXED, __HIP_MEMORY_SCOPE_WORKGROUP);
    __hip_atomic_fetch_or(w + 1, (u32)(x >> 32), __ATOMIC_RELAXED, __HIP_MEMORY_SCOPE_WORKGROUP);
    __hip_atomic_fetch_or(w + 2, (hi >> 1) >> (31 - sh), __ATOMIC_RELAXED, __HIP_MEMORY_SCOPE_WORKGROUP);
    bp += o[t];
  }
  write_records<1>(M, rec0 - 2048 + 2 * ((ir - nr) & 0xffffu), rec0 - 2048 + 2 * (nhalf + ((ir - nr) >> 16)), scr, tail, 2048 + 2 * lane, vo);
  return len;
}

__device__ __forceinline__ void unpack_b(const unsigned char* src, LAS unsigned char* dst, LAS unsigned char* lut, int lane) {
  LAS u32x4* o = (LAS u32x4*)(dst + lane * 16);
  u32x4 N[2] = {o[0], o[64]};
  i32x2 S = *(LAS i32x2*)(dst + 2048 + lane * 8);
  i32x2 H = *(LAS i32x2*)(dst + B_HDR);
  const u32 so = B_STR + ((6 * lane) & ~3);
  u32 c0 = *(LAS u32*)(dst + so), c1 = *(LAS u32*)(dst + so + 4);
  asm volatile("" : "+v"(N[0]), "+v"(N[1]), "+v"(S), "+v"(H), "+v"(c0), "+v"(c1));
  const u32 hdr = __builtin_amdgcn_readfirstlane(H.x), h1 = __builtin_amdgcn_readfirstlane(H.y);
  if (hdr >> 31) {
    u32x4 D3, D4;
    asm volatile("global_load_dwordx4 %0, %2, %3 offset:3072 sc0 sc1\n\tglobal_load_dwordx4 %1, %4, %3 sc0 sc1\n\ts_waitcnt vmcnt(0)"
                 : "=&v"(D3), "=&v"(D4) : "v"(lane * 16), "s"(src), "v"(4096u) : "memory");
    o[192] = D3;
    *(LAS u32x4*)(dst + B_HDR) = D4;
    return;
  }
  const int b = (hdr >> 20) & 31;
  const u32 nhalf = hdr & 1023, nexc = (hdr >> 10) & 1023, bits = h1 & 0xffffu, rec0 = h1 >> 16;
  if (rec0 + 2 * nexc > 3072) {
    u32x4 D3;
    asm volatile("global_load_dwordx4 %0, %1, %2 offset:3072 sc0 sc1\n\ts_waitcnt vmcnt(0)" : "=&v"(D3) : "v"(lane * 16), "s"(src) : "memory");
    o[192] = D3;
  }
  u32 ev = *(LAS u32*)(dst + min(rec0 + lane * 4, 4092u));
  u32 c2 = 0, c3 = 0;
  if (bits > 3072) { c2 = *(LAS u32*)(dst + so + 384); c3 = *(LAS u32*)(dst + so + 388); }
  const u64 ex = tier2_exponents(b) - 0x0c0c0c0c0c0c0c0cull;
  const u32 exlo = (u32)ex, exhi = (u32)(ex >> 32);
  auto expand = [&](u32 a0, u32 a1, int p) {
    const u32 sh = (lane & 1) * 16;
    const u32 x = __builtin_amdgcn_alignbit(a1, a0, sh), y = a1 >> sh;
    const u32 g[4] = {x & 0xfffu, (x >> 12) & 0xfffu, __builtin_amdgcn_alignbit(y, x, 24) & 0xfffu, (y >> 4) & 0xfffu};
    u32x4 r;
#pragma unroll
    for (int i = 0; i < 4; ++i) {
      const u32 z = (g[i] | (g[i] << 10)) & 0x003f003fu;
      r[i] = __builtin_amdgcn_perm(exhi, exlo, (z | (z << 5)) & 0x07070707u);
    }
    o[p * 64] = r;
    o[128 + p * 64] = u32x4{r[1], r[2], r[3], (u32)__builtin_amdgcn_update_dpp(0u, r[0], 0x130, 0xf, 0xf, true)};
    if (p && lane == 0) *(LAS u32*)(dst + 2044 + 1024) = r[0];
  };
  expand(c0, c1, 0);
  if (bits > 3072) expand(c2, c3, 1);
  u32 G[8], n = 0;
#pragma unroll
  for (int i = 0; i < 8; ++i) {
    const u32 x = N[i >> 2][i & 3];
    G[i] = x & (x >> 1) & 0x44444444u;
    n += __builtin_popcount(G[i]);
  }
  u32 wa = (u32)(size_t)dst + wave_scan(n) - n;
  const u32 b4 = (u32)(4 * b) * 0x01010101u;
  u32 wad[16], sel[16];
  i32x2 W[16];
#pragma unroll
  for (int j = 0; j < 16; ++j) {
    const u32 f = ((j & 1) ? G[j >> 1] >> 4 : G[j >> 1]) & 0x04040404u;
    sel[j] = *(LAS u32*)(lut + __builtin_amdgcn_udot4(f, 0x08040201u, 0u, false));
    W[j] = *(LAS i32x2*)__builtin_assume_aligned((LAS void*)(size_t)((wa + (wa & 4u) * 511u) & ~3u), 8);
    wad[j] = wa;
    wa += __builtin_popcount(f);
  }
  asm volatile("" : "+v"(W[0]), "+v"(W[1]), "+v"(W[2]), "+v"(W[3]), "+v"(W[4]), "+v"(W[5]), "+v"(W[6]), "+v"(W[7]),
               "+v"(sel[0]), "+v"(sel[1]), "+v"(sel[2]), "+v"(sel[3]), "+v"(sel[4]), "+v"(sel[5]), "+v"(sel[6]), "+v"(sel[7]));
  asm volatile("" : "+v"(W[8]), "+v"(W[9]), "+v"(W[10]), "+v"(W[11]), "+v"(W[12]), "+v"(W[13]), "+v"(W[14]), "+v"(W[15]),
               "+v"(sel[8]), "+v"(sel[9]), "+v"(sel[10]), "+v"(sel[11]), "+v"(sel[12]), "+v"(sel[13]), "+v"(sel[14]), "+v"(sel[15]));
#pragma unroll
  for (int t = 0; t < 4; ++t) {
    u32x4 O;
#pragma unroll
    for (int e = 0; e < 4; ++e) {
      const int j = 4 * t + e;
      const u32 x = N[j >> 3][(j >> 1) & 3];
      const u32 nib = ((j & 1) ? x >> 4 : x) & 0x0f0f0f0fu;
      const u32 sg = ((u32)S[t >> 1] << (j & 7)) & 0x80808080u;
      O[e] = (nib | sg) + __builtin_amdgcn_perm(b4, __builtin_amdgcn_alignbyte((u32)W[j].y, (u32)W[j].x, wad[j]), sel[j]);
    }
    o[t * 64] = O;
  }
  for (u32 base = 0;;) {
    apply_records(ev, base, nexc, nhalf, dst, lane);
    base += 128;
    if (base >= nexc) break;
    asm volatile("global_load_dword %0, %1, %2 sc0 sc1\n\ts_waitcnt vmcnt(0)" : "=&v"(ev) : "v"(rec0 + lane * 4 + 2 * base), "s"(src) : "memory");
  }
}

__device__ __forceinline__ void init_luts(LAS unsigned char* lut, int lane) {
  if (lane < 16) {
    u32 w = 0, sel = 0, r = 0;
#pragma unroll
    for (int i = 0; i < 4; ++i) {
      const bool on = (lane >> i) & 1;
      w |= (on && r < 3 ? 1u << (3 * r) : 0u) << (8 * i);
      sel |= (on ? r : 4u) << (8 * i);
      r += on;
    }
    *(LAS i32x2*)(lut + lane * 8) = i32x2{(int)w, lane == 15 ? 0xe00 : 0};
    *(LAS u32*)(lut + 128 + lane * 4) = sel;
  }
}

__global__ void __launch_bounds__(NTHREADS, 3) __attribute__((amdgpu_num_vgpr(128))) fused_kernel(Params P) {
  extern __shared__ __attribute__((aligned(16))) unsigned char lds_raw[];
  LAS unsigned char* L = (LAS unsigned char*)lds_raw;
  volatile LAS u32* READY = (volatile LAS u32*)(L + LDS_READY);
  volatile LAS u32* DONE = (volatile LAS u32*)(L + LDS_DONE);
  volatile LAS u32* BAD = (volatile LAS u32*)(L + LDS_BAD);
  volatile LAS u32* GO = (volatile LAS u32*)(L + LDS_GO);

  const int c = blockIdx.x;
  const int lane = threadIdx.x & 63;
  const int wave = __builtin_amdgcn_readfirstlane(threadIdx.x >> 6);
  const int u = (c & 7) * 8 + ((c >> 3) & 7);
  const int mt = c >> 6;
  const int pr = ((c >> 3) & 7) * 32 + mt * 8 + (c & 7);
  unsigned char* const ws = P.peer[0];
  STAMP(0);

  unsigned char* cntp = ws + WS_CNT + c * 128 + wave * 4;
#define START_BARRIER() do { __builtin_amdgcn_sched_barrier(0); __builtin_amdgcn_s_barrier(); __builtin_amdgcn_sched_barrier(0); } while (0)
  const u32 w_lane = (u32)(lane >> 1) * NN + (lane & 1) * 16;
  const u32 vo = lane * 16;

  if (wave == NWAVES - 1) {
    __builtin_amdgcn_s_waitcnt(0x0F70);
    {
      const int d = pr < 224 ? 1 + pr % 7 : 0, j = pr < 224 ? pr / 7 : pr - 224;
      const unsigned char* xs = P.x + (size_t)(((P.rank + d) & 7) * MS + (j >> 1) * 32) * KS + (j & 1) * 256;
      const u32 xo = (u32)(lane & 31) * KS + 32 * (lane >> 5);
#define ACLOB "v128", "v129", "v130", "v131", "v132", "v133", "v134", "v135", "v136", "v137", "v138", "v139", "v140", "v141", "v142", "v143", "v144", "v145", "v146", "v147", \
              "v148", "v149", "v150", "v151", "v152", "v153", "v154", "v155", "v156", "v157", "v158", "v159", "v160", "v161", "memory"
      asm volatile("global_load_dword v160, %0, %1 sc1\n\t"
                   "global_load_dwordx4 v[128:131], %2, %3\n\tglobal_load_dwordx4 v[132:135], %2, %3 offset:16\n\t"
                   "global_load_dwordx4 v[136:139], %2, %3 offset:64\n\tglobal_load_dwordx4 v[140:143], %2, %3 offset:80\n\t"
                   "global_load_dwordx4 v[144:147], %2, %3 offset:128\n\tglobal_load_dwordx4 v[148:151], %2, %3 offset:144\n\t"
                   "global_load_dwordx4 v[152:155], %2, %3 offset:192\n\tglobal_load_dwordx4 v[156:159], %2, %3 offset:208\n\t"
                   "v_mov_b32 v161, 1"
                   :: "v"(0u), "s"(cntp), "v"(xo), "s"(xs) : ACLOB);
      START_BARRIER();
      __builtin_amdgcn_s_setprio(2);
      u32 cnt;
      asm volatile("s_waitcnt vmcnt(8)\n\tv_mov_b32 %0, v160" : "=v"(cnt) :: "memory");
      const u32 call = __builtin_amdgcn_readfirstlane(cnt) + 1;
      unsigned char* pd = P.peer[d];
      unsigned char* dst = pd + WS_XB + (call & 1) * XB_PAR + (size_t)(d * 32 + j) * 8192;
#define XST(k, r) asm volatile("s_waitcnt vmcnt(%2)\n\tglobal_store_dwordx4 %0, " r ", %1 offset:%3 sc0 sc1\n\ts_nop 1" \
                               :: "v"(vo), "s"(dst + ((k) >> 2) * 4096), "i"((k) < XW ? 7 : XW - 1), "i"(((k) & 3) * 1024) : "memory")
#define XSTS(a, b) do { if (a <= 0 && 0 < b) XST(0, "v[128:131]"); if (a <= 1 && 1 < b) XST(1, "v[132:135]"); if (a <= 2 && 2 < b) XST(2, "v[136:139]"); \
        if (a <= 3 && 3 < b) XST(3, "v[140:143]"); if (a <= 4 && 4 < b) XST(4, "v[144:147]"); if (a <= 5 && 5 < b) XST(5, "v[148:151]"); \
        if (a <= 6 && 6 < b) XST(6, "v[152:155]"); if (a <= 7 && 7 < b) XST(7, "v[156:159]"); } while (0)
      XSTS(0, XW);
      STAMP(1);
      XSTS(XW, 8);
      GO[0] = 1u;
#undef XSTS
#undef XST
      if (lane == 0) ag_store(cntp, call);
      VMWAIT(0);
      if (lane < 16) flag_store(pd + WS_FX + (d * 32 + j) * 64 + lane * 4, call);
      STAMP(8);
      if (d != 0 && j < NPW) {
        const unsigned char* tkp = ws + WS_TK + d * 128;
        const unsigned char* const rsrc = P.peer[R - d] + WS_ST + (call & 1) * WB_SRC;
        unsigned char* const ldst = ws + WS_WB + (call & 1) * WB_PAR + d * WB_SRC;
        unsigned char* const lfw = ws + WS_FW + d * 64 * 512;
        const u32 tbase = (call - 1) * 512u - 2 * NPW;
        u32 staged = 0;
#define FETCHR(t) do { \
          while ((t) >= staged) { staged = __builtin_amdgcn_readfirstlane(__hip_atomic_load((LAS u32*)(L + LDS_STG), __ATOMIC_ACQUIRE, __HIP_MEMORY_SCOPE_WORKGROUP)); \
                                  if ((t) >= staged) __builtin_amdgcn_s_sleep(2); } \
          if (staged > 512u) { STAMP_OUT(); return; }       \
          const unsigned char* s_ = rsrc + (size_t)(t) * SLOT; \
          asm volatile("global_load_dwordx4 v[128:131], %0, %1 sc0 sc1\n\tglobal_load_dwordx4 v[132:135], %0, %1 offset:1024 sc0 sc1\n\t" \
                       "global_load_dwordx4 v[136:139], %0, %1 offset:2048 sc0 sc1" :: "v"(vo), "s"(s_) : ACLOB); \
          if ((t) >= U1 && lane < 16) asm volatile("global_load_dwordx4 v[140:143], %0, %1 offset:3072 sc0 sc1" :: "v"(vo), "s"(s_) : ACLOB); } while (0)
        u32 cur = j;
        asm volatile("v_mov_b32 v160, %0" :: "v"(tbase + NPW + j) : ACLOB);
        FETCHR(cur);
#pragma unroll 1
        while (true) {
          u32 tk, hb, ha;
          asm volatile("s_waitcnt vmcnt(0)\n\tv_readfirstlane_b32 %0, v160\n\tv_readlane_b32 %1, v138, 32\n\tv_readfirstlane_b32 %2, v140"
                       : "=s"(tk), "=s"(hb), "=s"(ha) :: "memory");
          const u32 nxt = tk - tbase;
          const u32 len = cur < U1 ? hb : ((ha >> 31) ? 5120u : 3076u + 2 * ((ha >> 10) & 1023));
          unsigned char* o_ = ldst + (size_t)((cur & ~63u) + UTILE(cur)) * SLOT;
          if (len > (cur < U1 ? 3072u : 3328u)) {
            u32x4 r3, r4;
            const unsigned char* s_ = rsrc + (size_t)cur * SLOT;
            asm volatile("global_load_dwordx4 %0, %2, %3 offset:3072 sc0 sc1\n\tglobal_load_dwordx4 %1, %2, %4 sc0 sc1\n\ts_waitcnt vmcnt(0)"
                         : "=&v"(r3), "=&v"(r4) : "v"(vo), "s"(s_), "s"(s_ + 4096) : "memory");
            unsigned char* o2 = o_ + 4096;
            ST16A(o_, vo, r3, 3072); ST16A(o2, vo, r4, 0);
          } else if (cur >= U1 && lane < 16) {
            asm volatile("global_store_dwordx4 %0, v[140:143], %1 offset:3072 sc1\n\ts_nop 1" :: "v"(vo), "s"(o_) : "memory");
          }
          asm volatile("global_store_dwordx4 %0, v[128:131], %1 sc1\n\tglobal_store_dwordx4 %0, v[132:135], %1 offset:1024 sc1\n\t"
                       "global_store_dwordx4 %0, v[136:139], %1 offset:2048 sc1\n\ts_nop 1" :: "v"(vo), "s"(o_) : "memory");
          if (lane == 0) asm volatile("global_atomic_add v160, %0, v161, %1 sc0 sc1" :: "v"(0u), "s"(tkp) : ACLOB);
          if (nxt < 512u) {
            FETCHR(nxt);
            VMWAIT(4);
          } else {
            VMWAIT(0);
          }
          if (lane == 0) ag_store(lfw + UTILE(cur) * 512 + (cur >> 6) * 64, call);
          if ((cur & 32) == 0 && cur >= 64) STAMP(8 + (cur >> 6));
          if (nxt >= 512u) break;
          cur = nxt;
        }
#undef FETCHR
      }
      STAMP(3);
#undef ACLOB
    }
    STAMP_OUT();
    return;
  }
  const u64 t_start = rt();
  const u32 cnt_old = ag_load(cntp);
  if (wave == 5) asm volatile("ds_write_b32 %0, %1\n\ts_waitcnt lgkmcnt(0)" :: "v"((u32)(size_t)L + LDS_CTRL + lane * 4), "v"(0u) : "memory");
  if (wave == 6) { init_luts(L + LDS_LUT, lane); asm volatile("s_waitcnt lgkmcnt(0)" ::: "memory"); }
  u32 call;
  if (pr >= 224) {
    START_BARRIER();
    call = __builtin_amdgcn_readfirstlane(cnt_old) + 1;
    LAS unsigned char* scr = L + LDS_SCR + wave * 6400;
    if (wave < 4) __builtin_amdgcn_s_setprio(1);
    unsigned char* lane_peer = nullptr;
#pragma unroll
    for (int d = 1; d < R; ++d) lane_peer = lane == d ? P.peer[d] : lane_peer;
#pragma unroll 1
    for (u32 e = (pr - 224) + 32 * wave; e < 512u && wave < 8; e += 256) {
      const unsigned char* src = P.w + (size_t)(e >> 6) * 64 * NN + UTILE(e) * 64 + w_lane;
      u32x4 V[4], D[4], T;
      V[0] = *(const GAS u32x4*)(src);
      V[1] = *(const GAS u32x4*)(src + 32 * NN);
      V[2] = *(const GAS u32x4*)(src + 32);
      V[3] = *(const GAS u32x4*)(src + 32 * NN + 32);
      u32 len;
      if (e >= U1) {
        const u32 nexc = encode_piece(V, choose_window(V), D, T, scr, lane);
        len = nexc > MAXEXC ? 5120u : 3076u + 2 * nexc;
      } else {
        u32x4 N[2];
        len = encode_b(V, choose_base(V), N, scr, L + LDS_LUT, lane);
        if (len) {
          D[0] = N[0]; D[1] = N[1];
          D[2] = *(LAS u32x4*)(scr + 4096 + vo); D[3] = *(LAS u32x4*)(scr + 5120 + vo);
        } else {
          len = 4112;
          D[0] = V[0]; D[1] = V[1]; D[3] = V[3];
          D[2] = lane == 32 ? u32x4{0x80000000u, 0u, 4112u, 0u} : V[2];
#pragma unroll
          for (int q = 0; q < 4; ++q) T[q] = __builtin_amdgcn_readlane(V[2][q], 32);
        }
      }
      unsigned char* dst = ws + WS_ST + (call & 1) * WB_SRC + (size_t)e * SLOT;
      ST16(dst, vo, D[0], 0); ST16(dst, vo, D[1], 1024); ST16(dst, vo, D[2], 2048);
      if (3072 + (vo & ~63u) < len) ST16(dst, vo, D[3], 3072);
      if (4096 + (vo & ~63u) < len) { unsigned char* d2 = dst + 4096; ST16(d2, vo, T, 0); }
      if (e < 128) STAMP(6);
      VMWAIT(0);
      {
        const u32 grp = min(e >> 7, 2u);
        u32 old = 0;
        if (lane == 0) old = __hip_atomic_fetch_add((LAS u32*)(L + LDS_GRP + grp * 4), 1u, __ATOMIC_RELAXED, __HIP_MEMORY_SCOPE_WORKGROUP);
        if (__builtin_amdgcn_readfirstlane(old) == (grp == 2 ? 7 : 3) && lane >= 1 && lane < R)
          flag_store(lane_peer + WS_FS + lane * 512 + grp * 128 + (pr - 224) * 4, call);
      }
      STAMP(5);
      __builtin_amdgcn_s_setprio(0);
    }
    if (lane == 0) __hip_atomic_fetch_add((LAS u32*)(L + LDS_ENCD), 1u, __ATOMIC_RELAXED, __HIP_MEMORY_SCOPE_WORKGROUP);
    while (*(volatile LAS u32*)(L + LDS_ENCD) < 11u) __builtin_amdgcn_s_sleep(2);
    asm volatile("" ::: "memory");
  } else {
    START_BARRIER();
    call = __builtin_amdgcn_readfirstlane(cnt_old) + 1;
  }
  if (lane == 0) ag_store(cntp, call);
  if (wave == 3 && pr < 7 * NPW) {
    while (rt() - t_start < TPOLL) __builtin_amdgcn_s_sleep(8);
    const unsigned char* fs = ws + WS_FS + (1 + pr % 7) * 512 + (lane & 31) * 4;
    bool ok = wait_all<true, 8>(fs, call, ws, call);
    __hip_atomic_store((LAS u32*)(L + LDS_STG), ok ? 128u : 1024u, __ATOMIC_RELEASE, __HIP_MEMORY_SCOPE_WORKGROUP);
    STAMP(4);
    __builtin_amdgcn_s_sleep(80);
    ok = ok && wait_all<true, 32>(fs + 128, call, ws, call);
    __hip_atomic_store((LAS u32*)(L + LDS_STG), ok ? 256u : 1024u, __ATOMIC_RELEASE, __HIP_MEMORY_SCOPE_WORKGROUP);
    __builtin_amdgcn_s_sleep(80);
    ok = ok && wait_all<true, 32>(fs + 256, call, ws, call);
    __hip_atomic_store((LAS u32*)(L + LDS_STG), ok ? 512u : 1024u, __ATOMIC_RELEASE, __HIP_MEMORY_SCOPE_WORKGROUP);
  }
  while (GO[0] == 0u) __builtin_amdgcn_s_sleep(4);

  if (wave < 4) {
    const int mymb = mt * 4 + wave;
    const u32 par = call & 1;
    const float scale = P.sx[0] * P.sw[0];
    const u32 b_lane = (u32)(size_t)L + ((lane >> 5) * 32 + ((lane & 15) >> 1)) * 32 + ((lane >> 4) & 1) * 16 + (lane & 1) * 8;
    f32x16 acc0 = {}, acc1 = {};
    u32 rv;
    u32x4 AL[8] = {}, AH[8] = {};
#define AWAIT() asm volatile("s_waitcnt vmcnt(0)" : "+v"(AL[0]), "+v"(AL[1]), "+v"(AL[2]), "+v"(AL[3]), "+v"(AL[4]), "+v"(AL[5]), "+v"(AL[6]), "+v"(AL[7]), \
                             "+v"(AH[0]), "+v"(AH[1]), "+v"(AH[2]), "+v"(AH[3]), "+v"(AH[4]), "+v"(AH[5]), "+v"(AH[6]), "+v"(AH[7]) :: "memory")
#define MMA(p, off) do { \
          i32x8 b0, b1; read_bfrag(b0, baddr, off); read_bfrag(b1, baddr, off + 2048); \
          const i32x8 a = {(int)AL[p].x, (int)AL[p].y, (int)AL[p].z, (int)AL[p].w, (int)AH[p].x, (int)AH[p].y, (int)AH[p].z, (int)AH[p].w}; \
          __builtin_amdgcn_sched_barrier(0); \
          acc0 = __builtin_amdgcn_mfma_scale_f32_32x32x64_f8f6f4(b0, a, acc0, 1, 0, 0, 0, 0, 0); \
          acc1 = __builtin_amdgcn_mfma_scale_f32_32x32x64_f8f6f4(b1, a, acc1, 1, 0, 0, 0, 0, 0); } while (0)
#define POLL(row, want) do { while (true) { \
          rv = READY[(row) * 8 + (lane & 7)]; \
          m = (u32)__ballot(rv >= (u32)(want)) & rem; \
          if (m) break; \
          __builtin_amdgcn_s_sleep(1); } \
        asm volatile("" ::: "memory"); rem &= ~m; } while (0)
    const unsigned char* xb = ws + WS_XB + par * XB_PAR + (size_t)mymb * 16384;
    {
      if (!wait_all<true>(ws + WS_FX + mymb * 128 + (lane & 1) * 64, call, ws, call)) BAD[0] = 1u;
#define LA(p) do { const unsigned char* s_ = xb + p * 2048; LD16S(AL[p], s_, vo, 0); LD16S(AH[p], s_, vo, 1024); } while (0)
      LA(0); LA(1); LA(2); LA(3); LA(4); LA(5); LA(6); LA(7);
#undef LA
      AWAIT();
      const u32 baddr = b_lane + LDS_OWN;
      u32 rem = 0xFFu, m;
      do {
        POLL(RR, 1);
#define STEP(p) if (m & (1u << p)) MMA(p, p * PIECE);
        STEP(0) STEP(1) STEP(2) STEP(3) STEP(4) STEP(5) STEP(6) STEP(7)
#undef STEP
      } while (rem);
    }
    STAMP(7);
    if (!wait_all<true>(ws + WS_FX + ((1 + (lane & 7) % 7) * 16 + mymb) * 128 + ((lane >> 3) & 1) * 64, call, ws, call)) BAD[0] = 1u;
    STAMP(2);
#define LA(p) do { const unsigned char* s_ = xb + p * 262144; LD16S(AL[p], s_, vo, 0); LD16S(AH[p], s_, vo, 1024); } while (0)
    LA(1); LA(2); LA(3); LA(4); LA(5); LA(6); LA(7);
#pragma unroll 1
    for (int i = 0; i < 8; ++i) {
      const int row = i & (RR - 1);
      const u32 baddr = b_lane + row * 7 * PIECE;
      xb += 2048;
      AWAIT();
      u32 rem = 0xFEu, m;
      do {
        POLL(row, i + 1);
#define STEP(p) if (m & (1u << p)) { MMA(p, (p - 1) * PIECE); if (i < 7) LA(p); }
        STEP(1) STEP(2) STEP(3) STEP(4) STEP(5) STEP(6) STEP(7)
#undef STEP
      } while (rem);
      asm volatile("s_waitcnt lgkmcnt(0)" ::: "memory");
      if (lane == 0) DONE[wave] = (u32)(i + 1);
      STAMP(8 + i);
    }
#undef LA
    AWAIT();
    const bool bad = __builtin_amdgcn_readfirstlane(rv) != 0u;
    u32x4 o[8];
    const float sc = bad ? __builtin_nanf("") : scale;
    acc0 *= sc; acc1 *= sc;
#pragma unroll
    for (int q = 0; q < 4; ++q)
#pragma unroll
      for (int e = 0; e < 4; ++e) {
        o[q][e] = __builtin_bit_cast(u32, __builtin_elementwise_maximum(acc0[4 * q + e], 0.f));
        o[4 + q][e] = __builtin_bit_cast(u32, __builtin_elementwise_maximum(acc1[4 * q + e], 0.f));
      }
    const unsigned char* ob = (const unsigned char*)(P.out + (size_t)mymb * 32 * NN + u * 64);
#define OST(val, voff) asm volatile("global_store_dwordx4 %0, %1, %2 sc1\n\ts_nop 1" :: "v"(voff), "v"(val), "s"(ob) : "memory")
    {
      LAS unsigned char* tb = L + LDS_OWN + wave * 8192;
      const int m = lane & 31, h = lane >> 5;
#pragma unroll
      for (int q = 0; q < 8; ++q) *(LAS u32x4*)(tb + m * 256 + (((2 * q + h) ^ (m & 15)) << 4)) = o[q];
      const int r0 = lane >> 4, ch = lane & 15;
#pragma unroll
      for (int t = 0; t < 8; ++t) o[t] = *(LAS u32x4*)(tb + (4 * t + r0) * 256 + ((ch ^ ((4 * t + r0) & 15)) << 4));
      u32 oo = (u32)r0 * NN * 4 + ch * 16;
#pragma unroll
      for (int t = 0; t < 8; ++t) { OST(o[t], oo); oo += 4 * NN * 4; }
    }
#undef OST
    STAMP(3);
  } else {
    const int d = wave - 3;
    {
      const unsigned char* src = P.w + u * 64 + w_lane;
      for (int i = d - 1; i < 8; i += 7) {
        const unsigned char* s = src + (size_t)i * 64 * NN;
        LAS unsigned char* dst = L + LDS_OWN + i * PIECE;
        __builtin_amdgcn_global_load_lds((const GAS u32*)(s), (LAS u32*)(dst), 16, 0, 0);
        __builtin_amdgcn_global_load_lds((const GAS u32*)(s + 32 * NN), (LAS u32*)(dst + 1024), 16, 0, 0);
        __builtin_amdgcn_global_load_lds((const GAS u32*)(s + 32), (LAS u32*)(dst + 2048), 16, 0, 0);
        __builtin_amdgcn_global_load_lds((const GAS u32*)(s + 32 * NN + 32), (LAS u32*)(dst + 3072), 16, 0, 0);
      }
    }
    VMWAIT(0);
    if (lane == 0) { READY[RR * 8 + d - 1] = 1u; if (d == 1) READY[RR * 8 + 7] = 1u; }
    const unsigned char* fp = ws + WS_FW + (d * 64 + u) * 512;
    const unsigned char* src = ws + WS_WB + (call & 1) * WB_PAR + d * WB_SRC + (size_t)u * SLOT;
    u32 seen = 0;
    bool dead = false, pre = false;
    const unsigned char* tmop = ws + WS_TMO;
    const u32 l4 = lane * 4;
#define FMT_A(i_) (!FMT_B((i_) * 64 + UTILE(u)))
#define FETCH(i_, s_) do { u32 keep_; const u32 la_ = (u32)(size_t)L + ((((i_) & (RR - 1)) * 7 + d - 1) * PIECE); \
      asm volatile("s_mov_b32 %0, m0\n\ts_mov_b32 m0, %3\n\ts_nop 0\n\t" \
                   "global_load_lds_dwordx4 %1, %2 sc0 sc1\n\t" \
                   "global_load_lds_dwordx4 %1, %2 offset:1024 sc0 sc1\n\t" \
                   "global_load_lds_dwordx4 %1, %2 offset:2048 sc0 sc1\n\t" \
                   "s_mov_b32 m0, %0" : "=&s"(keep_) : "v"(vo), "s"(s_), "s"(la_) : "memory"); \
      if (FMT_A(i_)) \
        asm volatile("s_mov_b32 %0, m0\n\ts_mov_b32 m0, %3\n\ts_nop 0\n\tglobal_load_lds_dword %1, %2 offset:3072 sc0 sc1\n\ts_mov_b32 m0, %0" \
                     : "=&s"(keep_) : "v"(l4), "s"(s_), "s"(la_) : "memory"); } while (0)
#pragma unroll 1
    for (int i = 0; i < 8; ++i, src += 64 * SLOT) {
      if (!pre) {
        if (i >= RR) {
          while (!__all(DONE[lane & 3] + RR > (u32)i)) __builtin_amdgcn_s_sleep(2);
          asm volatile("" ::: "memory");
        }
        if (!((seen >> i) & 1) && !dead) {
          const u64 t0 = rt();
          for (u32 spin = 1;; ++spin) {
            seen = (u32)__ballot(flag_load(fp + max(lane & 7, i) * 64) >= call);
            if ((seen >> i) & 1) break;
            if ((spin & 63) == 0 && (__builtin_amdgcn_readfirstlane(ag_load(tmop)) == call || rt() - t0 > DEADLINE_TICKS)) { dead = true; break; }
            if (i < 7) __builtin_amdgcn_s_sleep(16);
          }
          if (dead) ag_store(ws + WS_TMO, call);
          asm volatile("" ::: "memory");
        }
        if (i == 7) STAMP(4);
        if (!dead) FETCH(i, src);
      }
      pre = !dead && i < 7 && ((seen >> (i + 1)) & 1) && __all(DONE[lane & 3] + RR > (u32)i + 1);
      if (pre) {
        const unsigned char* s2 = src + 64 * SLOT;
        FETCH(i + 1, s2);
        VMWAIT(3);
      } else {
        if (i == 7) dead = dead || __builtin_amdgcn_readfirstlane(ag_load(tmop)) == call;
        VMWAIT(0);
      }
      if (i == 7) STAMP(5);
      const int row = i & (RR - 1);
      if (dead) BAD[0] = 1u;
      else if (FMT_A(i)) unpack_piece(src, L + (row * 7 + d - 1) * PIECE, lane);
      else unpack_b(src, L + (row * 7 + d - 1) * PIECE, L + LDS_LUT + 128, lane);
      asm volatile("s_waitcnt lgkmcnt(0)" ::: "memory");
      READY[row * 8 + d] = (u32)(i + 1);
      STAMP(8 + i);
    }
#undef FETCH
#undef FMT_A
    STAMP(3);
  }
  STAMP_OUT();
}

}

extern "C" void kernel_launch(int rank, int nranks, ncclComm_t comm,
                              hipStream_t stream,
                              void* const* d_in, const int* in_sizes, int n_in,
                              void* d_out, int out_size,
                              const int* in_dtypes, int out_dtype,
                              void* const* d_out_all,
                              void* const* d_ws_all, size_t ws_bytes) {
  (void)comm; (void)in_dtypes; (void)out_dtype; (void)d_out_all;
  if (nranks != R || (unsigned)rank >= (unsigned)R || n_in != 4 || ws_bytes < WS_END ||
      in_sizes[0] != 4096 * KS || in_sizes[1] != KS * NN || out_size != MS * NN) {
    fprintf(stderr, "kernel_launch: unexpected problem shape\n");
    return;
  }
  HIP_CHECK(hipFuncSetAttribute((const void*)fused_kernel, hipFuncAttributeMaxDynamicSharedMemorySize, LDS_BYTES));
  Params P;
  P.x = (const unsigned char*)d_in[0];
  P.w = (const unsigned char*)d_in[1];
  P.sx = (const float*)d_in[2];
  P.sw = (const float*)d_in[3];
  P.out = (float*)d_out;
  for (int d = 0; d < R; ++d) P.peer[d] = (unsigned char*)d_ws_all[(rank + d) % R];
  P.rank = rank;
  P.pad = 0;
  hipLaunchKernelGGL(fused_kernel, dim3(NWG), dim3(NTHREADS), LDS_BYTES, stream, P);
  HIP_CHECK(hipGetLastError());
}
```

```cpp
#include <hip/hip_runtime.h>
#include <rccl/rccl.h>
#include <cstdio>

#define HIP_CHECK(expr) do { hipError_t _e = (expr); if (_e != hipSuccess) { \
    fprintf(stderr, "HIP error %s at %s:%d\n", hipGetErrorString(_e), __FILE__, __LINE__); \
    return; } } while (0)

namespace {

typedef unsigned u32;
typedef unsigned long long u64;
typedef unsigned u32x4 __attribute__((ext_vector_type(4)));
typedef int i32x2 __attribute__((ext_vector_type(2)));
typedef int i32x8 __attribute__((ext_vector_type(8)));
typedef float f32x4 __attribute__((ext_vector_type(4)));
typedef float f32x16 __attribute__((ext_vector_type(16)));
#define LAS __attribute__((address_space(3)))
#define GAS __attribute__((address_space(1)))

constexpr int R = 8;
constexpr int MS = 512;
constexpr int KS = 512;
constexpr int NN = 4096;
constexpr int NWG = 256;
constexpr int NWAVES = 12;
constexpr int NTHREADS = NWAVES * 64;
constexpr int RR = 4;
constexpr int PIECE = 4096;
constexpr int SLOT = 5120;
constexpr int MAXEXC = 510;
constexpr int NPW = 23;
constexpr int XW = 2;
constexpr int TPOLL = 420;
constexpr int LDS_OWN = RR * 7 * PIECE;
constexpr int LDS_CTRL = LDS_OWN + 8 * PIECE;
constexpr int LDS_READY = LDS_CTRL;
constexpr int LDS_DONE = LDS_CTRL + 192;
constexpr int LDS_BAD = LDS_READY + (7 & (RR - 1)) * 32;
constexpr int LDS_GO = LDS_CTRL + 232;
constexpr int LDS_ENCD = LDS_CTRL + 240;
constexpr int LDS_GRP = LDS_CTRL + 208;
constexpr int LDS_STG = LDS_CTRL + 248;
constexpr int LDS_LUT = LDS_CTRL + 256;
constexpr int LDS_SCR = 7 * PIECE;
constexpr int LDS_BYTES = LDS_CTRL + 640;
static_assert(LDS_BYTES <= 160 * 1024, "LDS");

constexpr size_t WS_CNT = 0;
constexpr size_t WS_TMO = 32768;
constexpr size_t WS_TK = 36864;
constexpr size_t WS_FX = 65536;
constexpr size_t WS_FW = 131072;
constexpr u32 U0 = 32, U1 = 448;
constexpr size_t WS_FS = 393216;
constexpr size_t WS_DBG = 524288;
constexpr size_t WS_XB = 1ull << 20;
constexpr size_t XB_PAR = 2ull << 20;
constexpr size_t WB_SRC = (size_t)512 * SLOT;
constexpr size_t WS_WB = 8ull << 20;
constexpr size_t WB_PAR = 8 * WB_SRC;
constexpr size_t WS_ST = WS_WB + 2 * WB_PAR;
constexpr size_t WS_END = WS_ST + 2 * WB_SRC;

constexpr u64 DEADLINE_TICKS = 40000000ull;

struct Params {
  const unsigned char* x;
  const unsigned char* w;
  const float* sx;
  const float* sw;
  float* out;
  unsigned char* peer[R];
  int rank;
  int pad;
};

#define UTILE(t) ((((t) & 7) << 3) | (((t) >> 3) & 7))
#define FMT_B(t) ((u32)(t) - U0 < U1 - U0)

__device__ __forceinline__ u64 rt() { return __builtin_amdgcn_s_memrealtime(); }

#define STAMP(slot) do {} while (0)
#define STAMP_OUT() do {} while (0)

#define ST16(base, voff, val, imm) \
  asm volatile("global_store_dwordx4 %0, %1, %2 offset:%3 sc0 sc1\n\ts_nop 1" :: "v"(voff), "v"(val), "s"(base), "i"(imm) : "memory")
#define LD16S(val, base, voff, imm) \
  asm volatile("global_load_dwordx4 %0, %1, %2 offset:%3 sc0 sc1" : "+v"(val) : "v"(voff), "s"(base), "i"(imm) : "memory")
#define VMWAIT(n) asm volatile("s_waitcnt vmcnt(%0)" :: "i"(n) : "memory")

__device__ __forceinline__ void flag_store(unsigned char* p, u32 v) {
  __hip_atomic_store((GAS u32*)p, v, __ATOMIC_RELAXED, __HIP_MEMORY_SCOPE_SYSTEM);
}
__device__ __forceinline__ u32 flag_load(const unsigned char* p) {
  return __hip_atomic_load((GAS u32*)p, __ATOMIC_RELAXED, __HIP_MEMORY_SCOPE_SYSTEM);
}
__device__ __forceinline__ u32 ag_load(const unsigned char* p) {
  return __hip_atomic_load((GAS u32*)p, __ATOMIC_RELAXED, __HIP_MEMORY_SCOPE_AGENT);
}
__device__ __forceinline__ void ag_store(unsigned char* p, u32 v) {
  __hip_atomic_store((GAS u32*)p, v, __ATOMIC_RELAXED, __HIP_MEMORY_SCOPE_AGENT);
}

template <bool SYS, int NAP = 4>
__device__ __forceinline__ bool wait_all(const unsigned char* fp, u32 want, unsigned char* ws, u32 call) {
  const u64 t0 = rt();
  for (u32 spin = 1;; ++spin) {
    const u32 v = SYS ? flag_load(fp) : ag_load(fp);
    if (__all(v >= want)) break;
    if ((spin & 15) == 0 && (__builtin_amdgcn_readfirstlane(ag_load(ws + WS_TMO)) == call || rt() - t0 > DEADLINE_TICKS)) { ag_store(ws + WS_TMO, call); return false; }
    __builtin_amdgcn_s_sleep(NAP);
  }
  asm volatile("" ::: "memory");
  return true;
}

__device__ __forceinline__ u32 wave_sum(u32 v) {
  v += __builtin_amdgcn_update_dpp(0u, v, 0x111, 0xf, 0xf, true);
  v += __builtin_amdgcn_update_dpp(0u, v, 0x112, 0xf, 0xf, true);
  v += __builtin_amdgcn_update_dpp(0u, v, 0x114, 0xf, 0xf, true);
  v += __builtin_amdgcn_update_dpp(0u, v, 0x118, 0xf, 0xf, true);
  return __builtin_amdgcn_readlane(v, 15) + __builtin_amdgcn_readlane(v, 31) +
         __builtin_amdgcn_readlane(v, 47) + __builtin_amdgcn_readlane(v, 63);
}

__device__ __forceinline__ void read_bfrag(i32x8& r, u32 addr, int imm) {
  LAS unsigned char* b = (LAS unsigned char*)(size_t)addr;
  i32x2 r0 = __builtin_amdgcn_ds_read_tr8_b64_v2i32((LAS i32x2*)(b + imm));
  i32x2 r1 = __builtin_amdgcn_ds_read_tr8_b64_v2i32((LAS i32x2*)(b + imm + 256));
  i32x2 r2 = __builtin_amdgcn_ds_read_tr8_b64_v2i32((LAS i32x2*)(b + imm + 512));
  i32x2 r3 = __builtin_amdgcn_ds_read_tr8_b64_v2i32((LAS i32x2*)(b + imm + 768));
  r[0] = r0.x; r[1] = r0.y; r[2] = r1.x; r[3] = r1.y; r[4] = r2.x; r[5] = r2.y; r[6] = r3.x; r[7] = r3.y;
}

__device__ __forceinline__ u32 miss_pattern(u32 m  , u32 cneg, u32 lo4, u32& code) {
  code = (m + cneg) & 0x1f1f1f1fu;
  return (code + lo4) ^ m;
}

__device__ __forceinline__ u32 wave_scan(u32 v) {
  v += __builtin_amdgcn_update_dpp(0u, v, 0x111, 0xf, 0xf, true);
  v += __builtin_amdgcn_update_dpp(0u, v, 0x112, 0xf, 0xf, true);
  v += __builtin_amdgcn_update_dpp(0u, v, 0x114, 0xf, 0xf, true);
  v += __builtin_amdgcn_update_dpp(0u, v, 0x118, 0xf, 0xf, true);
  v += __builtin_amdgcn_update_dpp(0u, v, 0x142, 0xa, 0xf, false);
  v += __builtin_amdgcn_update_dpp(0u, v, 0x143, 0xc, 0xf, false);
  return v;
}

__device__ __forceinline__ u32 gather16(const u32x4& X) {
  u32 f[4];
#pragma unroll
  for (int e = 0; e < 4; ++e) f[e] = (X[e] + 0x7c7c7c7cu) & 0x80808080u;
  const u32 b0 = __builtin_amdgcn_udot4(f[1], 0x80402010u, __builtin_amdgcn_udot4(f[0], 0x08040201u, 0u, false), false);
  const u32 b1 = __builtin_amdgcn_udot4(f[3], 0x80402010u, __builtin_amdgcn_udot4(f[2], 0x08040201u, 0u, false), false);
  return (b0 >> 7) | (b1 << 1);
}

template <int RECR>
__device__ __forceinline__ void write_records(u32 (&M)[2], u32 wa0, u32 wa1, LAS unsigned char* xs, LAS unsigned char* recs, u32 dummy, u32 vo) {
  u32 wa[2] = {wa0, wa1};
  do {
    u32 pos[2][RECR], x[2][RECR], ad[2][RECR];
#pragma unroll
    for (int r = 0; r < RECR; ++r)
#pragma unroll
      for (int h = 0; h < 2; ++h) {
        const u32 bit = __builtin_ctz(M[h] | 0x80000000u);
        pos[h][r] = vo + (bit & 15) + ((bit & 16) << 6);
        x[h][r] = xs[h * 2048 + pos[h][r]];
        ad[h][r] = M[h] ? wa[h] : dummy;
        wa[h] += 2;
        M[h] &= M[h] - 1;
      }
#pragma unroll
    for (int r = 0; r < RECR; ++r)
#pragma unroll
      for (int h = 0; h < 2; ++h)
        *(LAS unsigned short*)(recs + ad[h][r]) = (unsigned short)(pos[h][r] | (x[h][r] << 9));
  } while (__any((M[0] | M[1]) != 0));
}

__device__ __forceinline__ void apply_records(u32 ev, u32 first, u32 lim, u32 nhalf, LAS unsigned char* dst, int lane) {
#pragma unroll
  for (int h = 0; h < 2; ++h) {
    const u32 rec = h ? ev >> 16 : ev & 0xffffu;
    const u32 idx = first + 2 * lane + h;
    const u32 off = (rec & 0x7ffu) | (idx >= nhalf ? 0x800u : 0u);
    if (idx < lim)
      __hip_atomic_fetch_xor((LAS u32*)(dst + (off & ~3u)), ((rec >> 11) << 2) << (8 * (off & 3)), __ATOMIC_RELAXED, __HIP_MEMORY_SCOPE_WORKGROUP);
  }
}

__device__ __forceinline__ int choose_window(const u32x4 (&V)[4]) {
  u32 sum = 0;
#pragma unroll
  for (int t = 0; t < 4; ++t) sum = __builtin_amdgcn_sad_u8(V[t][3 - t] & 0x7c7c7c7cu, 0u, sum);
  int lo = (int)((wave_sum(sum) + 1638u) >> 12) - 6;
  lo = lo < 0 ? 0 : (lo > 22 ? 22 : lo);
  u32 cnts = 0;
#pragma unroll
  for (int k = 0; k < 3; ++k) {
    const u32 lo4 = (u32)(lo + k) * 0x04040404u, cneg = (u32)((-4 * (lo + k)) & 31) * 0x01010101u;
    u32 n = 0, code;
#pragma unroll
    for (int t = 0; t < 4; ++t)
      n += __builtin_popcount((miss_pattern(V[t][t] & 0x7f7f7f7fu, cneg, lo4, code) + 0x7c7c7c7cu) & 0x80808080u);
    cnts |= min(n, 15u) << (10 * k);
  }
  const u32 tot = wave_sum(cnts);
  const u32 n0 = tot & 1023, n1 = (tot >> 10) & 1023, n2 = tot >> 20;
  return lo + ((n1 <= n0 && n1 <= n2) ? 1 : (n0 <= n2 ? 0 : 2));
}

__device__ __forceinline__ u32 encode_piece(const u32x4 (&V)[4], int lo, u32x4 (&D)[4], u32x4& T, LAS unsigned char* scr, int lane) {
  const u32 lo4 = (u32)lo * 0x04040404u, cneg = (u32)((-4 * lo) & 31) * 0x01010101u;
  const u32 vo = lane * 16;
  u32x4 C[4];
  u32 M[2];
#pragma unroll
  for (int t = 0; t < 4; ++t) {
    u32x4 X;
#pragma unroll
    for (int e = 0; e < 4; ++e) {
      u32 code;
      X[e] = miss_pattern(V[t][e] & 0x7f7f7f7fu, cneg, lo4, code);
      C[t][e] = code | (V[t][e] & 0x80808080u);
    }
    *(LAS u32x4*)(scr + t * 1024 + vo) = X;
    const u32 m16 = gather16(X);
    if (t & 1) M[t >> 1] |= m16 << 16; else M[t >> 1] = m16;
  }
  const u32 n = __builtin_popcount(M[0]) | (__builtin_popcount(M[1]) << 16);
  const u32 inc = wave_scan(n);
  const u32 tot = __builtin_amdgcn_readlane(inc, 63);
  const u32 nhalf = tot & 0xffffu, nexc = nhalf + (tot >> 16);
  LAS unsigned char* recs = scr + 4096;
  if (nexc > MAXEXC) {
    D[0] = V[0]; D[1] = V[1]; D[2] = V[2]; T = V[3];
    D[3] = u32x4{0x80000000u, 0u, 0u, 0u};
    return nexc;
  }
  if (lane == 0) *(LAS u32*)recs = nhalf | (nexc << 10) | ((u32)lo << 20);
  write_records<3>(M, 4 + 2 * ((inc - n) & 0xffffu), 4 + 2 * (nhalf + ((inc - n) >> 16)), scr, recs, 1024 + 2 * lane, vo);
#pragma unroll
  for (int e = 0; e < 4; ++e) {
    const u32 c3 = C[3][e];
    D[0][e] = C[0][e] | ((c3 << 5) & 0x60606060u);
    D[1][e] = C[1][e] | ((c3 << 3) & 0x60606060u);
    D[2][e] = C[2][e] | ((c3 << 1) & 0x20202020u) | ((c3 >> 1) & 0x40404040u);
  }
  D[3] = *(LAS u32x4*)(recs + vo);
  return nexc;
}

__device__ __forceinline__ void unpack_piece(const unsigned char* src, LAS unsigned char* dst, int lane) {
  LAS u32x4* o = (LAS u32x4*)(dst + lane * 16);
  u32 ev = *(LAS u32*)(dst + 3072 + lane * 4);
  u32x4 D[3] = {o[0], o[64], o[128]};
  asm volatile("" : "+v"(ev), "+v"(D[0]), "+v"(D[1]), "+v"(D[2]));
  const u32 hdr = __builtin_amdgcn_readfirstlane(ev);
  if (hdr >> 31) {
    u32x4 D3;
    asm volatile("global_load_dwordx4 %0, %1, %2 sc0 sc1\n\ts_waitcnt vmcnt(0)" : "=&v"(D3) : "v"(lane * 16), "s"(src + 4096) : "memory");
    o[192] = D3;
    return;
  }
  const u32 lo4 = ((hdr >> 20) & 31) * 0x04040404u, nhalf = hdr & 1023, nexc = (hdr >> 10) & 1023;
  u32x4 O0, O1, O2, O3;
#pragma unroll
  for (int e = 0; e < 4; ++e) {
    const u32 d0 = D[0][e], d1 = D[1][e], d2 = D[2][e];
    u32 s0 = d0 & 0x60606060u, s1 = d1 & 0x60606060u, s2 = d2 & 0x60606060u;
    asm("" : "+v"(s0), "+v"(s1), "+v"(s2));
    O0[e] = (d0 ^ s0) + lo4; O1[e] = (d1 ^ s1) + lo4; O2[e] = (d2 ^ s2) + lo4;
    const u32 t = (s0 >> 5) | (s1 >> 3) | (s2 >> 1), z = t & 0x20202020u;
    O3[e] = t + lo4 + 3 * z;
  }
  o[0] = O0; o[64] = O1; o[128] = O2; o[192] = O3;
  for (u32 base = 0;;) {
    apply_records(ev, base - 2, nexc, nhalf, dst, lane);
    base += 128;
    if (base >= nexc + 2) break;
    asm volatile("global_load_dword %0, %1, %2 offset:3072 sc0 sc1\n\ts_waitcnt vmcnt(0)" : "=&v"(ev) : "v"(lane * 4 + 2 * base), "s"(src) : "memory");
  }
}

constexpr int B_HDR = 2560, B_STR = 2576;
__device__ __forceinline__ u64 tier3_exponents(int b) {
  if (b >= 13) return (u64)(4 * (b - 3)) * 0x0101010101010101ull - 0x1c1814100c080400ull;
  u64 r = 0;
#pragma unroll
  for (int k = 0; k < 8; ++k) r |= (u64)(4 * max(b - 3 - k, 3)) << (8 * k);
  return r;
}
__device__ __forceinline__ u32 tier2_exponents(int b) {
  return (u32)(4 * max(b - 1, 3)) | (u32)(4 * max(b - 2, 3)) << 8 | (u32)(4 * (b + 3)) << 16;
}
__device__ __forceinline__ int choose_base(const u32x4 (&V)[4]) {
  u32 sum = 0;
#pragma unroll
  for (int t = 0; t < 4; ++t) sum = __builtin_amdgcn_sad_u8(V[t][3 - t] & 0x7c7c7c7cu, 0u, sum);
  int b = (int)((wave_sum(sum) + 1638u) >> 12) - 2;
  b = b < 0 ? 0 : (b > 26 ? 26 : b);
  u32 cnts = 0;
#pragma unroll
  for (int k = 0; k < 3; ++k) {
    const u32 c0 = (u32)(0x80 - 4 * (b + k)) * 0x01010101u;
    u32 n = 0;
#pragma unroll
    for (int t = 0; t < 4; ++t) {
      const u32 q = (V[t][t] & 0x7f7f7f7fu) + c0;
      n += __builtin_popcount(q & ~(q - 0x0c0c0c0cu) & 0x80808080u);
    }
    cnts |= min(n, 15u) << (10 * k);
  }
  const u32 tot = wave_sum(cnts);
  const u32 n0 = tot & 1023, n1 = (tot >> 10) & 1023, n2 = tot >> 20;
  return b + ((n1 >= n0 && n1 >= n2) ? 1 : (n0 >= n2 ? 0 : 2));
}

__device__ __forceinline__ void or_bits(LAS unsigned char* base, u32 bp, u64 a) {
  LAS u32* w = (LAS u32*)(base + ((bp >> 5) << 2));
  const u32 sh = bp & 31;
  const u64 x = a << sh;
  __hip_atomic_fetch_or(w, (u32)x, __ATOMIC_RELAXED, __HIP_MEMORY_SCOPE_WORKGROUP);
  __hip_atomic_fetch_or(w + 1, (u32)(x >> 32), __ATOMIC_RELAXED, __HIP_MEMORY_SCOPE_WORKGROUP);
  __hip_atomic_fetch_or(w + 2, ((u32)(a >> 32) >> 1) >> (31 - sh), __ATOMIC_RELAXED, __HIP_MEMORY_SCOPE_WORKGROUP);
}

__device__ __forceinline__ u32 encode_b(const u32x4 (&V)[4], int b, u32x4 (&N)[2], LAS unsigned char* scr, LAS unsigned char* lut, int lane) {
  const u32 vo = lane * 16;
  LAS unsigned char* tail = scr + 4096;
  *(LAS u32x4*)(tail + vo) = u32x4{0u, 0u, 0u, 0u};
  *(LAS u32x4*)(tail + 1024 + vo) = u32x4{0u, 0u, 0u, 0u};
  const u32 c1 = (u32)(0x83 + 4 * (b + 3)) * 0x01010101u;
  const u64 ex = tier3_exponents(b);
  const u32 exlo = (u32)ex, exhi = (u32)(ex >> 32), ex2 = tier2_exponents(b);
  const u32 dlo = (u32)(4 * (b + 3)) * 0x01010101u - 0x0c080400u, dhi = ex2 & 0xffffu;
  u32 M[2], S[2] = {0u, 0u}, o2[2], o3[4];
  u64 a2[2], a3[4];
  u32 k2[2][4], k3[2][4], e1[2][4], e3[2][4], w2[2][4];
  i32x2 wm[2][4];
#define BYR(hi, lo) __builtin_amdgcn_perm(hi, lo, r)
#pragma unroll
  for (int t = 0; t < 5; ++t) {
    if (t < 4) {
      u32x4 X;
      u32 n1[4];
      const int s = t & 1;
#pragma unroll
      for (int e = 0; e < 4; ++e) {
        const u32 v = V[t][e], m = v & 0x7f7f7f7fu;
        const u32 q = c1 - m;
        const u32 H = ((((q ^ 0x80808080u) & 0xe0e0e0e0u) >> 1) + 0x70707070u) & 0x80808080u;
        const u32 r = ((q >> 2) | ((H >> 4) - (H >> 7))) & 0x07070707u;
        n1[e] = BYR(0x0c0c0c0cu, 0x0004080cu) | (m & 0x03030303u);
        e1[s][e] = BYR(0x01010101u, 0x00000001u);
        e3[s][e] = BYR(0x01010000u, 0u);
        k2[s][e] = BYR(0x03030100u, 0x00000002u);
        k3[s][e] = ((q - 0x18181818u) >> 2) & BYR(0x07070000u, 0u);
        X[e] = ((BYR(dhi, dlo) | (__builtin_amdgcn_perm(exhi, exlo, k3[s][e]) & BYR(0x7c7c0000u, 0u))) ^ m) & 0x7c7c7c7cu;
        S[t >> 1] |= (v >> (4 * s + e)) & (0x80808080u >> (4 * s + e));
        w2[s][e] = *(LAS u32*)(lut + 8 + __builtin_amdgcn_udot4(e1[s][e], 0x80402010u, 0u, false));
        wm[s][e] = *(LAS i32x2*)(lut + __builtin_amdgcn_udot4(e3[s][e], 0x80402010u, 0u, false));
      }
      *(LAS u32x4*)(scr + t * 1024 + vo) = X;
      const u32 m16 = gather16(X);
      if (s) M[t >> 1] |= m16 << 16; else M[t >> 1] = m16;
      N[t >> 1][2 * s] = n1[0] | (n1[1] << 4);
      N[t >> 1][2 * s + 1] = n1[2] | (n1[3] << 4);
      asm volatile("" : "+v"(N[t >> 1][2 * s]), "+v"(N[t >> 1][2 * s + 1]), "+v"(S[t >> 1]), "+v"(M[t >> 1]));
    }
    if (t > 0) {
      const int p = t - 1, s = p & 1;
      u32 ob2 = 0, ob3 = 0, s2 = 0;
      u64 s3 = 0;
#pragma unroll
      for (int e = 0; e < 4; ++e) {
        s2 |= __builtin_amdgcn_udot4(k2[s][e], w2[s][e], 0u, false) << ob2;
        ob2 = __builtin_amdgcn_udot4(e1[s][e], 0x02020202u, ob2, false);
        s3 |= (u64)__builtin_amdgcn_udot4(k3[s][e], (u32)wm[s][e].x, (k3[s][e] >> 15) & (u32)wm[s][e].y, false) << ob3;
        ob3 = __builtin_amdgcn_udot4(e3[s][e], 0x03030303u, ob3, false);
      }
      if (s) { a2[p >> 1] |= (u64)s2 << o2[p >> 1]; o2[p >> 1] += ob2; } else { a2[p >> 1] = s2; o2[p >> 1] = ob2; }
      a3[p] = s3; o3[p] = ob3;
      asm volatile("" : "+v"(a2[p >> 1]), "+v"(a3[p]));
    }
  }
#undef BYR
  const u32 nb = (o2[0] + o2[1]) | (o3[0] + o3[1] + o3[2] + o3[3]) << 16;
  const u32 nr = __builtin_popcount(M[0]) | (__builtin_popcount(M[1]) << 16);
  const u32 ib = wave_scan(nb), ir = wave_scan(nr);
  const u32 totb = __builtin_amdgcn_readlane(ib, 63), totr = __builtin_amdgcn_readlane(ir, 63);
  const u32 bits2 = totb & 0xffffu, bits3 = totb >> 16;
  const u32 nhalf = totr & 0xffffu, nexc = nhalf + (totr >> 16);
  const u32 st3 = B_STR + ((bits2 + 31) >> 5) * 4, rec0 = st3 + ((bits3 + 31) >> 5) * 4;
  const u32 len = rec0 + 2 * nexc;
  if (bits2 > 2 * 2040 || bits3 > 3 * 512 || nexc > MAXEXC || len > 4032) return 0;
  *(LAS i32x2*)(tail + lane * 8) = i32x2{(int)S[0], (int)S[1]};
  if (lane == 0) *(LAS u32x4*)(tail + B_HDR - 2048) = u32x4{nhalf | (nexc << 10) | ((u32)b << 20), bits2 | (st3 << 16), len, bits3 | (rec0 << 16)};
  {
    u32 bp = ((ib - nb) & 0xffffu) + (B_STR - 2048) * 8;
#pragma unroll
    for (int h = 0; h < 2; ++h) {
      LAS u32* w = (LAS u32*)(tail + ((bp >> 5) << 2));
      const u32 sh = bp & 31;
      const u64 x = a2[h] << sh;
      __hip_atomic_fetch_or(w, (u32)x, __ATOMIC_RELAXED, __HIP_MEMORY_SCOPE_WORKGROUP);
      __hip_atomic_fetch_or(w + 1, (u32)(x >> 32), __ATOMIC_RELAXED, __HIP_MEMORY_SCOPE_WORKGROUP);
      __hip_atomic_fetch_or(w + 2, ((u32)(a2[h] >> 32) >> 1) >> (31 - sh), __ATOMIC_RELAXED, __HIP_MEMORY_SCOPE_WORKGROUP);
      bp += o2[h];
    }
    bp = ((ib - nb) >> 16) + (st3 - 2048) * 8;
#pragma unroll
    for (int t = 0; t < 4; ++t) { or_bits(tail, bp, a3[t]); bp += o3[t]; }
  }
  write_records<1>(M, rec0 - 2048 + 2 * ((ir - nr) & 0xffffu), rec0 - 2048 + 2 * (nhalf + ((ir - nr) >> 16)), scr, tail, 2048 + 2 * lane, vo);
  return len;
}

__device__ __forceinline__ void unpack_b(const unsigned char* src, LAS unsigned char* dst, LAS unsigned char* lut, int lane) {
  LAS u32x4* o = (LAS u32x4*)(dst + lane * 16);
  u32x4 N[2] = {o[0], o[64]};
  i32x2 S = *(LAS i32x2*)(dst + 2048 + lane * 8);
  u32x4 H = *(LAS u32x4*)(dst + B_HDR);
  u32 c0 = *(LAS u32*)(dst + B_STR + lane * 4);
  asm volatile("" : "+v"(N[0]), "+v"(N[1]), "+v"(S), "+v"(H), "+v"(c0));
  const u32 hdr = __builtin_amdgcn_readfirstlane(H.x), h1 = __builtin_amdgcn_readfirstlane(H.y), h3 = __builtin_amdgcn_readfirstlane(H.w);
  if (hdr >> 31) {
    u32x4 D3, D4;
    asm volatile("global_load_dwordx4 %0, %2, %3 offset:3072 sc0 sc1\n\tglobal_load_dwordx4 %1, %4, %3 sc0 sc1\n\ts_waitcnt vmcnt(0)"
                 : "=&v"(D3), "=&v"(D4) : "v"(lane * 16), "s"(src), "v"(4096u) : "memory");
    o[192] = D3;
    *(LAS u32x4*)(dst + B_HDR) = D4;
    return;
  }
  const int b = (hdr >> 20) & 31;
  const u32 nhalf = hdr & 1023, nexc = (hdr >> 10) & 1023, bits2 = h1 & 0xffffu, st3 = h1 >> 16, bits3 = h3 & 0xffffu, rec0 = h3 >> 16;
  if (rec0 + 2 * nexc > 3072) {
    u32x4 D3;
    asm volatile("global_load_dwordx4 %0, %1, %2 offset:3072 sc0 sc1\n\ts_waitcnt vmcnt(0)" : "=&v"(D3) : "v"(lane * 16), "s"(src) : "memory");
    o[192] = D3;
  }
  u32 ev = *(LAS u32*)(dst + min(rec0 + lane * 4, 4092u));
  const u32 so = st3 + (((3 * lane) >> 1) & ~3);
  const u32 t0 = *(LAS u32*)(dst + so), t1 = *(LAS u32*)(dst + so + 4);
  u32 c1 = 0, t2 = 0, t3 = 0;
  if (bits2 > 2048) c1 = *(LAS u32*)(dst + B_STR + 256 + lane * 4);
  if (bits3 > 768) { t2 = *(LAS u32*)(dst + so + 96); t3 = *(LAS u32*)(dst + so + 100); }
#define WINDOW(a) (*(LAS i32x2*)__builtin_assume_aligned((LAS void*)(size_t)(((a) + ((a) & 4u) * 511u) & ~3u), 8))
  {
    const u64 ex = tier3_exponents(b) - 0x0c0c0c0c0c0c0c0cull;
    auto expand3 = [&](u32 a0, u32 a1, int p) {
      const u32 g = __builtin_amdgcn_alignbit(a1, a0, 12 * lane) & 0xfffu;
      const u32 z = (g | (g << 10)) & 0x003f003fu;
      const u32 r = __builtin_amdgcn_perm((u32)(ex >> 32), (u32)ex, (z | (z << 5)) & 0x07070707u);
      *(LAS u32*)(dst + p * 256 + lane * 4) = r;
      *(LAS u32*)(dst + 2044 + p * 256 + lane * 4) = r;
    };
    expand3(t0, t1, 0);
    if (bits3 > 768) expand3(t2, t3, 1);
  }
  {
    const u32 ex2 = (tier2_exponents(b) - 0x0c0c0cu) & 0xffffffu;
    const u32 n3 = __builtin_popcount(c0 & (c0 >> 1) & 0x55555555u) | __builtin_popcount(c1 & (c1 >> 1) & 0x55555555u) << 16;
    const u32 i3 = wave_scan(n3);
    u32x4 r[2];
    auto expand2 = [&](u32 cw, u32 wa, u32x4& out) {
      u32 y[4], sl[4], ad[4];
      i32x2 w[4];
#pragma unroll
      for (int i = 0; i < 4; ++i) {
        const u32 g = (cw >> (8 * i)) & 0xffu;
        const u32 z = (g | (g << 12)) & 0x000f000fu;
        y[i] = (z | (z << 6)) & 0x03030303u;
        const u32 f = y[i] & (y[i] >> 1);
        sl[i] = *(LAS u32*)(lut + __builtin_amdgcn_udot4(f, 0x20100804u, 0u, false));
        w[i] = WINDOW(wa);
        ad[i] = wa;
        wa += __builtin_popcount(f);
      }
      asm volatile("" : "+v"(w[0]), "+v"(w[1]), "+v"(w[2]), "+v"(w[3]), "+v"(sl[0]), "+v"(sl[1]), "+v"(sl[2]), "+v"(sl[3]));
#pragma unroll
      for (int i = 0; i < 4; ++i)
        out[i] = __builtin_amdgcn_perm(0u, ex2, y[i]) | __builtin_amdgcn_perm(0u, __builtin_amdgcn_alignbyte((u32)w[i].y, (u32)w[i].x, ad[i]), sl[i]);
    };
    expand2(c0, (u32)(size_t)dst + ((i3 - n3) & 0xffffu), r[0]);
    if (bits2 > 2048) expand2(c1, (u32)(size_t)dst + (__builtin_amdgcn_readlane(i3, 63) & 0xffffu) + ((i3 - n3) >> 16), r[1]);
#pragma unroll
    for (int p = 0; p < 2; ++p)
      if (p == 0 || bits2 > 2048) {
        o[p * 64] = r[p];
        o[128 + p * 64] = u32x4{r[p][1], r[p][2], r[p][3], (u32)__builtin_amdgcn_update_dpp(0u, r[p][0], 0x130, 0xf, 0xf, true)};
        if (p && lane == 0) *(LAS u32*)(dst + 2044 + 1024) = r[p][0];
      }
  }
  u32 G[8], n = 0;
#pragma unroll
  for (int i = 0; i < 8; ++i) {
    const u32 x = N[i >> 2][i & 3];
    G[i] = x & (x >> 1) & 0x44444444u;
    n += __builtin_popcount(G[i]);
  }
  u32 wa = (u32)(size_t)dst + wave_scan(n) - n;
  const u32 b4 = (u32)(4 * b) * 0x01010101u;
  u32 wad[16], sel[16];
  i32x2 W[16];
#pragma unroll
  for (int j = 0; j < 16; ++j) {
    const u32 f = ((j & 1) ? G[j >> 1] >> 4 : G[j >> 1]) & 0x04040404u;
    sel[j] = *(LAS u32*)(lut + __builtin_amdgcn_udot4(f, 0x08040201u, 0u, false));
    W[j] = WINDOW(wa);
    wad[j] = wa;
    wa += __builtin_popcount(f);
  }
#undef WINDOW
  asm volatile("" : "+v"(W[0]), "+v"(W[1]), "+v"(W[2]), "+v"(W[3]), "+v"(W[4]), "+v"(W[5]), "+v"(W[6]), "+v"(W[7]),
               "+v"(sel[0]), "+v"(sel[1]), "+v"(sel[2]), "+v"(sel[3]), "+v"(sel[4]), "+v"(sel[5]), "+v"(sel[6]), "+v"(sel[7]));
  asm volatile("" : "+v"(W[8]), "+v"(W[9]), "+v"(W[10]), "+v"(W[11]), "+v"(W[12]), "+v"(W[13]), "+v"(W[14]), "+v"(W[15]),
               "+v"(sel[8]), "+v"(sel[9]), "+v"(sel[10]), "+v"(sel[11]), "+v"(sel[12]), "+v"(sel[13]), "+v"(sel[14]), "+v"(sel[15]));
#pragma unroll
  for (int t = 0; t < 4; ++t) {
    u32x4 O;
#pragma unroll
    for (int e = 0; e < 4; ++e) {
      const int j = 4 * t + e;
      const u32 x = N[j >> 3][(j >> 1) & 3];
      const u32 nib = ((j & 1) ? x >> 4 : x) & 0x0f0f0f0fu;
      const u32 sg = ((u32)S[t >> 1] << (j & 7)) & 0x80808080u;
      O[e] = (nib | sg) + __builtin_amdgcn_perm(b4, __builtin_amdgcn_alignbyte((u32)W[j].y, (u32)W[j].x, wad[j]), sel[j]);
    }
    o[t * 64] = O;
  }
  for (u32 base = 0;;) {
    apply_records(ev, base, nexc, nhalf, dst, lane);
    base += 128;
    if (base >= nexc) break;
    asm volatile("global_load_dword %0, %1, %2 sc0 sc1\n\ts_waitcnt vmcnt(0)" : "=&v"(ev) : "v"(rec0 + lane * 4 + 2 * base), "s"(src) : "memory");
  }
}

__device__ __forceinline__ void init_luts(LAS unsigned char* lut, int lane) {
  if (lane < 16) {
    u32 w3 = 0, w2 = 0, sel = 0, r = 0;
#pragma unroll
    for (int i = 0; i < 4; ++i) {
      const bool on = (lane >> i) & 1;
      w3 |= (on && r < 3 ? 1u << (3 * r) : 0u) << (8 * i);
      w2 |= (on ? 1u << (2 * r) : 0u) << (8 * i);
      sel |= (on ? r : 4u) << (8 * i);
      r += on;
    }
    *(LAS u32x4*)(lut + lane * 16) = u32x4{w3, lane == 15 ? 0xe00u : 0u, w2, 0u};
    *(LAS u32*)(lut + 256 + lane * 4) = sel;
  }
}

__global__ void __launch_bounds__(NTHREADS, 3) __attribute__((amdgpu_num_vgpr(128))) fused_kernel(Params P) {
  extern __shared__ __attribute__((aligned(16))) unsigned char lds_raw[];
  LAS unsigned char* L = (LAS unsigned char*)lds_raw;
  volatile LAS u32* READY = (volatile LAS u32*)(L + LDS_READY);
  volatile LAS u32* DONE = (volatile LAS u32*)(L + LDS_DONE);
  volatile LAS u32* BAD = (volatile LAS u32*)(L + LDS_BAD);
  volatile LAS u32* GO = (volatile LAS u32*)(L + LDS_GO);

  const int c = blockIdx.x;
  const int lane = threadIdx.x & 63;
  const int wave = __builtin_amdgcn_readfirstlane(threadIdx.x >> 6);
  const int u = (c & 7) * 8 + ((c >> 3) & 7);
  const int mt = c >> 6;
  const int pr = ((c >> 3) & 7) * 32 + mt * 8 + (c & 7);
  unsigned char* const ws = P.peer[0];
  STAMP(0);

  unsigned char* cntp = ws + WS_CNT + c * 128 + wave * 4;
#define START_BARRIER() do { __builtin_amdgcn_sched_barrier(0); __builtin_amdgcn_s_barrier(); __builtin_amdgcn_sched_barrier(0); } while (0)
  const u32 w_lane = (u32)(lane >> 1) * NN + (lane & 1) * 16;
  const u32 vo = lane * 16;

  if (wave == NWAVES - 1) {
    __builtin_amdgcn_s_waitcnt(0x0F70);
    {
      const int d = pr < 224 ? 1 + pr % 7 : 0, j = pr < 224 ? pr / 7 : pr - 224;
      const unsigned char* xs = P.x + (size_t)(((P.rank + d) & 7) * MS + (j >> 1) * 32) * KS + (j & 1) * 256;
      const u32 xo = (u32)(lane & 31) * KS + 32 * (lane >> 5);
#define ACLOB "v128", "v129", "v130", "v131", "v132", "v133", "v134", "v135", "v136", "v137", "v138", "v139", "v140", "v141", "v142", "v143", "v144", "v145", "v146", "v147", \
              "v148", "v149", "v150", "v151", "v152", "v153", "v154", "v155", "v156", "v157", "v158", "v159", "v160", "v161", "memory"
      asm volatile("global_load_dword v160, %0, %1 sc1\n\t"
                   "global_load_dwordx4 v[128:131], %2, %3\n\tglobal_load_dwordx4 v[132:135], %2, %3 offset:16\n\t"
                   "global_load_dwordx4 v[136:139], %2, %3 offset:64\n\tglobal_load_dwordx4 v[140:143], %2, %3 offset:80\n\t"
                   "global_load_dwordx4 v[144:147], %2, %3 offset:128\n\tglobal_load_dwordx4 v[148:151], %2, %3 offset:144\n\t"
                   "global_load_dwordx4 v[152:155], %2, %3 offset:192\n\tglobal_load_dwordx4 v[156:159], %2, %3 offset:208\n\t"
                   "v_mov_b32 v161, 1"
                   :: "v"(0u), "s"(cntp), "v"(xo), "s"(xs) : ACLOB);
      START_BARRIER();
      __builtin_amdgcn_s_setprio(2);
      u32 cnt;
      asm volatile("s_waitcnt vmcnt(8)\n\tv_mov_b32 %0, v160" : "=v"(cnt) :: "memory");
      const u32 call = __builtin_amdgcn_readfirstlane(cnt) + 1;
      unsigned char* pd = P.peer[d];
      unsigned char* dst = pd + WS_XB + (call & 1) * XB_PAR + (size_t)(d * 32 + j) * 8192;
#define XST(k, r) asm volatile("s_waitcnt vmcnt(%2)\n\tglobal_store_dwordx4 %0, " r ", %1 offset:%3 sc0 sc1\n\ts_nop 1" \
                               :: "v"(vo), "s"(dst + ((k) >> 2) * 4096), "i"((k) < XW ? 7 : XW - 1), "i"(((k) & 3) * 1024) : "memory")
#define XSTS(a, b) do { if (a <= 0 && 0 < b) XST(0, "v[128:131]"); if (a <= 1 && 1 < b) XST(1, "v[132:135]"); if (a <= 2 && 2 < b) XST(2, "v[136:139]"); \
        if (a <= 3 && 3 < b) XST(3, "v[140:143]"); if (a <= 4 && 4 < b) XST(4, "v[144:147]"); if (a <= 5 && 5 < b) XST(5, "v[148:151]"); \
        if (a <= 6 && 6 < b) XST(6, "v[152:155]"); if (a <= 7 && 7 < b) XST(7, "v[156:159]"); } while (0)
      XSTS(0, XW);
      STAMP(1);
      XSTS(XW, 8);
      GO[0] = 1u;
#undef XSTS
#undef XST
      if (lane == 0) ag_store(cntp, call);
      VMWAIT(0);
      if (lane < 16) flag_store(pd + WS_FX + (d * 32 + j) * 64 + lane * 4, call);
      STAMP(8);
      if (d != 0 && j < NPW) {
        const unsigned char* tkp = ws + WS_TK + d * 128;
        const unsigned char* const rsrc = P.peer[R - d] + WS_ST + (call & 1) * WB_SRC;
        unsigned char* const ldst = ws + WS_WB + (call & 1) * WB_PAR + d * WB_SRC;
        unsigned char* const lfw = ws + WS_FW + d * 64 * 512;
        const u32 tbase = (call - 1) * 512u - 2 * NPW;
        u32 staged = 0;
#define FETCHR(t) do { \
          while ((t) >= staged) { staged = __builtin_amdgcn_readfirstlane(__hip_atomic_load((LAS u32*)(L + LDS_STG), __ATOMIC_ACQUIRE, __HIP_MEMORY_SCOPE_WORKGROUP)); \
                                  if ((t) >= staged) __builtin_amdgcn_s_sleep(2); } \
          if (staged > 512u) { STAMP_OUT(); return; }       \
          const unsigned char* s_ = rsrc + (size_t)(t) * SLOT; \
          got = FMT_B(t) ? wantb : max(wanta, (t) < U0 ? 3200u : 3328u); \
          asm volatile("global_load_dwordx4 v[128:131], %0, %1 sc0 sc1\n\tglobal_load_dwordx4 v[132:135], %0, %1 offset:1024 sc0 sc1" \
                       :: "v"(vo), "s"(s_) : ACLOB); \
          if (2048 + vo < got) asm volatile("global_load_dwordx4 v[136:139], %0, %1 offset:2048 sc0 sc1" :: "v"(vo), "s"(s_) : ACLOB); \
          if (3072 + vo < got) asm volatile("global_load_dwordx4 v[140:143], %0, %1 offset:3072 sc0 sc1" :: "v"(vo), "s"(s_) : ACLOB); \
          if (4096 + vo < got) asm volatile("global_load_dwordx4 v[144:147], %0, %1 sc0 sc1" :: "v"(vo), "s"(s_ + 4096) : ACLOB); } while (0)
        u32 wantb = 2944, wanta = 0, got;
        u32 cur = j;
        asm volatile("v_mov_b32 v160, %0" :: "v"(tbase + NPW + j) : ACLOB);
        FETCHR(cur);
#pragma unroll 1
        while (true) {
          u32 tk, hb, ha;
          asm volatile("s_waitcnt vmcnt(0)\n\tv_readfirstlane_b32 %0, v160\n\tv_readlane_b32 %1, v138, 32\n\tv_readfirstlane_b32 %2, v140"
                       : "=s"(tk), "=s"(hb), "=s"(ha) :: "memory");
          const u32 nxt = tk - tbase;
          const u32 len = FMT_B(cur) ? hb : ((ha >> 31) ? 5120u : 3076u + 2 * ((ha >> 10) & 1023));
          unsigned char* o_ = ldst + (size_t)((cur & ~63u) + UTILE(cur)) * SLOT;
          if (len > got) {
            const unsigned char* s_ = rsrc + (size_t)cur * SLOT;
            asm volatile("global_load_dwordx4 v[136:139], %0, %1 offset:2048 sc0 sc1\n\tglobal_load_dwordx4 v[140:143], %0, %1 offset:3072 sc0 sc1\n\t"
                         "global_load_dwordx4 v[144:147], %0, %2 sc0 sc1\n\ts_waitcnt vmcnt(0)" :: "v"(vo), "s"(s_), "s"(s_ + 4096) : ACLOB);
            if (FMT_B(cur)) wantb = min((len + 127u) & ~127u, 5120u); else wanta = min((len + 127u) & ~127u, 5120u);
          }
          asm volatile("global_store_dwordx4 %0, v[128:131], %1 sc1\n\tglobal_store_dwordx4 %0, v[132:135], %1 offset:1024 sc1\n\t"
                       "global_store_dwordx4 %0, v[136:139], %1 offset:2048 sc1\n\ts_nop 1" :: "v"(vo), "s"(o_) : "memory");
          if (3072 + (vo & ~127u) < len) asm volatile("global_store_dwordx4 %0, v[140:143], %1 offset:3072 sc1\n\ts_nop 1" :: "v"(vo), "s"(o_) : "memory");
          if (4096 + (vo & ~127u) < len) asm volatile("global_store_dwordx4 %0, v[144:147], %1 sc1\n\ts_nop 1" :: "v"(vo), "s"(o_ + 4096) : "memory");
          if (lane == 0) asm volatile("global_atomic_add v160, %0, v161, %1 sc0 sc1" :: "v"(0u), "s"(tkp) : ACLOB);
          if (nxt < 512u) {
            FETCHR(nxt);
            VMWAIT(4);
          } else {
            VMWAIT(0);
          }
          if (lane == 0) ag_store(lfw + UTILE(cur) * 512 + (cur >> 6) * 64, call);
          if ((cur & 32) == 0 && cur >= 64) STAMP(8 + (cur >> 6));
          if (nxt >= 512u) break;
          cur = nxt;
        }
#undef FETCHR
      }
      STAMP(3);
#undef ACLOB
    }
    STAMP_OUT();
    return;
  }
  const u64 t_start = rt();
  const u32 cnt_old = ag_load(cntp);
  if (wave == 5) asm volatile("ds_write_b32 %0, %1\n\ts_waitcnt lgkmcnt(0)" :: "v"((u32)(size_t)L + LDS_CTRL + lane * 4), "v"(0u) : "memory");
  if (wave == 6) { init_luts(L + LDS_LUT, lane); asm volatile("s_waitcnt lgkmcnt(0)" ::: "memory"); }
  u32 call;
  if (pr >= 224) {
    START_BARRIER();
    call = __builtin_amdgcn_readfirstlane(cnt_old) + 1;
    LAS unsigned char* scr = L + LDS_SCR + wave * 6400;
    if (wave < 4) __builtin_amdgcn_s_setprio(1);
    unsigned char *fl0 = P.peer[1], *fl1 = P.peer[5];
#pragma unroll
    for (int q = 1; q < 4; ++q) { fl0 = lane >> 4 == q ? P.peer[1 + q] : fl0; if (q < 3) fl1 = lane >> 4 == q ? P.peer[5 + q] : fl1; }
    fl0 += WS_FS + (1 + (lane >> 4)) * 8192 + (pr - 224) * 64 + (lane & 15) * 4;
    fl1 += WS_FS + (5 + (lane >> 4)) * 8192 + (pr - 224) * 64 + (lane & 15) * 4;
#pragma unroll 1
    for (u32 e = (pr - 224) + 32 * wave; e < 512u && wave < 8; e += 256) {
      const unsigned char* src = P.w + (size_t)(e >> 6) * 64 * NN + UTILE(e) * 64 + w_lane;
      u32x4 V[4], D[4], T;
      V[0] = *(const GAS u32x4*)(src);
      V[1] = *(const GAS u32x4*)(src + 32 * NN);
      V[2] = *(const GAS u32x4*)(src + 32);
      V[3] = *(const GAS u32x4*)(src + 32 * NN + 32);
      u32 len;
      if (!FMT_B(e)) {
        const u32 nexc = encode_piece(V, choose_window(V), D, T, scr, lane);
        len = nexc > MAXEXC ? 5120u : 3076u + 2 * nexc;
      } else {
        u32x4 N[2];
        len = encode_b(V, choose_base(V), N, scr, L + LDS_LUT, lane);
        if (len) {
          D[0] = N[0]; D[1] = N[1];
          D[2] = *(LAS u32x4*)(scr + 4096 + vo); D[3] = *(LAS u32x4*)(scr + 5120 + vo);
        } else {
          len = 4112;
          D[0] = V[0]; D[1] = V[1]; D[3] = V[3];
          D[2] = lane == 32 ? u32x4{0x80000000u, 0u, 4112u, 0u} : V[2];
#pragma unroll
          for (int q = 0; q < 4; ++q) T[q] = __builtin_amdgcn_readlane(V[2][q], 32);
        }
      }
      unsigned char* dst = ws + WS_ST + (call & 1) * WB_SRC + (size_t)e * SLOT;
      ST16(dst, vo, D[0], 0); ST16(dst, vo, D[1], 1024); ST16(dst, vo, D[2], 2048);
      if (3072 + (vo & ~63u) < len) ST16(dst, vo, D[3], 3072);
      if (4096 + (vo & ~63u) < len) { unsigned char* d2 = dst + 4096; ST16(d2, vo, T, 0); }
      if (e < 32) STAMP(6);
      VMWAIT(0);
      {
        const u32 grp = e < U0 ? 0 : 1 + min(e >> 7, 2u);
        u32 old = 0;
        if (lane == 0) old = __hip_atomic_fetch_add((LAS u32*)(L + LDS_GRP + grp * 4), 1u, __ATOMIC_RELAXED, __HIP_MEMORY_SCOPE_WORKGROUP);
        if (__builtin_amdgcn_readfirstlane(old) == (grp == 0 ? 0 : grp == 1 ? 2 : grp == 2 ? 3 : 7)) {
          flag_store(fl0 + grp * 2048, call);
          if (lane < 48) flag_store(fl1 + grp * 2048, call);
        }
      }
      STAMP(5);
      __builtin_amdgcn_s_setprio(0);
    }
    if (lane == 0) __hip_atomic_fetch_add((LAS u32*)(L + LDS_ENCD), 1u, __ATOMIC_RELAXED, __HIP_MEMORY_SCOPE_WORKGROUP);
    while (*(volatile LAS u32*)(L + LDS_ENCD) < 11u) __builtin_amdgcn_s_sleep(2);
    asm volatile("" ::: "memory");
  } else {
    START_BARRIER();
    call = __builtin_amdgcn_readfirstlane(cnt_old) + 1;
  }
  if (lane == 0) ag_store(cntp, call);
  if (wave == 3 && pr < 7 * NPW) {
    while (rt() - t_start < TPOLL) __builtin_amdgcn_s_sleep(8);
    const unsigned char* fs = ws + WS_FS + (1 + pr % 7) * 8192 + (lane & 31) * 64;
    const u64 t0 = rt();
    int g = 0;
#pragma unroll 1
    for (u32 spin = 1; g < 4; ++spin) {
      const u64 m = __ballot(flag_load(fs + min(g + (lane >> 5), 3) * 2048) >= call);
      if ((u32)m == ~0u) {
        g += m == ~0ull ? 2 : 1;
        __hip_atomic_store((LAS u32*)(L + LDS_STG), g == 1 ? U0 : min(32u << g, 512u), __ATOMIC_RELEASE, __HIP_MEMORY_SCOPE_WORKGROUP);
        if (g <= 2) STAMP(g == 1 ? 4 : 1);
      } else if ((spin & 15) == 0 && (__builtin_amdgcn_readfirstlane(ag_load(ws + WS_TMO)) == call || rt() - t0 > DEADLINE_TICKS)) {
        ag_store(ws + WS_TMO, call);
        __hip_atomic_store((LAS u32*)(L + LDS_STG), 1024u, __ATOMIC_RELEASE, __HIP_MEMORY_SCOPE_WORKGROUP);
        break;
      }
      if (g < 2) __builtin_amdgcn_s_sleep(8); else __builtin_amdgcn_s_sleep(40);
    }
  }
  while (GO[0] == 0u) __builtin_amdgcn_s_sleep(4);

  if (wave < 4) {
    const int mymb = mt * 4 + wave;
    const u32 par = call & 1;
    const float scale = P.sx[0] * P.sw[0];
    const u32 b_lane = (u32)(size_t)L + ((lane >> 5) * 32 + ((lane & 15) >> 1)) * 32 + ((lane >> 4) & 1) * 16 + (lane & 1) * 8;
    f32x16 acc0 = {}, acc1 = {};
    u32 rv;
    u32x4 AL[8] = {}, AH[8] = {};
#define AWAIT() asm volatile("s_waitcnt vmcnt(0)" : "+v"(AL[0]), "+v"(AL[1]), "+v"(AL[2]), "+v"(AL[3]), "+v"(AL[4]), "+v"(AL[5]), "+v"(AL[6]), "+v"(AL[7]), \
                             "+v"(AH[0]), "+v"(AH[1]), "+v"(AH[2]), "+v"(AH[3]), "+v"(AH[4]), "+v"(AH[5]), "+v"(AH[6]), "+v"(AH[7]) :: "memory")
#define MMA(p, off) do { \
          i32x8 b0, b1; read_bfrag(b0, baddr, off); read_bfrag(b1, baddr, off + 2048); \
          const i32x8 a = {(int)AL[p].x, (int)AL[p].y, (int)AL[p].z, (int)AL[p].w, (int)AH[p].x, (int)AH[p].y, (int)AH[p].z, (int)AH[p].w}; \
          __builtin_amdgcn_sched_barrier(0); \
          acc0 = __builtin_amdgcn_mfma_scale_f32_32x32x64_f8f6f4(b0, a, acc0, 1, 0, 0, 0, 0, 0); \
          acc1 = __builtin_amdgcn_mfma_scale_f32_32x32x64_f8f6f4(b1, a, acc1, 1, 0, 0, 0, 0, 0); } while (0)
#define POLL(row, want) do { while (true) { \
          rv = READY[(row) * 8 + (lane & 7)]; \
          m = (u32)__ballot(rv >= (u32)(want)) & rem; \
          if (m) break; \
          __builtin_amdgcn_s_sleep(1); } \
        asm volatile("" ::: "memory"); rem &= ~m; } while (0)
    const unsigned char* xb = ws + WS_XB + par * XB_PAR + (size_t)mymb * 16384;
    {
      if (!wait_all<true>(ws + WS_FX + mymb * 128 + (lane & 1) * 64, call, ws, call)) BAD[0] = 1u;
#define LA(p) do { const unsigned char* s_ = xb + p * 2048; LD16S(AL[p], s_, vo, 0); LD16S(AH[p], s_, vo, 1024); } while (0)
      LA(0); LA(1); LA(2); LA(3); LA(4); LA(5); LA(6); LA(7);
#undef LA
      AWAIT();
      const u32 baddr = b_lane + LDS_OWN;
      u32 rem = 0xFFu, m;
      do {
        POLL(RR, 1);
#define STEP(p) if (m & (1u << p)) MMA(p, p * PIECE);
        STEP(0) STEP(1) STEP(2) STEP(3) STEP(4) STEP(5) STEP(6) STEP(7)
#undef STEP
      } while (rem);
    }
    STAMP(7);
    if (!wait_all<true>(ws + WS_FX + ((1 + (lane & 7) % 7) * 16 + mymb) * 128 + ((lane >> 3) & 1) * 64, call, ws, call)) BAD[0] = 1u;
    STAMP(2);
#define LA(p) do { const unsigned char* s_ = xb + p * 262144; LD16S(AL[p], s_, vo, 0); LD16S(AH[p], s_, vo, 1024); } while (0)
    LA(1); LA(2); LA(3); LA(4); LA(5); LA(6); LA(7);
#pragma unroll 1
    for (int i = 0; i < 8; ++i) {
      const int row = i & (RR - 1);
      const u32 baddr = b_lane + row * 7 * PIECE;
      xb += 2048;
      AWAIT();
      u32 rem = 0xFEu, m;
      do {
        POLL(row, i + 1);
#define STEP(p) if (m & (1u << p)) { MMA(p, (p - 1) * PIECE); if (i < 7) LA(p); }
        STEP(1) STEP(2) STEP(3) STEP(4) STEP(5) STEP(6) STEP(7)
#undef STEP
      } while (rem);
      asm volatile("s_waitcnt lgkmcnt(0)" ::: "memory");
      if (lane == 0) DONE[wave] = (u32)(i + 1);
      STAMP(8 + i);
    }
#undef LA
    AWAIT();
    const bool bad = __builtin_amdgcn_readfirstlane(rv) != 0u;
    u32x4 o[8];
    const float sc = bad ? __builtin_nanf("") : scale;
    acc0 *= sc; acc1 *= sc;
#pragma unroll
    for (int q = 0; q < 4; ++q)
#pragma unroll
      for (int e = 0; e < 4; ++e) {
        o[q][e] = __builtin_bit_cast(u32, __builtin_elementwise_maximum(acc0[4 * q + e], 0.f));
        o[4 + q][e] = __builtin_bit_cast(u32, __builtin_elementwise_maximum(acc1[4 * q + e], 0.f));
      }
    const unsigned char* ob = (const unsigned char*)(P.out + (size_t)mymb * 32 * NN + u * 64);
#define OST(val, voff) asm volatile("global_store_dwordx4 %0, %1, %2 sc1\n\ts_nop 1" :: "v"(voff), "v"(val), "s"(ob) : "memory")
    {
      LAS unsigned char* tb = L + LDS_OWN + wave * 8192;
      const int m = lane & 31, h = lane >> 5;
#pragma unroll
      for (int q = 0; q < 8; ++q) *(LAS u32x4*)(tb + m * 256 + (((2 * q + h) ^ (m & 15)) << 4)) = o[q];
      const int r0 = lane >> 4, ch = lane & 15;
#pragma unroll
      for (int t = 0; t < 8; ++t) o[t] = *(LAS u32x4*)(tb + (4 * t + r0) * 256 + ((ch ^ ((4 * t + r0) & 15)) << 4));
      u32 oo = (u32)r0 * NN * 4 + ch * 16;
#pragma unroll
      for (int t = 0; t < 8; ++t) { OST(o[t], oo); oo += 4 * NN * 4; }
    }
#undef OST
    STAMP(3);
  } else {
    const int d = wave - 3;
    {
      const unsigned char* src = P.w + u * 64 + w_lane;
      for (int i = d - 1; i < 8; i += 7) {
        const unsigned char* s = src + (size_t)i * 64 * NN;
        LAS unsigned char* dst = L + LDS_OWN + i * PIECE;
        __builtin_amdgcn_global_load_lds((const GAS u32*)(s), (LAS u32*)(dst), 16, 0, 0);
        __builtin_amdgcn_global_load_lds((const GAS u32*)(s + 32 * NN), (LAS u32*)(dst + 1024), 16, 0, 0);
        __builtin_amdgcn_global_load_lds((const GAS u32*)(s + 32), (LAS u32*)(dst + 2048), 16, 0, 0);
        __builtin_amdgcn_global_load_lds((const GAS u32*)(s + 32 * NN + 32), (LAS u32*)(dst + 3072), 16, 0, 0);
      }
    }
    VMWAIT(0);
    if (lane == 0) { READY[RR * 8 + d - 1] = 1u; if (d == 1) READY[RR * 8 + 7] = 1u; }
    const unsigned char* fp = ws + WS_FW + (d * 64 + u) * 512;
    const unsigned char* src = ws + WS_WB + (call & 1) * WB_PAR + d * WB_SRC + (size_t)u * SLOT;
    u32 seen = 0;
    bool dead = false, pre = false;
    const unsigned char* tmop = ws + WS_TMO;
    const u32 l4 = lane * 4;
#define FMT_A(i_) (!FMT_B((i_) * 64 + UTILE(u)))
#define FETCH(i_, s_) do { u32 keep_; const u32 la_ = (u32)(size_t)L + ((((i_) & (RR - 1)) * 7 + d - 1) * PIECE); \
      asm volatile("s_mov_b32 %0, m0\n\ts_mov_b32 m0, %3\n\ts_nop 0\n\t" \
                   "global_load_lds_dwordx4 %1, %2 sc0 sc1\n\t" \
                   "global_load_lds_dwordx4 %1, %2 offset:1024 sc0 sc1\n\t" \
                   "global_load_lds_dwordx4 %1, %2 offset:2048 sc0 sc1\n\t" \
                   "s_mov_b32 m0, %0" : "=&s"(keep_) : "v"(vo), "s"(s_), "s"(la_) : "memory"); \
      if (FMT_A(i_)) \
        asm volatile("s_mov_b32 %0, m0\n\ts_mov_b32 m0, %3\n\ts_nop 0\n\tglobal_load_lds_dword %1, %2 offset:3072 sc0 sc1\n\ts_mov_b32 m0, %0" \
                     : "=&s"(keep_) : "v"(l4), "s"(s_), "s"(la_) : "memory"); } while (0)
#pragma unroll 1
    for (int i = 0; i < 8; ++i, src += 64 * SLOT) {
      if (!pre) {
        if (i >= RR) {
          while (!__all(DONE[lane & 3] + RR > (u32)i)) __builtin_amdgcn_s_sleep(2);
          asm volatile("" ::: "memory");
        }
        if (!((seen >> i) & 1) && !dead) {
          const u64 t0 = rt();
          for (u32 spin = 1;; ++spin) {
            seen = (u32)__ballot(flag_load(fp + max(lane & 7, i) * 64) >= call);
            if ((seen >> i) & 1) break;
            if ((spin & 63) == 0 && (__builtin_amdgcn_readfirstlane(ag_load(tmop)) == call || rt() - t0 > DEADLINE_TICKS)) { dead = true; break; }
            if (i < 7) __builtin_amdgcn_s_sleep(16);
          }
          if (dead) ag_store(ws + WS_TMO, call);
          asm volatile("" ::: "memory");
        }
        if (i == 7) STAMP(4);
        if (!dead) FETCH(i, src);
      }
      pre = !dead && i < 7 && ((seen >> (i + 1)) & 1) && __all(DONE[lane & 3] + RR > (u32)i + 1);
      if (pre) {
        const unsigned char* s2 = src + 64 * SLOT;
        FETCH(i + 1, s2);
        VMWAIT(3);
      } else {
        if (i == 7) dead = dead || __builtin_amdgcn_readfirstlane(ag_load(tmop)) == call;
        VMWAIT(0);
      }
      if (i == 7) STAMP(5);
      const int row = i & (RR - 1);
      if (dead) BAD[0] = 1u;
      else if (FMT_A(i)) unpack_piece(src, L + (row * 7 + d - 1) * PIECE, lane);
      else unpack_b(src, L + (row * 7 + d - 1) * PIECE, L + LDS_LUT + 256, lane);
      asm volatile("s_waitcnt lgkmcnt(0)" ::: "memory");
      READY[row * 8 + d] = (u32)(i + 1);
      STAMP(8 + i);
    }
#undef FETCH
#undef FMT_A
    STAMP(3);
  }
  STAMP_OUT();
}

}

extern "C" void kernel_launch(int rank, int nranks, ncclComm_t comm,
                              hipStream_t stream,
                              void* const* d_in, const int* in_sizes, int n_in,
                              void* d_out, int out_size,
                              const int* in_dtypes, int out_dtype,
                              void* const* d_out_all,
                              void* const* d_ws_all, size_t ws_bytes) {
  (void)comm; (void)in_dtypes; (void)out_dtype; (void)d_out_all;
  if (nranks != R || (unsigned)rank >= (unsigned)R || n_in != 4 || ws_bytes < WS_END ||
      in_sizes[0] != 4096 * KS || in_sizes[1] != KS * NN || out_size != MS * NN) {
    fprintf(stderr, "kernel_launch: unexpected problem shape\n");
    return;
  }
  HIP_CHECK(hipFuncSetAttribute((const void*)fused_kernel, hipFuncAttributeMaxDynamicSharedMemorySize, LDS_BYTES));
  Params P;
  P.x = (const unsigned char*)d_in[0];
  P.w = (const unsigned char*)d_in[1];
  P.sx = (const float*)d_in[2];
  P.sw = (const float*)d_in[3];
  P.out = (float*)d_out;
  for (int d = 0; d < R; ++d) P.peer[d] = (unsigned char*)d_ws_all[(rank + d) % R];
  P.rank = rank;
  P.pad = 0;
  hipLaunchKernelGGL(fused_kernel, dim3(NWG), dim3(NTHREADS), LDS_BYTES, stream, P);
  HIP_CHECK(hipGetLastError());
}
```

```cpp
#include <hip/hip_runtime.h>
#include <rccl/rccl.h>
#include <cstdio>

#define HIP_CHECK(expr) do { hipError_t _e = (expr); if (_e != hipSuccess) { \
    fprintf(stderr, "HIP error %s at %s:%d\n", hipGetErrorString(_e), __FILE__, __LINE__); \
    return; } } while (0)

namespace {

typedef unsigned u32;
typedef unsigned long long u64;
typedef unsigned u32x4 __attribute__((ext_vector_type(4)));
typedef int i32x2 __attribute__((ext_vector_type(2)));
typedef int i32x8 __attribute__((ext_vector_type(8)));
typedef float f32x16 __attribute__((ext_vector_type(16)));
#define LAS __attribute__((address_space(3)))
#define GAS __attribute__((address_space(1)))

constexpr int R = 8;
constexpr int MS = 512;
constexpr int KS = 512;
constexpr int NN = 4096;
constexpr int NWG = 256;
constexpr int NWAVES = 12;
constexpr int NTHREADS = NWAVES * 64;
constexpr int RR = 4;
constexpr int PIECE = 4096;
constexpr int SLOT = 5120;
constexpr int MAXEXC = 510;
constexpr int NPW = 23;
constexpr int XW = 2;
constexpr int TPOLL = 420;
constexpr int LDS_OWN = RR * 7 * PIECE;
constexpr int LDS_CTRL = LDS_OWN + 8 * PIECE;
constexpr int LDS_READY = LDS_CTRL;
constexpr int LDS_DONE = LDS_CTRL + 192;
constexpr int LDS_BAD = LDS_READY + (7 & (RR - 1)) * 32;
constexpr int LDS_GO = LDS_CTRL + 232;
constexpr int LDS_ENCD = LDS_CTRL + 240;
constexpr int LDS_GRP = LDS_CTRL + 208;
constexpr int LDS_STG = LDS_CTRL + 248;
constexpr int LDS_LUT = LDS_CTRL + 256;
constexpr int LDS_SCR = 7 * PIECE;
constexpr int LDS_BYTES = LDS_CTRL + 640;
static_assert(LDS_BYTES <= 160 * 1024, "LDS");

constexpr size_t WS_CNT = 0;
constexpr size_t WS_TMO = 32768;
constexpr size_t WS_TK = 36864;
constexpr size_t WS_FX = 65536;
constexpr size_t WS_FW = 131072;
constexpr u32 U0 = 32, U1 = 448;
constexpr size_t WS_FS = 393216;
constexpr size_t WS_XB = 1ull << 20;
constexpr size_t XB_PAR = 2ull << 20;
constexpr size_t WB_SRC = (size_t)512 * SLOT;
constexpr size_t WS_WB = 8ull << 20;
constexpr size_t WB_PAR = 8 * WB_SRC;
constexpr size_t WS_ST = WS_WB + 2 * WB_PAR;
constexpr size_t WS_END = WS_ST + 2 * WB_SRC;

constexpr u64 DEADLINE_TICKS = 40000000ull;

struct Params {
  const unsigned char* x;
  const unsigned char* w;
  const float* sx;
  const float* sw;
  float* out;
  unsigned char* peer[R];
  int rank;
  int pad;
};

#define UTILE(t) ((((t) & 7) << 3) | (((t) >> 3) & 7))
#define FMT_B(t) ((u32)(t) - U0 < U1 - U0)

__device__ __forceinline__ u64 rt() { return __builtin_amdgcn_s_memrealtime(); }

#define ST16(base, voff, val, imm) \
  asm volatile("global_store_dwordx4 %0, %1, %2 offset:%3 sc0 sc1\n\ts_nop 1" :: "v"(voff), "v"(val), "s"(base), "i"(imm) : "memory")
#define LD16S(val, base, voff, imm) \
  asm volatile("global_load_dwordx4 %0, %1, %2 offset:%3 sc0 sc1" : "+v"(val) : "v"(voff), "s"(base), "i"(imm) : "memory")
#define VMWAIT(n) asm volatile("s_waitcnt vmcnt(%0)" :: "i"(n) : "memory")

__device__ __forceinline__ void flag_store(unsigned char* p, u32 v) {
  __hip_atomic_store((GAS u32*)p, v, __ATOMIC_RELAXED, __HIP_MEMORY_SCOPE_SYSTEM);
}
__device__ __forceinline__ u32 flag_load(const unsigned char* p) {
  return __hip_atomic_load((GAS u32*)p, __ATOMIC_RELAXED, __HIP_MEMORY_SCOPE_SYSTEM);
}
__device__ __forceinline__ u32 ag_load(const unsigned char* p) {
  return __hip_atomic_load((GAS u32*)p, __ATOMIC_RELAXED, __HIP_MEMORY_SCOPE_AGENT);
}
__device__ __forceinline__ void ag_store(unsigned char* p, u32 v) {
  __hip_atomic_store((GAS u32*)p, v, __ATOMIC_RELAXED, __HIP_MEMORY_SCOPE_AGENT);
}

template <bool SYS, int NAP = 4>
__device__ __forceinline__ bool wait_all(const unsigned char* fp, u32 want, unsigned char* ws, u32 call) {
  const u64 t0 = rt();
  for (u32 spin = 1;; ++spin) {
    const u32 v = SYS ? flag_load(fp) : ag_load(fp);
    if (__all(v >= want)) break;
    if ((spin & 15) == 0 && (__builtin_amdgcn_readfirstlane(ag_load(ws + WS_TMO)) == call || rt() - t0 > DEADLINE_TICKS)) { ag_store(ws + WS_TMO, call); return false; }
    __builtin_amdgcn_s_sleep(NAP);
  }
  asm volatile("" ::: "memory");
  return true;
}

__device__ __forceinline__ u32 wave_sum(u32 v) {
  v += __builtin_amdgcn_update_dpp(0u, v, 0x111, 0xf, 0xf, true);
  v += __builtin_amdgcn_update_dpp(0u, v, 0x112, 0xf, 0xf, true);
  v += __builtin_amdgcn_update_dpp(0u, v, 0x114, 0xf, 0xf, true);
  v += __builtin_amdgcn_update_dpp(0u, v, 0x118, 0xf, 0xf, true);
  return __builtin_amdgcn_readlane(v, 15) + __builtin_amdgcn_readlane(v, 31) +
         __builtin_amdgcn_readlane(v, 47) + __builtin_amdgcn_readlane(v, 63);
}

__device__ __forceinline__ void read_bfrag(i32x8& r, u32 addr, int imm) {
  LAS unsigned char* b = (LAS unsigned char*)(size_t)addr;
  i32x2 r0 = __builtin_amdgcn_ds_read_tr8_b64_v2i32((LAS i32x2*)(b + imm));
  i32x2 r1 = __builtin_amdgcn_ds_read_tr8_b64_v2i32((LAS i32x2*)(b + imm + 256));
  i32x2 r2 = __builtin_amdgcn_ds_read_tr8_b64_v2i32((LAS i32x2*)(b + imm + 512));
  i32x2 r3 = __builtin_amdgcn_ds_read_tr8_b64_v2i32((LAS i32x2*)(b + imm + 768));
  r[0] = r0.x; r[1] = r0.y; r[2] = r1.x; r[3] = r1.y; r[4] = r2.x; r[5] = r2.y; r[6] = r3.x; r[7] = r3.y;
}

__device__ __forceinline__ u32 miss_pattern(u32 m  , u32 cneg, u32 lo4, u32& code) {
  code = (m + cneg) & 0x1f1f1f1fu;
  return (code + lo4) ^ m;
}

__device__ __forceinline__ u32 wave_scan(u32 v) {
  v += __builtin_amdgcn_update_dpp(0u, v, 0x111, 0xf, 0xf, true);
  v += __builtin_amdgcn_update_dpp(0u, v, 0x112, 0xf, 0xf, true);
  v += __builtin_amdgcn_update_dpp(0u, v, 0x114, 0xf, 0xf, true);
  v += __builtin_amdgcn_update_dpp(0u, v, 0x118, 0xf, 0xf, true);
  v += __builtin_amdgcn_update_dpp(0u, v, 0x142, 0xa, 0xf, false);
  v += __builtin_amdgcn_update_dpp(0u, v, 0x143, 0xc, 0xf, false);
  return v;
}

__device__ __forceinline__ u32 gather16(const u32x4& X) {
  u32 f[4];
#pragma unroll
  for (int e = 0; e < 4; ++e) f[e] = (X[e] + 0x7c7c7c7cu) & 0x80808080u;
  const u32 b0 = __builtin_amdgcn_udot4(f[1], 0x80402010u, __builtin_amdgcn_udot4(f[0], 0x08040201u, 0u, false), false);
  const u32 b1 = __builtin_amdgcn_udot4(f[3], 0x80402010u, __builtin_amdgcn_udot4(f[2], 0x08040201u, 0u, false), false);
  return (b0 >> 7) | (b1 << 1);
}

template <int RECR>
__device__ __forceinline__ void write_records(u32 (&M)[2], u32 wa0, u32 wa1, LAS unsigned char* xs, LAS unsigned char* recs, u32 dummy, u32 vo) {
  u32 wa[2] = {wa0, wa1};
  do {
    u32 pos[2][RECR], x[2][RECR], ad[2][RECR];
#pragma unroll
    for (int r = 0; r < RECR; ++r)
#pragma unroll
      for (int h = 0; h < 2; ++h) {
        const u32 bit = __builtin_ctz(M[h] | 0x80000000u);
        pos[h][r] = vo + (bit & 15) + ((bit & 16) << 6);
        x[h][r] = xs[h * 2048 + pos[h][r]];
        ad[h][r] = M[h] ? wa[h] : dummy;
        wa[h] += 2;
        M[h] &= M[h] - 1;
      }
#pragma unroll
    for (int r = 0; r < RECR; ++r)
#pragma unroll
      for (int h = 0; h < 2; ++h)
        *(LAS unsigned short*)(recs + ad[h][r]) = (unsigned short)(pos[h][r] | (x[h][r] << 9));
  } while (__any((M[0] | M[1]) != 0));
}

__device__ __forceinline__ void apply_records(u32 ev, u32 first, u32 lim, u32 nhalf, LAS unsigned char* dst, int lane) {
#pragma unroll
  for (int h = 0; h < 2; ++h) {
    const u32 rec = h ? ev >> 16 : ev & 0xffffu;
    const u32 idx = first + 2 * lane + h;
    const u32 off = (rec & 0x7ffu) | (idx >= nhalf ? 0x800u : 0u);
    if (idx < lim)
      __hip_atomic_fetch_xor((LAS u32*)(dst + (off & ~3u)), ((rec >> 11) << 2) << (8 * (off & 3)), __ATOMIC_RELAXED, __HIP_MEMORY_SCOPE_WORKGROUP);
  }
}

__device__ __forceinline__ int choose_window(const u32x4 (&V)[4]) {
  u32 sum = 0;
#pragma unroll
  for (int t = 0; t < 4; ++t) sum = __builtin_amdgcn_sad_u8(V[t][3 - t] & 0x7c7c7c7cu, 0u, sum);
  int lo = (int)((wave_sum(sum) + 1638u) >> 12) - 6;
  lo = lo < 0 ? 0 : (lo > 22 ? 22 : lo);
  u32 cnts = 0;
#pragma unroll
  for (int k = 0; k < 3; ++k) {
    const u32 lo4 = (u32)(lo + k) * 0x04040404u, cneg = (u32)((-4 * (lo + k)) & 31) * 0x01010101u;
    u32 n = 0, code;
#pragma unroll
    for (int t = 0; t < 4; ++t)
      n += __builtin_popcount((miss_pattern(V[t][t] & 0x7f7f7f7fu, cneg, lo4, code) + 0x7c7c7c7cu) & 0x80808080u);
    cnts |= min(n, 15u) << (10 * k);
  }
  const u32 tot = wave_sum(cnts);
  const u32 n0 = tot & 1023, n1 = (tot >> 10) & 1023, n2 = tot >> 20;
  return lo + ((n1 <= n0 && n1 <= n2) ? 1 : (n0 <= n2 ? 0 : 2));
}

__device__ __forceinline__ u32 encode_piece(const u32x4 (&V)[4], int lo, u32x4 (&D)[4], u32x4& T, LAS unsigned char* scr, int lane) {
  const u32 lo4 = (u32)lo * 0x04040404u, cneg = (u32)((-4 * lo) & 31) * 0x01010101u;
  const u32 vo = lane * 16;
  u32x4 C[4];
  u32 M[2];
#pragma unroll
  for (int t = 0; t < 4; ++t) {
    u32x4 X;
#pragma unroll
    for (int e = 0; e < 4; ++e) {
      u32 code;
      X[e] = miss_pattern(V[t][e] & 0x7f7f7f7fu, cneg, lo4, code);
      C[t][e] = code | (V[t][e] & 0x80808080u);
    }
    *(LAS u32x4*)(scr + t * 1024 + vo) = X;
    const u32 m16 = gather16(X);
    if (t & 1) M[t >> 1] |= m16 << 16; else M[t >> 1] = m16;
  }
  const u32 n = __builtin_popcount(M[0]) | (__builtin_popcount(M[1]) << 16);
  const u32 inc = wave_scan(n);
  const u32 tot = __builtin_amdgcn_readlane(inc, 63);
  const u32 nhalf = tot & 0xffffu, nexc = nhalf + (tot >> 16);
  LAS unsigned char* recs = scr + 4096;
  if (nexc > MAXEXC) {
    D[0] = V[0]; D[1] = V[1]; D[2] = V[2]; T = V[3];
    D[3] = u32x4{0x80000000u, 0u, 0u, 0u};
    return nexc;
  }
  if (lane == 0) *(LAS u32*)recs = nhalf | (nexc << 10) | ((u32)lo << 20);
  write_records<3>(M, 4 + 2 * ((inc - n) & 0xffffu), 4 + 2 * (nhalf + ((inc - n) >> 16)), scr, recs, 1024 + 2 * lane, vo);
#pragma unroll
  for (int e = 0; e < 4; ++e) {
    const u32 c3 = C[3][e];
    D[0][e] = C[0][e] | ((c3 << 5) & 0x60606060u);
    D[1][e] = C[1][e] | ((c3 << 3) & 0x60606060u);
    D[2][e] = C[2][e] | ((c3 << 1) & 0x20202020u) | ((c3 >> 1) & 0x40404040u);
  }
  D[3] = *(LAS u32x4*)(recs + vo);
  return nexc;
}

__device__ __forceinline__ void unpack_piece(const unsigned char* src, LAS unsigned char* dst, int lane) {
  LAS u32x4* o = (LAS u32x4*)(dst + lane * 16);
  u32 ev = *(LAS u32*)(dst + 3072 + lane * 4);
  u32x4 D[3] = {o[0], o[64], o[128]};
  asm volatile("" : "+v"(ev), "+v"(D[0]), "+v"(D[1]), "+v"(D[2]));
  const u32 hdr = __builtin_amdgcn_readfirstlane(ev);
  if (hdr >> 31) {
    u32x4 D3;
    asm volatile("global_load_dwordx4 %0, %1, %2 sc0 sc1\n\ts_waitcnt vmcnt(0)" : "=&v"(D3) : "v"(lane * 16), "s"(src + 4096) : "memory");
    o[192] = D3;
    return;
  }
  const u32 lo4 = ((hdr >> 20) & 31) * 0x04040404u, nhalf = hdr & 1023, nexc = (hdr >> 10) & 1023;
  u32x4 O0, O1, O2, O3;
#pragma unroll
  for (int e = 0; e < 4; ++e) {
    const u32 d0 = D[0][e], d1 = D[1][e], d2 = D[2][e];
    u32 s0 = d0 & 0x60606060u, s1 = d1 & 0x60606060u, s2 = d2 & 0x60606060u;
    asm("" : "+v"(s0), "+v"(s1), "+v"(s2));
    O0[e] = (d0 ^ s0) + lo4; O1[e] = (d1 ^ s1) + lo4; O2[e] = (d2 ^ s2) + lo4;
    const u32 t = (s0 >> 5) | (s1 >> 3) | (s2 >> 1), z = t & 0x20202020u;
    O3[e] = t + lo4 + 3 * z;
  }
  o[0] = O0; o[64] = O1; o[128] = O2; o[192] = O3;
  for (u32 base = 0;;) {
    apply_records(ev, base - 2, nexc, nhalf, dst, lane);
    base += 128;
    if (base >= nexc + 2) break;
    asm volatile("global_load_dword %0, %1, %2 offset:3072 sc0 sc1\n\ts_waitcnt vmcnt(0)" : "=&v"(ev) : "v"(lane * 4 + 2 * base), "s"(src) : "memory");
  }
}

constexpr int B_HDR = 2560, B_STR = 2576;
__device__ __forceinline__ u64 tier3_exponents(int b) {
  if (b >= 13) return (u64)(4 * (b - 3)) * 0x0101010101010101ull - 0x1c1814100c080400ull;
  u64 r = 0;
#pragma unroll
  for (int k = 0; k < 8; ++k) r |= (u64)(4 * max(b - 3 - k, 3)) << (8 * k);
  return r;
}
__device__ __forceinline__ u32 tier2_exponents(int b) {
  return (u32)(4 * max(b - 1, 3)) | (u32)(4 * max(b - 2, 3)) << 8 | (u32)(4 * (b + 3)) << 16;
}
__device__ __forceinline__ int choose_base(const u32x4 (&V)[4]) {
  u32 sum = 0;
#pragma unroll
  for (int t = 0; t < 4; ++t) sum = __builtin_amdgcn_sad_u8(V[t][3 - t] & 0x7c7c7c7cu, 0u, sum);
  int b = (int)((wave_sum(sum) + 1638u) >> 12) - 2;
  b = b < 0 ? 0 : (b > 26 ? 26 : b);
  u32 cnts = 0;
#pragma unroll
  for (int k = 0; k < 3; ++k) {
    const u32 c0 = (u32)(0x80 - 4 * (b + k)) * 0x01010101u;
    u32 n = 0;
#pragma unroll
    for (int t = 0; t < 4; ++t) {
      const u32 q = (V[t][t] & 0x7f7f7f7fu) + c0;
      n += __builtin_popcount(q & ~(q - 0x0c0c0c0cu) & 0x80808080u);
    }
    cnts |= min(n, 15u) << (10 * k);
  }
  const u32 tot = wave_sum(cnts);
  const u32 n0 = tot & 1023, n1 = (tot >> 10) & 1023, n2 = tot >> 20;
  return b + ((n1 >= n0 && n1 >= n2) ? 1 : (n0 >= n2 ? 0 : 2));
}

__device__ __forceinline__ void or_bits(LAS unsigned char* base, u32 bp, u64 a) {
  LAS u32* w = (LAS u32*)(base + ((bp >> 5) << 2));
  const u32 sh = bp & 31;
  const u64 x = a << sh;
  __hip_atomic_fetch_or(w, (u32)x, __ATOMIC_RELAXED, __HIP_MEMORY_SCOPE_WORKGROUP);
  __hip_atomic_fetch_or(w + 1, (u32)(x >> 32), __ATOMIC_RELAXED, __HIP_MEMORY_SCOPE_WORKGROUP);
  __hip_atomic_fetch_or(w + 2, ((u32)(a >> 32) >> 1) >> (31 - sh), __ATOMIC_RELAXED, __HIP_MEMORY_SCOPE_WORKGROUP);
}

__device__ __forceinline__ u32 encode_b(const u32x4 (&V)[4], int b, u32x4 (&N)[2], LAS unsigned char* scr, LAS unsigned char* lut, int lane) {
  const u32 vo = lane * 16;
  LAS unsigned char* tail = scr + 4096;
  *(LAS u32x4*)(tail + vo) = u32x4{0u, 0u, 0u, 0u};
  *(LAS u32x4*)(tail + 1024 + vo) = u32x4{0u, 0u, 0u, 0u};
  const u32 c1 = (u32)(0x83 + 4 * (b + 3)) * 0x01010101u;
  const u64 ex = tier3_exponents(b);
  const u32 exlo = (u32)ex, exhi = (u32)(ex >> 32), ex2 = tier2_exponents(b);
  const u32 dlo = (u32)(4 * (b + 3)) * 0x01010101u - 0x0c080400u, dhi = ex2 & 0xffffu;
  u32 M[2], S[2] = {0u, 0u}, o2[2], o3[4];
  u64 a2[2], a3[4];
  u32 k2[2][4], k3[2][4], e1[2][4], e3[2][4], w2[2][4];
  i32x2 wm[2][4];
#define BYR(hi, lo) __builtin_amdgcn_perm(hi, lo, r)
#pragma unroll
  for (int t = 0; t < 5; ++t) {
    if (t < 4) {
      u32x4 X;
      u32 n1[4];
      const int s = t & 1;
#pragma unroll
      for (int e = 0; e < 4; ++e) {
        const u32 v = V[t][e], m = v & 0x7f7f7f7fu;
        const u32 q = c1 - m;
        const u32 H = ((((q ^ 0x80808080u) & 0xe0e0e0e0u) >> 1) + 0x70707070u) & 0x80808080u;
        const u32 r = ((q >> 2) | ((H >> 4) - (H >> 7))) & 0x07070707u;
        n1[e] = BYR(0x0c0c0c0cu, 0x0004080cu) | (m & 0x03030303u);
        e1[s][e] = BYR(0x01010101u, 0x00000001u);
        e3[s][e] = BYR(0x01010000u, 0u);
        k2[s][e] = BYR(0x03030100u, 0x00000002u);
        k3[s][e] = ((q - 0x18181818u) >> 2) & BYR(0x07070000u, 0u);
        X[e] = ((BYR(dhi, dlo) | (__builtin_amdgcn_perm(exhi, exlo, k3[s][e]) & BYR(0x7c7c0000u, 0u))) ^ m) & 0x7c7c7c7cu;
        S[t >> 1] |= (v >> (4 * s + e)) & (0x80808080u >> (4 * s + e));
        w2[s][e] = *(LAS u32*)(lut + 8 + __builtin_amdgcn_udot4(e1[s][e], 0x80402010u, 0u, false));
        wm[s][e] = *(LAS i32x2*)(lut + __builtin_amdgcn_udot4(e3[s][e], 0x80402010u, 0u, false));
      }
      *(LAS u32x4*)(scr + t * 1024 + vo) = X;
      const u32 m16 = gather16(X);
      if (s) M[t >> 1] |= m16 << 16; else M[t >> 1] = m16;
      N[t >> 1][2 * s] = n1[0] | (n1[1] << 4);
      N[t >> 1][2 * s + 1] = n1[2] | (n1[3] << 4);
      asm volatile("" : "+v"(N[t >> 1][2 * s]), "+v"(N[t >> 1][2 * s + 1]), "+v"(S[t >> 1]), "+v"(M[t >> 1]));
    }
    if (t > 0) {
      const int p = t - 1, s = p & 1;
      u32 ob2 = 0, ob3 = 0, s2 = 0;
      u64 s3 = 0;
#pragma unroll
      for (int e = 0; e < 4; ++e) {
        s2 |= __builtin_amdgcn_udot4(k2[s][e], w2[s][e], 0u, false) << ob2;
        ob2 = __builtin_amdgcn_udot4(e1[s][e], 0x02020202u, ob2, false);
        s3 |= (u64)__builtin_amdgcn_udot4(k3[s][e], (u32)wm[s][e].x, (k3[s][e] >> 15) & (u32)wm[s][e].y, false) << ob3;
        ob3 = __builtin_amdgcn_udot4(e3[s][e], 0x03030303u, ob3, false);
      }
      if (s) { a2[p >> 1] |= (u64)s2 << o2[p >> 1]; o2[p >> 1] += ob2; } else { a2[p >> 1] = s2; o2[p >> 1] = ob2; }
      a3[p] = s3; o3[p] = ob3;
      asm volatile("" : "+v"(a2[p >> 1]), "+v"(a3[p]));
    }
  }
#undef BYR
  const u32 nb = (o2[0] + o2[1]) | (o3[0] + o3[1] + o3[2] + o3[3]) << 16;
  const u32 nr = __builtin_popcount(M[0]) | (__builtin_popcount(M[1]) << 16);
  const u32 ib = wave_scan(nb), ir = wave_scan(nr);
  const u32 totb = __builtin_amdgcn_readlane(ib, 63), totr = __builtin_amdgcn_readlane(ir, 63);
  const u32 bits2 = totb & 0xffffu, bits3 = totb >> 16;
  const u32 nhalf = totr & 0xffffu, nexc = nhalf + (totr >> 16);
  const u32 st3 = B_STR + ((bits2 + 31) >> 5) * 4, rec0 = st3 + ((bits3 + 31) >> 5) * 4;
  const u32 len = rec0 + 2 * nexc;
  if (bits2 > 2 * 2040 || bits3 > 3 * 512 || nexc > MAXEXC || len > 4032) return 0;
  *(LAS i32x2*)(tail + lane * 8) = i32x2{(int)S[0], (int)S[1]};
  if (lane == 0) *(LAS u32x4*)(tail + B_HDR - 2048) = u32x4{nhalf | (nexc << 10) | ((u32)b << 20), bits2 | (st3 << 16), len, bits3 | (rec0 << 16)};
  {
    u32 bp = ((ib - nb) & 0xffffu) + (B_STR - 2048) * 8;
#pragma unroll
    for (int h = 0; h < 2; ++h) { or_bits(tail, bp, a2[h]); bp += o2[h]; }
    bp = ((ib - nb) >> 16) + (st3 - 2048) * 8;
#pragma unroll
    for (int t = 0; t < 4; ++t) { or_bits(tail, bp, a3[t]); bp += o3[t]; }
  }
  write_records<1>(M, rec0 - 2048 + 2 * ((ir - nr) & 0xffffu), rec0 - 2048 + 2 * (nhalf + ((ir - nr) >> 16)), scr, tail, 2048 + 2 * lane, vo);
  return len;
}

__device__ __forceinline__ void unpack_b(const unsigned char* src, LAS unsigned char* dst, LAS unsigned char* lut, int lane) {
  LAS u32x4* o = (LAS u32x4*)(dst + lane * 16);
  u32x4 N[2] = {o[0], o[64]};
  i32x2 S = *(LAS i32x2*)(dst + 2048 + lane * 8);
  u32x4 H = *(LAS u32x4*)(dst + B_HDR);
  u32 c0 = *(LAS u32*)(dst + B_STR + lane * 4);
  asm volatile("" : "+v"(N[0]), "+v"(N[1]), "+v"(S), "+v"(H), "+v"(c0));
  const u32 hdr = __builtin_amdgcn_readfirstlane(H.x), h1 = __builtin_amdgcn_readfirstlane(H.y), h3 = __builtin_amdgcn_readfirstlane(H.w);
  if (hdr >> 31) {
    u32x4 D3, D4;
    asm volatile("global_load_dwordx4 %0, %2, %3 offset:3072 sc0 sc1\n\tglobal_load_dwordx4 %1, %4, %3 sc0 sc1\n\ts_waitcnt vmcnt(0)"
                 : "=&v"(D3), "=&v"(D4) : "v"(lane * 16), "s"(src), "v"(4096u) : "memory");
    o[192] = D3;
    *(LAS u32x4*)(dst + B_HDR) = D4;
    return;
  }
  const int b = (hdr >> 20) & 31;
  const u32 nhalf = hdr & 1023, nexc = (hdr >> 10) & 1023, bits2 = h1 & 0xffffu, st3 = h1 >> 16, bits3 = h3 & 0xffffu, rec0 = h3 >> 16;
  if (rec0 + 2 * nexc > 3072) {
    u32x4 D3;
    asm volatile("global_load_dwordx4 %0, %1, %2 offset:3072 sc0 sc1\n\ts_waitcnt vmcnt(0)" : "=&v"(D3) : "v"(lane * 16), "s"(src) : "memory");
    o[192] = D3;
  }
  u32 ev = *(LAS u32*)(dst + min(rec0 + lane * 4, 4092u));
  const u32 so = st3 + (((3 * lane) >> 1) & ~3);
  const u32 t0 = *(LAS u32*)(dst + so), t1 = *(LAS u32*)(dst + so + 4);
  u32 c1 = 0, t2 = 0, t3 = 0;
  if (bits2 > 2048) c1 = *(LAS u32*)(dst + B_STR + 256 + lane * 4);
  if (bits3 > 768) { t2 = *(LAS u32*)(dst + so + 96); t3 = *(LAS u32*)(dst + so + 100); }
#define WINDOW(a) (*(LAS i32x2*)(size_t)(((a) + ((a) & 4u) * 511u) & ~3u))
  {
    const u64 ex = tier3_exponents(b) - 0x0c0c0c0c0c0c0c0cull;
    auto expand3 = [&](u32 a0, u32 a1, int p) {
      const u32 g = __builtin_amdgcn_alignbit(a1, a0, 12 * lane) & 0xfffu;
      const u32 z = (g | (g << 10)) & 0x003f003fu;
      const u32 r = __builtin_amdgcn_perm((u32)(ex >> 32), (u32)ex, (z | (z << 5)) & 0x07070707u);
      *(LAS u32*)(dst + p * 256 + lane * 4) = r;
      *(LAS u32*)(dst + 2044 + p * 256 + lane * 4) = r;
    };
    expand3(t0, t1, 0);
    if (bits3 > 768) expand3(t2, t3, 1);
  }
  {
    const u32 ex2 = (tier2_exponents(b) - 0x0c0c0cu) & 0xffffffu;
    const u32 n3 = __builtin_popcount(c0 & (c0 >> 1) & 0x55555555u) | __builtin_popcount(c1 & (c1 >> 1) & 0x55555555u) << 16;
    const u32 i3 = wave_scan(n3);
    u32x4 r[2];
    auto expand2 = [&](u32 cw, u32 wa, u32x4& out) {
      u32 y[4], sl[4], ad[4];
      i32x2 w[4];
#pragma unroll
      for (int i = 0; i < 4; ++i) {
        const u32 g = (cw >> (8 * i)) & 0xffu;
        const u32 z = (g | (g << 12)) & 0x000f000fu;
        y[i] = (z | (z << 6)) & 0x03030303u;
        const u32 f = y[i] & (y[i] >> 1);
        sl[i] = *(LAS u32*)(lut + __builtin_amdgcn_udot4(f, 0x20100804u, 0u, false));
        w[i] = WINDOW(wa);
        ad[i] = wa;
        wa += __builtin_popcount(f);
      }
      asm volatile("" : "+v"(w[0]), "+v"(w[1]), "+v"(w[2]), "+v"(w[3]), "+v"(sl[0]), "+v"(sl[1]), "+v"(sl[2]), "+v"(sl[3]));
#pragma unroll
      for (int i = 0; i < 4; ++i)
        out[i] = __builtin_amdgcn_perm(0u, ex2, y[i]) | __builtin_amdgcn_perm(0u, __builtin_amdgcn_alignbyte((u32)w[i].y, (u32)w[i].x, ad[i]), sl[i]);
    };
    expand2(c0, (u32)(size_t)dst + ((i3 - n3) & 0xffffu), r[0]);
    if (bits2 > 2048) expand2(c1, (u32)(size_t)dst + (__builtin_amdgcn_readlane(i3, 63) & 0xffffu) + ((i3 - n3) >> 16), r[1]);
#pragma unroll
    for (int p = 0; p < 2; ++p)
      if (p == 0 || bits2 > 2048) {
        o[p * 64] = r[p];
        o[128 + p * 64] = u32x4{r[p][1], r[p][2], r[p][3], (u32)__builtin_amdgcn_update_dpp(0u, r[p][0], 0x130, 0xf, 0xf, true)};
        if (p && lane == 0) *(LAS u32*)(dst + 2044 + 1024) = r[p][0];
      }
  }
  u32 G[8], n = 0;
#pragma unroll
  for (int i = 0; i < 8; ++i) {
    const u32 x = N[i >> 2][i & 3];
    G[i] = x & (x >> 1) & 0x44444444u;
    n += __builtin_popcount(G[i]);
  }
  u32 wa = (u32)(size_t)dst + wave_scan(n) - n;
  const u32 b4 = (u32)(4 * b) * 0x01010101u;
  u32 wad[16], sel[16];
  i32x2 W[16];
#pragma unroll
  for (int j = 0; j < 16; ++j) {
    const u32 f = ((j & 1) ? G[j >> 1] >> 4 : G[j >> 1]) & 0x04040404u;
    sel[j] = *(LAS u32*)(lut + __builtin_amdgcn_udot4(f, 0x08040201u, 0u, false));
    W[j] = WINDOW(wa);
    wad[j] = wa;
    wa += __builtin_popcount(f);
  }
#undef WINDOW
  asm volatile("" : "+v"(W[0]), "+v"(W[1]), "+v"(W[2]), "+v"(W[3]), "+v"(W[4]), "+v"(W[5]), "+v"(W[6]), "+v"(W[7]),
               "+v"(sel[0]), "+v"(sel[1]), "+v"(sel[2]), "+v"(sel[3]), "+v"(sel[4]), "+v"(sel[5]), "+v"(sel[6]), "+v"(sel[7]));
  asm volatile("" : "+v"(W[8]), "+v"(W[9]), "+v"(W[10]), "+v"(W[11]), "+v"(W[12]), "+v"(W[13]), "+v"(W[14]), "+v"(W[15]),
               "+v"(sel[8]), "+v"(sel[9]), "+v"(sel[10]), "+v"(sel[11]), "+v"(sel[12]), "+v"(sel[13]), "+v"(sel[14]), "+v"(sel[15]));
#pragma unroll
  for (int t = 0; t < 4; ++t) {
    u32x4 O;
#pragma unroll
    for (int e = 0; e < 4; ++e) {
      const int j = 4 * t + e;
      const u32 x = N[j >> 3][(j >> 1) & 3];
      const u32 nib = ((j & 1) ? x >> 4 : x) & 0x0f0f0f0fu;
      const u32 sg = ((u32)S[t >> 1] << (j & 7)) & 0x80808080u;
      O[e] = (nib | sg) + __builtin_amdgcn_perm(b4, __builtin_amdgcn_alignbyte((u32)W[j].y, (u32)W[j].x, wad[j]), sel[j]);
    }
    o[t * 64] = O;
  }
  for (u32 base = 0;;) {
    apply_records(ev, base, nexc, nhalf, dst, lane);
    base += 128;
    if (base >= nexc) break;
    asm volatile("global_load_dword %0, %1, %2 sc0 sc1\n\ts_waitcnt vmcnt(0)" : "=&v"(ev) : "v"(rec0 + lane * 4 + 2 * base), "s"(src) : "memory");
  }
}

__device__ __forceinline__ void init_luts(LAS unsigned char* lut, int lane) {
  if (lane < 16) {
    u32 w3 = 0, w2 = 0, sel = 0, r = 0;
#pragma unroll
    for (int i = 0; i < 4; ++i) {
      const bool on = (lane >> i) & 1;
      w3 |= (on && r < 3 ? 1u << (3 * r) : 0u) << (8 * i);
      w2 |= (on ? 1u << (2 * r) : 0u) << (8 * i);
      sel |= (on ? r : 4u) << (8 * i);
      r += on;
    }
    *(LAS u32x4*)(lut + lane * 16) = u32x4{w3, lane == 15 ? 0xe00u : 0u, w2, 0u};
    *(LAS u32*)(lut + 256 + lane * 4) = sel;
  }
}

__global__ void __launch_bounds__(NTHREADS, 3) __attribute__((amdgpu_num_vgpr(128))) fused_kernel(Params P) {
  extern __shared__ __attribute__((aligned(16))) unsigned char lds_raw[];
  LAS unsigned char* L = (LAS unsigned char*)lds_raw;
  volatile LAS u32* READY = (volatile LAS u32*)(L + LDS_READY);
  volatile LAS u32* DONE = (volatile LAS u32*)(L + LDS_DONE);
  volatile LAS u32* BAD = (volatile LAS u32*)(L + LDS_BAD);
  volatile LAS u32* GO = (volatile LAS u32*)(L + LDS_GO);

  const int c = blockIdx.x;
  const int lane = threadIdx.x & 63;
  const int wave = __builtin_amdgcn_readfirstlane(threadIdx.x >> 6);
  const int u = (c & 7) * 8 + ((c >> 3) & 7);
  const int mt = c >> 6;
  const int pr = ((c >> 3) & 7) * 32 + mt * 8 + (c & 7);
  unsigned char* const ws = P.peer[0];

  unsigned char* cntp = ws + WS_CNT + c * 128 + wave * 4;
#define START_BARRIER() do { __builtin_amdgcn_sched_barrier(0); __builtin_amdgcn_s_barrier(); __builtin_amdgcn_sched_barrier(0); } while (0)
  const u32 w_lane = (u32)(lane >> 1) * NN + (lane & 1) * 16;
  const u32 vo = lane * 16;

  if (wave == NWAVES - 1) {
    __builtin_amdgcn_s_waitcnt(0x0F70);
    {
      const int d = pr < 224 ? 1 + pr % 7 : 0, j = pr < 224 ? pr / 7 : pr - 224;
      const unsigned char* xs = P.x + (size_t)(((P.rank + d) & 7) * MS + (j >> 1) * 32) * KS + (j & 1) * 256;
      const u32 xo = (u32)(lane & 31) * KS + 32 * (lane >> 5);
#define ACLOB "v128", "v129", "v130", "v131", "v132", "v133", "v134", "v135", "v136", "v137", "v138", "v139", "v140", "v141", "v142", "v143", "v144", "v145", "v146", "v147", \
              "v148", "v149", "v150", "v151", "v152", "v153", "v154", "v155", "v156", "v157", "v158", "v159", "v160", "v161", "memory"
      asm volatile("global_load_dword v160, %0, %1 sc1\n\t"
                   "global_load_dwordx4 v[128:131], %2, %3\n\tglobal_load_dwordx4 v[132:135], %2, %3 offset:16\n\t"
                   "global_load_dwordx4 v[136:139], %2, %3 offset:64\n\tglobal_load_dwordx4 v[140:143], %2, %3 offset:80\n\t"
                   "global_load_dwordx4 v[144:147], %2, %3 offset:128\n\tglobal_load_dwordx4 v[148:151], %2, %3 offset:144\n\t"
                   "global_load_dwordx4 v[152:155], %2, %3 offset:192\n\tglobal_load_dwordx4 v[156:159], %2, %3 offset:208\n\t"
                   "v_mov_b32 v161, 1"
                   :: "v"(0u), "s"(cntp), "v"(xo), "s"(xs) : ACLOB);
      START_BARRIER();
      __builtin_amdgcn_s_setprio(2);
      u32 cnt;
      asm volatile("s_waitcnt vmcnt(8)\n\tv_mov_b32 %0, v160" : "=v"(cnt) :: "memory");
      const u32 call = __builtin_amdgcn_readfirstlane(cnt) + 1;
      unsigned char* pd = P.peer[d];
      unsigned char* dst = pd + WS_XB + (call & 1) * XB_PAR + (size_t)(d * 32 + j) * 8192;
#define XST(k, r) asm volatile("s_waitcnt vmcnt(%2)\n\tglobal_store_dwordx4 %0, " r ", %1 offset:%3 sc0 sc1\n\ts_nop 1" \
                               :: "v"(vo), "s"(dst + ((k) >> 2) * 4096), "i"((k) < XW ? 7 : XW - 1), "i"(((k) & 3) * 1024) : "memory")
#define XSTS(a, b) do { if (a <= 0 && 0 < b) XST(0, "v[128:131]"); if (a <= 1 && 1 < b) XST(1, "v[132:135]"); if (a <= 2 && 2 < b) XST(2, "v[136:139]"); \
        if (a <= 3 && 3 < b) XST(3, "v[140:143]"); if (a <= 4 && 4 < b) XST(4, "v[144:147]"); if (a <= 5 && 5 < b) XST(5, "v[148:151]"); \
        if (a <= 6 && 6 < b) XST(6, "v[152:155]"); if (a <= 7 && 7 < b) XST(7, "v[156:159]"); } while (0)
      XSTS(0, 8);
      GO[0] = 1u;
#undef XSTS
#undef XST
      if (lane == 0) ag_store(cntp, call);
      VMWAIT(0);
      if (lane < 16) flag_store(pd + WS_FX + (d * 32 + j) * 64 + lane * 4, call);
      if (d != 0 && j < NPW) {
        const unsigned char* tkp = ws + WS_TK + d * 128;
        const unsigned char* const rsrc = P.peer[R - d] + WS_ST + (call & 1) * WB_SRC;
        unsigned char* const ldst = ws + WS_WB + (call & 1) * WB_PAR + d * WB_SRC;
        unsigned char* const lfw = ws + WS_FW + d * 64 * 512;
        const u32 tbase = (call - 1) * 512u - 2 * NPW;
        u32 staged = 0;
#define FETCHR(t) do { \
          while ((t) >= staged) { staged = __builtin_amdgcn_readfirstlane(__hip_atomic_load((LAS u32*)(L + LDS_STG), __ATOMIC_ACQUIRE, __HIP_MEMORY_SCOPE_WORKGROUP)); \
                                  if ((t) >= staged) __builtin_amdgcn_s_sleep(2); } \
          if (staged > 512u) return;       \
          const unsigned char* s_ = rsrc + (size_t)(t) * SLOT; \
          got = FMT_B(t) ? wantb : max(wanta, (t) < U0 ? 3200u : 3328u); \
          asm volatile("global_load_dwordx4 v[128:131], %0, %1 sc0 sc1\n\tglobal_load_dwordx4 v[132:135], %0, %1 offset:1024 sc0 sc1" \
                       :: "v"(vo), "s"(s_) : ACLOB); \
          if (2048 + vo < got) asm volatile("global_load_dwordx4 v[136:139], %0, %1 offset:2048 sc0 sc1" :: "v"(vo), "s"(s_) : ACLOB); \
          if (3072 + vo < got) asm volatile("global_load_dwordx4 v[140:143], %0, %1 offset:3072 sc0 sc1" :: "v"(vo), "s"(s_) : ACLOB); \
          if (4096 + vo < got) asm volatile("global_load_dwordx4 v[144:147], %0, %1 sc0 sc1" :: "v"(vo), "s"(s_ + 4096) : ACLOB); } while (0)
        u32 wantb = 2944, wanta = 0, got;
        u32 cur = j;
        asm volatile("v_mov_b32 v160, %0" :: "v"(tbase + NPW + j) : ACLOB);
        FETCHR(cur);
#pragma unroll 1
        while (true) {
          u32 tk, hb, ha;
          asm volatile("s_waitcnt vmcnt(0)\n\tv_readfirstlane_b32 %0, v160\n\tv_readlane_b32 %1, v138, 32\n\tv_readfirstlane_b32 %2, v140"
                       : "=s"(tk), "=s"(hb), "=s"(ha) :: "memory");
          const u32 nxt = tk - tbase;
          const u32 len = FMT_B(cur) ? hb : ((ha >> 31) ? 5120u : 3076u + 2 * ((ha >> 10) & 1023));
          unsigned char* o_ = ldst + (size_t)((cur & ~63u) + UTILE(cur)) * SLOT;
          if (len > got) {
            const unsigned char* s_ = rsrc + (size_t)cur * SLOT;
            asm volatile("global_load_dwordx4 v[136:139], %0, %1 offset:2048 sc0 sc1\n\tglobal_load_dwordx4 v[140:143], %0, %1 offset:3072 sc0 sc1\n\t"
                         "global_load_dwordx4 v[144:147], %0, %2 sc0 sc1\n\ts_waitcnt vmcnt(0)" :: "v"(vo), "s"(s_), "s"(s_ + 4096) : ACLOB);
            if (FMT_B(cur)) wantb = min((len + 127u) & ~127u, 5120u); else wanta = min((len + 127u) & ~127u, 5120u);
          }
          asm volatile("global_store_dwordx4 %0, v[128:131], %1 sc1\n\tglobal_store_dwordx4 %0, v[132:135], %1 offset:1024 sc1\n\t"
                       "global_store_dwordx4 %0, v[136:139], %1 offset:2048 sc1\n\ts_nop 1" :: "v"(vo), "s"(o_) : "memory");
          if (3072 + (vo & ~127u) < len) asm volatile("global_store_dwordx4 %0, v[140:143], %1 offset:3072 sc1\n\ts_nop 1" :: "v"(vo), "s"(o_) : "memory");
          if (4096 + (vo & ~127u) < len) asm volatile("global_store_dwordx4 %0, v[144:147], %1 sc1\n\ts_nop 1" :: "v"(vo), "s"(o_ + 4096) : "memory");
          if (lane == 0) asm volatile("global_atomic_add v160, %0, v161, %1 sc0 sc1" :: "v"(0u), "s"(tkp) : ACLOB);
          if (nxt < 512u) {
            FETCHR(nxt);
            VMWAIT(4);
          } else {
            VMWAIT(0);
          }
          if (lane == 0) ag_store(lfw + UTILE(cur) * 512 + (cur >> 6) * 64, call);
          if (nxt >= 512u) break;
          cur = nxt;
        }
#undef FETCHR
      }
#undef ACLOB
    }
    return;
  }
  const u64 t_start = rt();
  const u32 cnt_old = ag_load(cntp);
  if (wave == 5) asm volatile("ds_write_b32 %0, %1\n\ts_waitcnt lgkmcnt(0)" :: "v"((u32)(size_t)L + LDS_CTRL + lane * 4), "v"(0u) : "memory");
  if (wave == 6) { init_luts(L + LDS_LUT, lane); asm volatile("s_waitcnt lgkmcnt(0)" ::: "memory"); }
  u32 call;
  if (pr >= 224) {
    START_BARRIER();
    call = __builtin_amdgcn_readfirstlane(cnt_old) + 1;
    LAS unsigned char* scr = L + LDS_SCR + wave * 6400;
    if (wave < 4) __builtin_amdgcn_s_setprio(1);
    unsigned char *fl0 = P.peer[1], *fl1 = P.peer[5];
#pragma unroll
    for (int q = 1; q < 4; ++q) { fl0 = lane >> 4 == q ? P.peer[1 + q] : fl0; if (q < 3) fl1 = lane >> 4 == q ? P.peer[5 + q] : fl1; }
    fl0 += WS_FS + (1 + (lane >> 4)) * 8192 + (pr - 224) * 64 + (lane & 15) * 4;
    fl1 += WS_FS + (5 + (lane >> 4)) * 8192 + (pr - 224) * 64 + (lane & 15) * 4;
#pragma unroll 1
    for (u32 e = (pr - 224) + 32 * wave; e < 512u && wave < 8; e += 256) {
      const unsigned char* src = P.w + (size_t)(e >> 6) * 64 * NN + UTILE(e) * 64 + w_lane;
      u32x4 V[4], D[4], T;
      V[0] = *(const GAS u32x4*)(src);
      V[1] = *(const GAS u32x4*)(src + 32 * NN);
      V[2] = *(const GAS u32x4*)(src + 32);
      V[3] = *(const GAS u32x4*)(src + 32 * NN + 32);
      u32 len;
      if (!FMT_B(e)) {
        const u32 nexc = encode_piece(V, choose_window(V), D, T, scr, lane);
        len = nexc > MAXEXC ? 5120u : 3076u + 2 * nexc;
      } else {
        u32x4 N[2];
        len = encode_b(V, choose_base(V), N, scr, L + LDS_LUT, lane);
        if (len) {
          D[0] = N[0]; D[1] = N[1];
          D[2] = *(LAS u32x4*)(scr + 4096 + vo); D[3] = *(LAS u32x4*)(scr + 5120 + vo);
        } else {
          len = 4112;
          D[0] = V[0]; D[1] = V[1]; D[3] = V[3];
          D[2] = lane == 32 ? u32x4{0x80000000u, 0u, 4112u, 0u} : V[2];
#pragma unroll
          for (int q = 0; q < 4; ++q) T[q] = __builtin_amdgcn_readlane(V[2][q], 32);
        }
      }
      unsigned char* dst = ws + WS_ST + (call & 1) * WB_SRC + (size_t)e * SLOT;
      ST16(dst, vo, D[0], 0); ST16(dst, vo, D[1], 1024); ST16(dst, vo, D[2], 2048);
      if (3072 + (vo & ~63u) < len) ST16(dst, vo, D[3], 3072);
      if (4096 + (vo & ~63u) < len) { unsigned char* d2 = dst + 4096; ST16(d2, vo, T, 0); }
      VMWAIT(0);
      {
        const u32 grp = e < U0 ? 0 : 1 + min(e >> 7, 2u);
        u32 old = 0;
        if (lane == 0) old = __hip_atomic_fetch_add((LAS u32*)(L + LDS_GRP + grp * 4), 1u, __ATOMIC_RELAXED, __HIP_MEMORY_SCOPE_WORKGROUP);
        if (__builtin_amdgcn_readfirstlane(old) == (grp == 0 ? 0 : grp == 1 ? 2 : grp == 2 ? 3 : 7)) {
          flag_store(fl0 + grp * 2048, call);
          if (lane < 48) flag_store(fl1 + grp * 2048, call);
        }
      }
      __builtin_amdgcn_s_setprio(0);
    }
    if (lane == 0) __hip_atomic_fetch_add((LAS u32*)(L + LDS_ENCD), 1u, __ATOMIC_RELAXED, __HIP_MEMORY_SCOPE_WORKGROUP);
    while (*(volatile LAS u32*)(L + LDS_ENCD) < 11u) __builtin_amdgcn_s_sleep(2);
    asm volatile("" ::: "memory");
  } else {
    START_BARRIER();
    call = __builtin_amdgcn_readfirstlane(cnt_old) + 1;
  }
  if (lane == 0) ag_store(cntp, call);
  if (wave == 3 && pr < 7 * NPW) {
    while (rt() - t_start < TPOLL) __builtin_amdgcn_s_sleep(8);
    const unsigned char* fs = ws + WS_FS + (1 + pr % 7) * 8192 + (lane & 31) * 64;
    const u64 t0 = rt();
    int g = 0;
#pragma unroll 1
    for (u32 spin = 1; g < 4; ++spin) {
      const u64 m = __ballot(flag_load(fs + min(g + (lane >> 5), 3) * 2048) >= call);
      if ((u32)m == ~0u) {
        g += m == ~0ull ? 2 : 1;
        __hip_atomic_store((LAS u32*)(L + LDS_STG), g == 1 ? U0 : min(32u << g, 512u), __ATOMIC_RELEASE, __HIP_MEMORY_SCOPE_WORKGROUP);
      } else if ((spin & 15) == 0 && (__builtin_amdgcn_readfirstlane(ag_load(ws + WS_TMO)) == call || rt() - t0 > DEADLINE_TICKS)) {
        ag_store(ws + WS_TMO, call);
        __hip_atomic_store((LAS u32*)(L + LDS_STG), 1024u, __ATOMIC_RELEASE, __HIP_MEMORY_SCOPE_WORKGROUP);
        break;
      }
      if (g < 2) __builtin_amdgcn_s_sleep(8); else __builtin_amdgcn_s_sleep(40);
    }
  }
  while (GO[0] == 0u) __builtin_amdgcn_s_sleep(4);

  if (wave < 4) {
    const int mymb = mt * 4 + wave;
    const u32 par = call & 1;
    const float scale = P.sx[0] * P.sw[0];
    const u32 b_lane = (u32)(size_t)L + ((lane >> 5) * 32 + ((lane & 15) >> 1)) * 32 + ((lane >> 4) & 1) * 16 + (lane & 1) * 8;
    f32x16 acc0 = {}, acc1 = {};
    u32 rv;
    u32x4 AL[8] = {}, AH[8] = {};
#define AWAIT() asm volatile("s_waitcnt vmcnt(0)" : "+v"(AL[0]), "+v"(AL[1]), "+v"(AL[2]), "+v"(AL[3]), "+v"(AL[4]), "+v"(AL[5]), "+v"(AL[6]), "+v"(AL[7]), \
                             "+v"(AH[0]), "+v"(AH[1]), "+v"(AH[2]), "+v"(AH[3]), "+v"(AH[4]), "+v"(AH[5]), "+v"(AH[6]), "+v"(AH[7]) :: "memory")
#define MMA(p, off) do { \
          i32x8 b0, b1; read_bfrag(b0, baddr, off); read_bfrag(b1, baddr, off + 2048); \
          const i32x8 a = {(int)AL[p].x, (int)AL[p].y, (int)AL[p].z, (int)AL[p].w, (int)AH[p].x, (int)AH[p].y, (int)AH[p].z, (int)AH[p].w}; \
          __builtin_amdgcn_sched_barrier(0); \
          acc0 = __builtin_amdgcn_mfma_scale_f32_32x32x64_f8f6f4(b0, a, acc0, 1, 0, 0, 0, 0, 0); \
          acc1 = __builtin_amdgcn_mfma_scale_f32_32x32x64_f8f6f4(b1, a, acc1, 1, 0, 0, 0, 0, 0); } while (0)
#define POLL(row, want) do { while (true) { \
          rv = READY[(row) * 8 + (lane & 7)]; \
          m = (u32)__ballot(rv >= (u32)(want)) & rem; \
          if (m) break; \
          __builtin_amdgcn_s_sleep(1); } \
        asm volatile("" ::: "memory"); rem &= ~m; } while (0)
    const unsigned char* xb = ws + WS_XB + par * XB_PAR + (size_t)mymb * 16384;
    {
      if (!wait_all<true>(ws + WS_FX + mymb * 128 + (lane & 1) * 64, call, ws, call)) BAD[0] = 1u;
#define LA(p) do { const unsigned char* s_ = xb + p * 2048; LD16S(AL[p], s_, vo, 0); LD16S(AH[p], s_, vo, 1024); } while (0)
      LA(0); LA(1); LA(2); LA(3); LA(4); LA(5); LA(6); LA(7);
#undef LA
      AWAIT();
      const u32 baddr = b_lane + LDS_OWN;
      u32 rem = 0xFFu, m;
      do {
        POLL(RR, 1);
#define STEP(p) if (m & (1u << p)) MMA(p, p * PIECE);
        STEP(0) STEP(1) STEP(2) STEP(3) STEP(4) STEP(5) STEP(6) STEP(7)
#undef STEP
      } while (rem);
    }
    if (!wait_all<true>(ws + WS_FX + ((1 + (lane & 7) % 7) * 16 + mymb) * 128 + ((lane >> 3) & 1) * 64, call, ws, call)) BAD[0] = 1u;
#define LA(p) do { const unsigned char* s_ = xb + p * 262144; LD16S(AL[p], s_, vo, 0); LD16S(AH[p], s_, vo, 1024); } while (0)
    LA(1); LA(2); LA(3); LA(4); LA(5); LA(6); LA(7);
#pragma unroll 1
    for (int i = 0; i < 8; ++i) {
      const int row = i & (RR - 1);
      const u32 baddr = b_lane + row * 7 * PIECE;
      xb += 2048;
      AWAIT();
      u32 rem = 0xFEu, m;
      do {
        POLL(row, i + 1);
#define STEP(p) if (m & (1u << p)) { MMA(p, (p - 1) * PIECE); if (i < 7) LA(p); }
        STEP(1) STEP(2) STEP(3) STEP(4) STEP(5) STEP(6) STEP(7)
#undef STEP
      } while (rem);
      asm volatile("s_waitcnt lgkmcnt(0)" ::: "memory");
      if (lane == 0) DONE[wave] = (u32)(i + 1);
    }
#undef LA
    AWAIT();
    const bool bad = __builtin_amdgcn_readfirstlane(rv) != 0u;
    u32x4 o[8];
    const float sc = bad ? __builtin_nanf("") : scale;
    acc0 *= sc; acc1 *= sc;
#pragma unroll
    for (int q = 0; q < 4; ++q)
#pragma unroll
      for (int e = 0; e < 4; ++e) {
        o[q][e] = __builtin_bit_cast(u32, __builtin_elementwise_maximum(acc0[4 * q + e], 0.f));
        o[4 + q][e] = __builtin_bit_cast(u32, __builtin_elementwise_maximum(acc1[4 * q + e], 0.f));
      }
    const unsigned char* ob = (const unsigned char*)(P.out + (size_t)mymb * 32 * NN + u * 64);
#define OST(val, voff) asm volatile("global_store_dwordx4 %0, %1, %2 sc1\n\ts_nop 1" :: "v"(voff), "v"(val), "s"(ob) : "memory")
    {
      LAS unsigned char* tb = L + LDS_OWN + wave * 8192;
      const int m = lane & 31, h = lane >> 5;
#pragma unroll
      for (int q = 0; q < 8; ++q) *(LAS u32x4*)(tb + m * 256 + (((2 * q + h) ^ (m & 15)) << 4)) = o[q];
      const int r0 = lane >> 4, ch = lane & 15;
#pragma unroll
      for (int t = 0; t < 8; ++t) o[t] = *(LAS u32x4*)(tb + (4 * t + r0) * 256 + ((ch ^ ((4 * t + r0) & 15)) << 4));
      u32 oo = (u32)r0 * NN * 4 + ch * 16;
#pragma unroll
      for (int t = 0; t < 8; ++t) { OST(o[t], oo); oo += 4 * NN * 4; }
    }
#undef OST
  } else {
    const int d = wave - 3;
    {
      const unsigned char* src = P.w + u * 64 + w_lane;
      for (int i = d - 1; i < 8; i += 7) {
        const unsigned char* s = src + (size_t)i * 64 * NN;
        LAS unsigned char* dst = L + LDS_OWN + i * PIECE;
        __builtin_amdgcn_global_load_lds((const GAS u32*)(s), (LAS u32*)(dst), 16, 0, 0);
        __builtin_amdgcn_global_load_lds((const GAS u32*)(s + 32 * NN), (LAS u32*)(dst + 1024), 16, 0, 0);
        __builtin_amdgcn_global_load_lds((const GAS u32*)(s + 32), (LAS u32*)(dst + 2048), 16, 0, 0);
        __builtin_amdgcn_global_load_lds((const GAS u32*)(s + 32 * NN + 32), (LAS u32*)(dst + 3072), 16, 0, 0);
      }
    }
    VMWAIT(0);
    if (lane == 0) { READY[RR * 8 + d - 1] = 1u; if (d == 1) READY[RR * 8 + 7] = 1u; }
    const unsigned char* fp = ws + WS_FW + (d * 64 + u) * 512;
    const unsigned char* src = ws + WS_WB + (call & 1) * WB_PAR + d * WB_SRC + (size_t)u * SLOT;
    u32 seen = 0;
    bool dead = false, pre = false;
    const unsigned char* tmop = ws + WS_TMO;
    const u32 l4 = lane * 4;
#define FMT_A(i_) (!FMT_B((i_) * 64 + UTILE(u)))
#define FETCH(i_, s_) do { u32 keep_; const u32 la_ = (u32)(size_t)L + ((((i_) & (RR - 1)) * 7 + d - 1) * PIECE); \
      asm volatile("s_mov_b32 %0, m0\n\ts_mov_b32 m0, %3\n\ts_nop 0\n\t" \
                   "global_load_lds_dwordx4 %1, %2 sc0 sc1\n\t" \
                   "global_load_lds_dwordx4 %1, %2 offset:1024 sc0 sc1\n\t" \
                   "global_load_lds_dwordx4 %1, %2 offset:2048 sc0 sc1\n\t" \
                   "s_mov_b32 m0, %0" : "=&s"(keep_) : "v"(vo), "s"(s_), "s"(la_) : "memory"); \
      if (FMT_A(i_)) \
        asm volatile("s_mov_b32 %0, m0\n\ts_mov_b32 m0, %3\n\ts_nop 0\n\tglobal_load_lds_dword %1, %2 offset:3072 sc0 sc1\n\ts_mov_b32 m0, %0" \
                     : "=&s"(keep_) : "v"(l4), "s"(s_), "s"(la_) : "memory"); } while (0)
#pragma unroll 1
    for (int i = 0; i < 8; ++i, src += 64 * SLOT) {
      if (!pre) {
        if (i >= RR) {
          while (!__all(DONE[lane & 3] + RR > (u32)i)) __builtin_amdgcn_s_sleep(2);
          asm volatile("" ::: "memory");
        }
        if (!((seen >> i) & 1) && !dead) {
          const u64 t0 = rt();
          for (u32 spin = 1;; ++spin) {
            seen = (u32)__ballot(flag_load(fp + max(lane & 7, i) * 64) >= call);
            if ((seen >> i) & 1) break;
            if ((spin & 63) == 0 && (__builtin_amdgcn_readfirstlane(ag_load(tmop)) == call || rt() - t0 > DEADLINE_TICKS)) { dead = true; break; }
            if (i < 7) __builtin_amdgcn_s_sleep(16);
          }
          if (dead) ag_store(ws + WS_TMO, call);
          asm volatile("" ::: "memory");
        }
        if (!dead) FETCH(i, src);
      }
      pre = !dead && i < 7 && ((seen >> (i + 1)) & 1) && __all(DONE[lane & 3] + RR > (u32)i + 1);
      if (pre) {
        const unsigned char* s2 = src + 64 * SLOT;
        FETCH(i + 1, s2);
        VMWAIT(3);
      } else {
        if (i == 7) dead = dead || __builtin_amdgcn_readfirstlane(ag_load(tmop)) == call;
        VMWAIT(0);
      }
      const int row = i & (RR - 1);
      if (dead) BAD[0] = 1u;
      else if (FMT_A(i)) unpack_piece(src, L + (row * 7 + d - 1) * PIECE, lane);
      else unpack_b(src, L + (row * 7 + d - 1) * PIECE, L + LDS_LUT + 256, lane);
      asm volatile("s_waitcnt lgkmcnt(0)" ::: "memory");
      READY[row * 8 + d] = (u32)(i + 1);
    }
#undef FETCH
#undef FMT_A
  }
}

}

extern "C" void kernel_launch(int rank, int nranks, ncclComm_t comm,
                              hipStream_t stream,
                              void* const* d_in, const int* in_sizes, int n_in,
                              void* d_out, int out_size,
                              const int* in_dtypes, int out_dtype,
                              void* const* d_out_all,
                              void* const* d_ws_all, size_t ws_bytes) {
  (void)comm; (void)in_dtypes; (void)out_dtype; (void)d_out_all;
  if (nranks != R || (unsigned)rank >= (unsigned)R || n_in != 4 || ws_bytes < WS_END ||
      in_sizes[0] != 4096 * KS || in_sizes[1] != KS * NN || out_size != MS * NN) {
    fprintf(stderr, "kernel_launch: unexpected problem shape\n");
    return;
  }
  HIP_CHECK(hipFuncSetAttribute((const void*)fused_kernel, hipFuncAttributeMaxDynamicSharedMemorySize, LDS_BYTES));
  Params P;
  P.x = (const unsigned char*)d_in[0];
  P.w = (const unsigned char*)d_in[1];
  P.sx = (const float*)d_in[2];
  P.sw = (const float*)d_in[3];
  P.out = (float*)d_out;
  for (int d = 0; d < R; ++d) P.peer[d] = (unsigned char*)d_ws_all[(rank + d) % R];
  P.rank = rank;
  P.pad = 0;
  hipLaunchKernelGGL(fused_kernel, dim3(NWG), dim3(NTHREADS), LDS_BYTES, stream, P);
  HIP_CHECK(hipGetLastError());
}
```
